```python
import numpy as np

kernel_name = "gpu_gemm_rs_m4096_k4096_n16384_e4m3e4m3_dqsilu"


ABSMAX_RTOL = 0.03

COMPUTE_DTYPE = "float32"
EPILOGUE = "dequant_silu"
IN_QUANT = None
OUT_QUANT = None
IN_FP8 = "e4m3e4m3"
IN_DTYPES = {"x": "e4m3", "w_mat": "e4m3",
             "scale_x": "float32", "scale_w": "float32"}
WORLD_SIZE = 8

def _snap_fp8(x, mant_bits, max_abs, min_normal):
    a = np.abs(x.astype(np.float64))
    _m, _e = np.frexp(a)
    q = np.where(a >= min_normal, np.ldexp(1.0, _e - 1 - mant_bits),
                 min_normal * 2.0 ** -mant_bits)
    snapped = np.minimum(np.round(a / q) * q, max_abs)
    return (np.sign(x) * snapped).astype(np.float32)

def setup_inputs():

    x = _snap_fp8(np.random.randn(4096, 4096).astype(np.float32), 3, 448.0, 2.0 ** -6)
    w_mat = _snap_fp8(np.random.randn(4096, 16384).astype(np.float32), 3, 448.0, 2.0 ** -6)
    scale_x = (0.02 * (1.0 + np.random.rand(1))).astype(np.float32)
    scale_w = (0.02 * (1.0 + np.random.rand(1))).astype(np.float32)
    return {"x": x, "w_mat": w_mat, "scale_x": scale_x, "scale_w": scale_w}

def reference(x, w_mat, scale_x, scale_w):
    acc = (x @ w_mat).astype(np.float32)
    y = (acc * (scale_x * scale_w)).astype(np.float32)
    return (y / (1.0 + np.exp(-np.clip(y, -60.0, 60.0)))).astype(np.float32)
```

```cpp
#include <hip/hip_runtime.h>
#include <rccl/rccl.h>
#include <cstdio>
#include <cstdint>

#define OUT_MODE 1
#define SEND_WIN 1
#define GATE_O 29
#define TIMING 0

#define HIP_CHECK(expr) do { hipError_t _e = (expr); if (_e != hipSuccess) { \
    fprintf(stderr, "HIP error %s at %s:%d\n", hipGetErrorString(_e), __FILE__, __LINE__); \
    return; } } while (0)

typedef unsigned u32x4 __attribute__((ext_vector_type(4)));
typedef int i32x4 __attribute__((ext_vector_type(4)));
typedef int i32x8 __attribute__((ext_vector_type(8)));
typedef float f32x4 __attribute__((ext_vector_type(4)));
typedef unsigned long long u64;

#define GAS __attribute__((address_space(1)))
#define LAS __attribute__((address_space(3)))

constexpr int NR = 8;
constexpr int NFULL = 16384;
constexpr int KSH = 512;
constexpr int NSH = 2048;
constexpr u64 DEADLINE = 100000000ull;

constexpr size_t OFF_CALL = 0;
constexpr size_t OFF_CNTX = 32768;
constexpr size_t OFF_CNTW = OFF_CNTX + 8192;
constexpr size_t OFF_SENT = OFF_CNTW + 2048;
constexpr size_t OFF_QCNT = OFF_SENT + 2048;
constexpr size_t OFF_TMO  = OFF_QCNT + 2048;
constexpr size_t OFF_DONE = OFF_TMO + 128;
constexpr size_t OFF_FLAG = 65536;
constexpr size_t OFF_TIME = 131072;
constexpr size_t OFF_A    = 1u << 20;
constexpr size_t OFF_B    = OFF_A + (32u << 20);
constexpr size_t OFF_O    = OFF_B + (16u << 20);
constexpr size_t SLOT     = 16384 + 256;
constexpr size_t WS_NEED  = OFF_O + (size_t)28 * 64 * SLOT;
constexpr int EXC_CAP = 14;

constexpr int STAGE = 49152;
constexpr int LDS_BYTES = 3 * STAGE + 256;

struct Params {
  const uint8_t* x;
  const uint8_t* w;
  const float* sx;
  const float* sw;
  float* out[NR];
  char* ws[NR];
  int rank;
  int pad;
};

__device__ __forceinline__ float as_f(unsigned u) { return __builtin_bit_cast(float, u); }
__device__ __forceinline__ unsigned as_u(float f) { return __builtin_bit_cast(unsigned, f); }
__device__ __forceinline__ void st16_sys(void* p, u32x4 v) {
  asm volatile("global_store_dwordx4 %0, %1, off sc0 sc1\n\ts_nop 1" :: "v"(p), "v"(v) : "memory");
}
__device__ __forceinline__ void st16_plain(void* p, u32x4 v) {
  asm volatile("global_store_dwordx4 %0, %1, off\n\ts_nop 1" :: "v"(p), "v"(v) : "memory");
}
#define WAIT_VM(n) asm volatile("s_waitcnt vmcnt(%0)" :: "i"(n) : "memory")

__device__ __forceinline__ unsigned ld_sys(const void* p) {
  return __hip_atomic_load((const GAS unsigned*)p, __ATOMIC_RELAXED, __HIP_MEMORY_SCOPE_SYSTEM);
}
__device__ __forceinline__ u64 ld_sys64(const void* p) {
  return __hip_atomic_load((const GAS u64*)p, __ATOMIC_RELAXED, __HIP_MEMORY_SCOPE_SYSTEM);
}
__device__ __forceinline__ unsigned ld_agent(const void* p) {
  return __hip_atomic_load((const GAS unsigned*)p, __ATOMIC_RELAXED, __HIP_MEMORY_SCOPE_AGENT);
}
__device__ __forceinline__ void st_agent(void* p, unsigned v) {
  __hip_atomic_store((GAS unsigned*)p, v, __ATOMIC_RELAXED, __HIP_MEMORY_SCOPE_AGENT);
}
__device__ __forceinline__ void add_sys(void* p, unsigned v) {
  __hip_atomic_fetch_add((GAS unsigned*)p, v, __ATOMIC_RELAXED, __HIP_MEMORY_SCOPE_SYSTEM);
}

__device__ __forceinline__ void wait_ge(const void* p, unsigned target, void* tmo, unsigned call) {
  u64 t0 = __builtin_amdgcn_s_memrealtime();
  while ((int)(ld_sys(p) - target) < 0) {
    if (ld_agent(tmo) == call || __builtin_amdgcn_s_memrealtime() - t0 > DEADLINE) {
      st_agent(tmo, call);
      break;
    }
    __builtin_amdgcn_s_sleep(4);
  }
}

__device__ __forceinline__ float silu_dq(float acc, float s) {
  float y = acc * s;
  float c = fminf(fmaxf(y, -60.0f), 60.0f);
  return __fdividef(y, 1.0f + __expf(-c));
}

__device__ __forceinline__ unsigned pack_z(float a) {
  return __builtin_bit_cast(unsigned, __builtin_amdgcn_ldexpf(a, -131));
}
__device__ __forceinline__ unsigned pack_misfit(float a) {
  const float zf = __builtin_amdgcn_ldexpf(a, -131);
  const unsigned back = __builtin_bit_cast(unsigned, __builtin_amdgcn_ldexpf(zf, 131));
  return (__builtin_bit_cast(unsigned, zf) & 0x7E000000u) | (back ^ __builtin_bit_cast(unsigned, a));
}

__device__ __forceinline__ int blk_dest(int r, int o) { return o < 28 ? (r + 1 + o % 7) & 7 : r; }
__device__ __forceinline__ int blk_sub(int o) { return o < 28 ? o / 7 : o - 28; }

#define STAMP(i) do {} while (0)

struct Chunk { unsigned r[16]; };

__device__ __forceinline__ void chunk_decode(int ci, bool& isw, int& blk, int& e) {
  if (ci < 112) { isw = false; blk = ci >> 4; e = ci & 15; }
  else if (ci < 368) { isw = true; blk = (ci - 112) >> 5; e = (ci - 112) & 31; }
  else { isw = false; blk = 7 + ((ci - 368) >> 4); e = (ci - 368) & 15; }
}

__device__ __forceinline__ void chunk_load(const Params& p, int d, int ci, int lane, Chunk& c) {
  bool isw; int blk, e;
  chunk_decode(ci, isw, blk, e);
  if (isw) {
    const int kb64 = e >> 2, G = e & 3;
    const uint8_t* src = p.w + (size_t)(kb64 * 64 + (lane >> 4) * 16) * NFULL + d * NSH + blk * 256 + G * 64 + (lane & 15) * 4;
#pragma unroll
    for (int r = 0; r < 16; ++r) c.r[r] = *(const GAS unsigned*)(src + (size_t)r * NFULL);
  } else {
    const int mb = blk_dest(d, blk) * 4 + blk_sub(blk);
    const int mt = e >> 1, half = e & 1;
    const uint8_t* src = p.x + (size_t)(mb * 128 + mt * 16 + (lane & 15)) * KSH + half * 256 + (lane >> 4) * 16;
#pragma unroll
    for (int u = 0; u < 4; ++u) {
      u32x4 v = *(const GAS u32x4*)(src + u * 64);
      c.r[4 * u] = v[0]; c.r[4 * u + 1] = v[1]; c.r[4 * u + 2] = v[2]; c.r[4 * u + 3] = v[3];
    }
  }
}

__global__ void __launch_bounds__(512, 2) fused_gemm_rs(Params p) {
  __shared__ __attribute__((aligned(16))) char smem[LDS_BYTES];
  const int tid = threadIdx.x;
  const int lane = tid & 63;
  const int wave = __builtin_amdgcn_readfirstlane(tid >> 6);
  const int rank = p.rank;
  const int j = blockIdx.x & 7;
  const int oi = blockIdx.x >> 3;
  const int o = 31 - oi;
  char* const myws = p.ws[rank];
  volatile LAS unsigned* sctl = (volatile LAS unsigned*)(LAS char*)(smem + 3 * STAGE);

  STAMP(0);
  Chunk cur;
  const bool sender = oi < 8;
  const int sd = (rank + 1 + oi) & 7;
  const int swave = j * 8 + wave;
  if (sender) chunk_load(p, sd, swave, lane, cur);
  if (tid == 0) {
    unsigned c = __hip_atomic_fetch_add((GAS unsigned*)(myws + OFF_CALL + (size_t)blockIdx.x * 128), 1u,
                                        __ATOMIC_RELAXED, __HIP_MEMORY_SCOPE_AGENT);
    sctl[0] = c + 1;
  }
  __syncthreads();
  const unsigned call = __builtin_amdgcn_readfirstlane(sctl[0]);
  const unsigned par = call & 1, nuse = (call + 1) >> 1;
  void* const tmo = myws + OFF_TMO;

  if (sender) {
    char* const dws = p.ws[sd];
    void* pend = nullptr;
    for (int it = 0; it < 12; ++it) {
      const int ci = swave + it * 64;
      bool isw; int blk, e;
      chunk_decode(ci, isw, blk, e);
      u32x4 v[4];
      char* dst; int dstep01, dstep23;
      void* cnt;
      if (isw) {
#pragma unroll
        for (int u = 0; u < 4; ++u) {
          const unsigned r0 = cur.r[4 * u], r1 = cur.r[4 * u + 1], r2 = cur.r[4 * u + 2], r3 = cur.r[4 * u + 3];
          const unsigned a0 = __builtin_amdgcn_perm(r1, r0, 0x05010400u), a1 = __builtin_amdgcn_perm(r1, r0, 0x07030602u);
          const unsigned b0 = __builtin_amdgcn_perm(r3, r2, 0x05010400u), b1 = __builtin_amdgcn_perm(r3, r2, 0x07030602u);
          v[0][u] = __builtin_amdgcn_perm(b0, a0, 0x05040100u);
          v[1][u] = __builtin_amdgcn_perm(b0, a0, 0x07060302u);
          v[2][u] = __builtin_amdgcn_perm(b1, a1, 0x05040100u);
          v[3][u] = __builtin_amdgcn_perm(b1, a1, 0x07060302u);
        }
        const int kb64 = e >> 2, G = e & 3;
        dst = dws + OFF_B + ((size_t)((par * 8 + blk) * 32 + rank * 4 + (kb64 >> 1))) * 32768 + G * 8192 + (kb64 & 1) * 1024 + lane * 16;
        dstep01 = 2048; dstep23 = 4096;
        cnt = dws + OFF_CNTW + (size_t)(par * 8 + blk) * 128;
      } else {
#pragma unroll
        for (int u = 0; u < 4; ++u) v[u] = u32x4{cur.r[4 * u], cur.r[4 * u + 1], cur.r[4 * u + 2], cur.r[4 * u + 3]};
        const int mt = e >> 1, half = e & 1;
        dst = dws + OFF_A + ((size_t)((par * 32 + blk) * 32 + rank * 4 + half * 2)) * 16384 + mt * 2048 + lane * 16;
        dstep01 = 1024; dstep23 = 16384;
        cnt = dws + OFF_CNTX + (size_t)(par * 32 + blk) * 128;
      }
      asm volatile("" : "+v"(v[0]), "+v"(v[1]), "+v"(v[2]), "+v"(v[3]));
      if (it < 11) chunk_load(p, sd, ci + 64, lane, cur);
      st16_sys(dst, v[0]);
      WAIT_VM(SEND_WIN - 1);
      st16_sys(dst + dstep01, v[1]);
      WAIT_VM(SEND_WIN - 1);
      st16_sys(dst + dstep23, v[2]);
      WAIT_VM(SEND_WIN - 1);
      st16_sys(dst + dstep23 + dstep01, v[3]);
      WAIT_VM(SEND_WIN - 1);
      if (lane == 0) add_sys(cnt, 1u);
    }
    WAIT_VM(0);
    (void)pend;
  }
  if (blockIdx.x == 64 && tid < NR) st_agent(myws + OFF_QCNT + (size_t)((par ^ 1) * 8 + tid) * 128, 0u);
  STAMP(1);

  if (tid == 0) {
    wait_ge(myws + OFF_CNTW + (size_t)(par * 8 + j) * 128, nuse * 256u, tmo, call);
    wait_ge(myws + OFF_CNTX + (size_t)(par * 32 + o) * 128, nuse * 128u, tmo, call);
  }
  __syncthreads();
  STAMP(2);

  const int wm = wave >> 2, G = wave & 3;
  f32x4 acc[4][4];
#pragma unroll
  for (int a = 0; a < 4; ++a)
#pragma unroll
    for (int b = 0; b < 4; ++b) acc[a][b] = f32x4{0.f, 0.f, 0.f, 0.f};
  {
    const char* ga = myws + OFF_A + (size_t)(par * 32 + o) * 32 * 16384 + wave * 2048 + lane * 16;
    const char* gb = myws + OFF_B + (size_t)(par * 8 + j) * 32 * 32768 + wave * 4096 + lane * 16;
    auto stage = [&](int ks, int buf) {
      LAS char* la = (LAS char*)smem + buf * STAGE + wave * 2048;
      LAS char* lb = (LAS char*)smem + buf * STAGE + 16384 + wave * 4096;
      const char* a = ga + (size_t)ks * 16384;
      const char* b = gb + (size_t)ks * 32768;
#pragma unroll
      for (int i = 0; i < 2; ++i)
        __builtin_amdgcn_global_load_lds((const GAS void*)(a + i * 1024), (LAS void*)(la + i * 1024), 16, 0, 17);
#pragma unroll
      for (int i = 0; i < 4; ++i)
        __builtin_amdgcn_global_load_lds((const GAS void*)(b + i * 1024), (LAS void*)(lb + i * 1024), 16, 0, 17);
    };
    auto comp = [&](int buf) {
      const LAS char* la = (const LAS char*)smem + buf * STAGE + wm * 8192 + lane * 16;
      const LAS char* lb = (const LAS char*)smem + buf * STAGE + 16384 + G * 8192 + lane * 16;
      i32x8 fa[4], fb[4];
#pragma unroll
      for (int t = 0; t < 4; ++t) {
        i32x4 lo = *(const LAS i32x4*)(lb + t * 2048), hi = *(const LAS i32x4*)(lb + t * 2048 + 1024);
        fb[t] = __builtin_shufflevector(lo, hi, 0, 1, 2, 3, 4, 5, 6, 7);
      }
#pragma unroll
      for (int m = 0; m < 4; ++m) {
        i32x4 lo = *(const LAS i32x4*)(la + m * 2048), hi = *(const LAS i32x4*)(la + m * 2048 + 1024);
        fa[m] = __builtin_shufflevector(lo, hi, 0, 1, 2, 3, 4, 5, 6, 7);
      }
#pragma unroll
      for (int m = 0; m < 4; ++m)
#pragma unroll
        for (int t = 0; t < 4; ++t)
          acc[m][t] = __builtin_amdgcn_mfma_scale_f32_16x16x128_f8f6f4(fa[m], fb[t], acc[m][t], 0, 0, 0, 127, 0, 127);
    };
    stage(0, 0);
    stage(1, 1);
    int buf = 0;
    for (int ks = 0; ks < 31; ++ks) {
      WAIT_VM(6);
      asm volatile("s_waitcnt lgkmcnt(0)" ::: "memory");
      __builtin_amdgcn_s_barrier();
      const int nb = buf == 0 ? 2 : buf - 1;
      if (ks + 2 < 32) stage(ks + 2, nb);
      comp(buf);
      buf = buf == 2 ? 0 : buf + 1;
    }
    WAIT_VM(0);
    asm volatile("s_waitcnt lgkmcnt(0)" ::: "memory");
    __builtin_amdgcn_s_barrier();
    comp(buf);
  }
  STAMP(3);

  const float scale = p.sx[0] * p.sw[0];
  const int dest = blk_dest(rank, o), sub = blk_sub(o);
  const int q = lane >> 4, c = lane & 15;

  if (dest == rank) {
    const bool bad = ld_agent(tmo) == call;
    float* dst = p.out[rank] + (size_t)(sub * 128 + wm * 64 + 4 * q) * NFULL + rank * NSH + j * 256 + G * 64 + 4 * c;
#pragma unroll
    for (int m = 0; m < 4; ++m)
#pragma unroll
      for (int r = 0; r < 4; ++r) {
        f32x4 v;
#pragma unroll
        for (int t = 0; t < 4; ++t) v[t] = bad ? __builtin_nanf("") : silu_dq(acc[m][t][r], scale);
        *(GAS f32x4*)(dst + (size_t)(m * 16 + r) * NFULL) = v;
      }
  } else {
    char* slot = myws + OFF_O + ((size_t)(o * 8 + j) * 8 + wave) * SLOT;
    const bool bad = ld_agent(tmo) == call;
    unsigned misfit = 0;
#pragma unroll
    for (int m = 0; m < 4; ++m)
#pragma unroll
      for (int t = 0; t < 4; ++t) {
#pragma unroll
        for (int r = 0; r < 4; ++r) {
          if (bad) acc[m][t][r] = __builtin_nanf("");
          misfit |= pack_misfit(acc[m][t][r]);
        }
        asm volatile("" : "+v"(acc[m][t]), "+v"(misfit));
      }
    unsigned nexc = 0;
    if (__builtin_amdgcn_ballot_w64(misfit != 0) != 0) {
#pragma unroll
      for (int m = 0; m < 4; ++m)
#pragma unroll
        for (int t = 0; t < 4; ++t) {
#pragma unroll
          for (int r = 0; r < 4; ++r) {
            const bool mis = pack_misfit(acc[m][t][r]) != 0;
            const u64 mask = __builtin_amdgcn_ballot_w64(mis);
            if (mask != 0) {
              const unsigned idx = nexc + __builtin_amdgcn_mbcnt_hi((unsigned)(mask >> 32), __builtin_amdgcn_mbcnt_lo((unsigned)mask, 0u));
              if (mis && idx < (unsigned)EXC_CAP) {
                const u64 ent = ((u64)((m * 16 + 4 * q + r) * 64 + 4 * c + t) << 32) | as_u(acc[m][t][r]);
                __hip_atomic_store((GAS u64*)(slot + 16384 + idx * 8), ent, __ATOMIC_RELAXED, __HIP_MEMORY_SCOPE_SYSTEM);
              }
              nexc += __builtin_popcountll(mask);
            }
          }
          asm volatile("" : "+v"(acc[m][t]));
        }
    }
    if (nexc > (unsigned)EXC_CAP) {
      nexc = 15;
#pragma unroll
      for (int m = 0; m < 4; ++m)
#pragma unroll
        for (int r = 0; r < 4; ++r) {
          f32x4 v = {acc[m][0][r], acc[m][1][r], acc[m][2][r], acc[m][3][r]};
          st16_sys(slot + (m * 4 + r) * 1024 + lane * 16, __builtin_bit_cast(u32x4, v));
        }
    } else {
      unsigned zd[3][4];
#pragma unroll
      for (int a = 0; a < 3; ++a)
#pragma unroll
        for (int t = 0; t < 4; ++t) zd[a][t] = pack_z(acc[3][t][1 + a]);
#pragma unroll
      for (int i = 0; i < 13; ++i) {
        u32x4 v;
#pragma unroll
        for (int t = 0; t < 4; ++t) {
          unsigned x;
          if (i < 12) x = zd[i >> 2][t] << (25 - 6 * (i & 3));
          else x = ((zd[0][t] << 1) & 0x02000000u) | ((zd[1][t] << 2) & 0x04000000u) | ((zd[2][t] << 3) & 0x08000000u) |
                   ((zd[0][t] >> 3) & 0x10000000u) | ((zd[1][t] >> 2) & 0x20000000u) | ((zd[2][t] >> 1) & 0x40000000u);
          v[t] = (pack_z(acc[i >> 2][t][i & 3]) & 0x81FFFFFFu) | (x & 0x7E000000u);
        }
        st16_sys(slot + i * 1024 + lane * 16, v);
      }
    }
    if (lane == 0) sctl[8 + wave] = nexc;
    WAIT_VM(0);
    __syncthreads();
    if (tid == 0) {
      unsigned cnts = 0;
#pragma unroll
      for (int w = 0; w < 8; ++w) cnts |= sctl[8 + w] << (4 * w);
      __hip_atomic_store((GAS u64*)(p.ws[dest & 7] + OFF_FLAG + (size_t)(rank * 32 + sub * 8 + j) * 128),
                         ((u64)call << 32) | cnts, __ATOMIC_RELAXED, __HIP_MEMORY_SCOPE_SYSTEM);
    }
  }
  STAMP(4);

  if (o < 7) {
    const int s = (rank + 1 + o) & 7;
    const int o0 = (rank - s - 1) & 7;
    const char* sws = p.ws[s];
    if (lane == 0) wait_ge(myws + OFF_CNTX + (size_t)(par * 32 + GATE_O) * 128, nuse * 128u, tmo, call);
    STAMP(5);
    for (int guard = 0; guard < 256; ++guard) {
      unsigned item = 0;
      if (lane == 0) item = __hip_atomic_fetch_add((GAS unsigned*)(myws + OFF_QCNT + (size_t)(par * 8 + s) * 128), 1u,
                                                   __ATOMIC_RELAXED, __HIP_MEMORY_SCOPE_AGENT);
      item = __builtin_amdgcn_readfirstlane(item);
      if (item >= 256u) break;
      const int tl = item >> 3, w = item & 7;
      const int tsub = tl >> 3, tj = tl & 7;
      unsigned cnts = 0;
      if (lane == 0) {
        const char* f = myws + OFF_FLAG + (size_t)(s * 32 + tl) * 128;
        u64 t0 = __builtin_amdgcn_s_memrealtime();
        for (;;) {
          const u64 fv = ld_sys64(f);
          cnts = (unsigned)fv;
          if ((int)((unsigned)(fv >> 32) - call) >= 0) break;
          if (ld_agent(tmo) == call || __builtin_amdgcn_s_memrealtime() - t0 > DEADLINE) { st_agent(tmo, call); break; }
          __builtin_amdgcn_s_sleep(4);
        }
      }
      const unsigned nexc = (__builtin_amdgcn_readfirstlane(cnts) >> (4 * w)) & 15u;
      const bool bad = ld_agent(tmo) == call;
      const char* slot = sws + OFF_O + ((size_t)((o0 + 7 * tsub) * 8 + tj) * 8 + w) * SLOT;
      float* tbase = p.out[rank] + (size_t)(tsub * 128 + (w >> 2) * 64) * NFULL + s * NSH + tj * 256 + (w & 3) * 64;
      float* dst = tbase + (size_t)(4 * q) * NFULL + 4 * c;
      const char* src = slot + lane * 16;
      if (bad) {
        const float qn = __builtin_nanf("");
#pragma unroll
        for (int i = 0; i < 16; ++i) *(GAS f32x4*)(dst + (size_t)((i >> 2) * 16 + (i & 3)) * NFULL) = f32x4{qn, qn, qn, qn};
      } else if (nexc == 15u) {
#pragma unroll
        for (int i = 0; i < 16; ++i) {
          u32x4 v;
          asm volatile("global_load_dwordx4 %0, %1, off sc0 sc1\n\ts_waitcnt vmcnt(0)" : "=&v"(v) : "v"(src + i * 1024) : "memory");
          f32x4 y;
#pragma unroll
          for (int t = 0; t < 4; ++t) y[t] = silu_dq(as_f(v[t]), scale);
          *(GAS f32x4*)(dst + (size_t)((i >> 2) * 16 + (i & 3)) * NFULL) = y;
        }
      } else {
        u32x4 ld[13];
        u32x4 dv[3] = {u32x4{0, 0, 0, 0}, u32x4{0, 0, 0, 0}, u32x4{0, 0, 0, 0}};
#define PULL_LD(i) asm volatile("global_load_dwordx4 %0, %1, off sc0 sc1" : "=&v"(ld[i]) : "v"(src + (i) * 1024) : "memory")
        WAIT_VM(0);
        PULL_LD(0);
        PULL_LD(1);
#pragma unroll
        for (int i = 0; i < 13; ++i) {
          constexpr int dummy = 0; (void)dummy;
          if (i == 0) asm volatile("s_waitcnt vmcnt(1)" : "+v"(ld[i]) :: "memory");
          else if (i == 1 || i == 12) asm volatile("s_waitcnt vmcnt(2)" : "+v"(ld[i]) :: "memory");
          else asm volatile("s_waitcnt vmcnt(3)" : "+v"(ld[i]) :: "memory");
          if (i + 2 < 13) PULL_LD(i + 2);
          f32x4 y;
#pragma unroll
          for (int t = 0; t < 4; ++t) {
            const unsigned cw = ld[i][t];
            y[t] = silu_dq(__builtin_amdgcn_ldexpf(__builtin_bit_cast(float, cw & 0x81FFFFFFu), 131), scale);
            if (i < 12) dv[i >> 2][t] |= ((cw >> 25) & 63u) << (6 * (i & 3));
            else {
              dv[0][t] |= ((cw >> 1) & 0x01000000u) | ((cw << 3) & 0x80000000u);
              dv[1][t] |= ((cw >> 2) & 0x01000000u) | ((cw << 2) & 0x80000000u);
              dv[2][t] |= ((cw >> 3) & 0x01000000u) | ((cw << 1) & 0x80000000u);
            }
          }
          st16_plain(dst + (size_t)((i >> 2) * 16 + (i & 3)) * NFULL, __builtin_bit_cast(u32x4, y));
        }
#pragma unroll
        for (int a = 0; a < 3; ++a) {
          f32x4 y;
#pragma unroll
          for (int t = 0; t < 4; ++t) y[t] = silu_dq(__builtin_amdgcn_ldexpf(as_f(dv[a][t]), 131), scale);
          st16_plain(dst + (size_t)(48 + 1 + a) * NFULL, __builtin_bit_cast(u32x4, y));
        }
        if (nexc != 0) {
          u64 ent = 0;
          if ((unsigned)lane < nexc) ent = ld_sys64(slot + 16384 + lane * 8);
          WAIT_VM(0);
          if ((unsigned)lane < nexc) {
            const unsigned pos = (unsigned)(ent >> 32) & 4095u;
            tbase[(size_t)(pos >> 6) * NFULL + (pos & 63u)] = silu_dq(__builtin_bit_cast(float, (unsigned)ent), scale);
          }
        }
      }
    }
    STAMP(6);
  }
}

extern "C" void kernel_launch(int rank, int nranks, ncclComm_t comm,
                              hipStream_t stream,
                              void* const* d_in, const int* in_sizes, int n_in,
                              void* d_out, int out_size,
                              const int* in_dtypes, int out_dtype,
                              void* const* d_out_all,
                              void* const* d_ws_all, size_t ws_bytes) {
  (void)comm; (void)in_dtypes; (void)out_dtype; (void)d_out;
  if (nranks != NR || n_in != 4 || ws_bytes < WS_NEED || (unsigned)rank >= (unsigned)NR ||
      in_sizes[0] != 4096 * KSH || in_sizes[1] != KSH * NFULL || out_size != 512 * NFULL) {
    fprintf(stderr, "kernel_launch: unexpected problem shape\n");
    return;
  }
  Params p;
  p.x = (const uint8_t*)d_in[0];
  p.w = (const uint8_t*)d_in[1];
  p.sx = (const float*)d_in[2];
  p.sw = (const float*)d_in[3];
  for (int i = 0; i < NR; ++i) {
    p.out[i] = (float*)d_out_all[i];
    p.ws[i] = (char*)d_ws_all[i];
  }
  p.rank = rank;
  p.pad = 0;
  hipLaunchKernelGGL(fused_gemm_rs, dim3(256), dim3(512), 0, stream, p);
  HIP_CHECK(hipGetLastError());
}
```

```cpp
#include <hip/hip_runtime.h>
#include <rccl/rccl.h>
#include <cstdio>
#include <cstdint>

#define SEND_WIN 2
#define SEND_WAVES 4
#define PULL_WIN 4
#define GATE_O 26
#define POLL_SLEEP 4
#define TIMING 0

#define HIP_CHECK(expr) do { hipError_t _e = (expr); if (_e != hipSuccess) { \
    fprintf(stderr, "HIP error %s at %s:%d\n", hipGetErrorString(_e), __FILE__, __LINE__); \
    return; } } while (0)

typedef unsigned u32x4 __attribute__((ext_vector_type(4)));
typedef int i32x4 __attribute__((ext_vector_type(4)));
typedef int i32x8 __attribute__((ext_vector_type(8)));
typedef float f32x4 __attribute__((ext_vector_type(4)));
typedef unsigned long long u64;

#define GAS __attribute__((address_space(1)))
#define LAS __attribute__((address_space(3)))

constexpr int NR = 8;
constexpr int NFULL = 16384;
constexpr int KSH = 512;
constexpr int NSH = 2048;
constexpr u64 DEADLINE = 100000000ull;

constexpr size_t OFF_CALL = 0;
constexpr size_t OFF_PIECE = 32768;
constexpr size_t OFF_SENT = OFF_PIECE + 8192;
constexpr size_t OFF_QCNT = OFF_SENT + 2048;
constexpr size_t OFF_TMO  = OFF_QCNT + 2048;
constexpr size_t OFF_DUMMY = OFF_TMO + 128;
constexpr size_t OFF_FLAG = 65536;
constexpr size_t OFF_AGG  = 98304;
constexpr size_t OFF_TIME = 131072;
constexpr size_t OFF_A    = 1u << 20;
constexpr size_t OFF_B    = OFF_A + (32u << 20);
constexpr size_t OFF_O    = OFF_B + (16u << 20);
constexpr size_t SLOT     = 16384 + 256;
constexpr size_t WS_NEED  = OFF_O + (size_t)28 * 64 * SLOT;
constexpr int EXC_CAP = 14;

constexpr int STAGE = 49152;
constexpr int LDS_BYTES = 3 * STAGE + 256;

struct Params {
  const uint8_t* x;
  const uint8_t* w;
  const float* sx;
  const float* sw;
  float* out[NR];
  char* ws[NR];
  int rank;
  int pad;
};

__device__ __forceinline__ float as_f(unsigned u) { return __builtin_bit_cast(float, u); }
__device__ __forceinline__ unsigned as_u(float f) { return __builtin_bit_cast(unsigned, f); }
__device__ __forceinline__ void st16_sys(const void* base, unsigned off, u32x4 v) {
  asm volatile("global_store_dwordx4 %0, %1, %2 sc0 sc1\n\ts_nop 1" :: "v"(off), "v"(v), "s"(base) : "memory");
}
#define OUT_ST "sc1"
__device__ __forceinline__ void st16_out(const void* base, unsigned off, u32x4 v) {
  asm volatile("global_store_dwordx4 %0, %1, %2 " OUT_ST "\n\ts_nop 1" :: "v"(off), "v"(v), "s"(base) : "memory");
}
__device__ __forceinline__ void ticket_issue(unsigned& t, const void* ctr) {
  u64 sv;
  asm volatile("s_mov_b64 %1, exec\n\ts_mov_b64 exec, 1\n\tglobal_atomic_add %0, %2, %3, %4 sc0\n\ts_mov_b64 exec, %1"
               : "=&v"(t), "=&s"(sv) : "v"(0u), "v"(1u), "s"(ctr) : "memory");
}
#define WAIT_VM(n) asm volatile("s_waitcnt vmcnt(%0)" :: "i"(n) : "memory")

__device__ __forceinline__ unsigned ld_sys(const void* p) {
  return __hip_atomic_load((const GAS unsigned*)p, __ATOMIC_RELAXED, __HIP_MEMORY_SCOPE_SYSTEM);
}
__device__ __forceinline__ u64 ld_sys64(const void* p) {
  return __hip_atomic_load((const GAS u64*)p, __ATOMIC_RELAXED, __HIP_MEMORY_SCOPE_SYSTEM);
}
__device__ __forceinline__ unsigned ld_agent(const void* p) {
  return __hip_atomic_load((const GAS unsigned*)p, __ATOMIC_RELAXED, __HIP_MEMORY_SCOPE_AGENT);
}
__device__ __forceinline__ void st_agent(void* p, unsigned v) {
  __hip_atomic_store((GAS unsigned*)p, v, __ATOMIC_RELAXED, __HIP_MEMORY_SCOPE_AGENT);
}
__device__ __forceinline__ void st_sys(void* p, unsigned v) {
  __hip_atomic_store((GAS unsigned*)p, v, __ATOMIC_RELAXED, __HIP_MEMORY_SCOPE_SYSTEM);
}

__device__ __forceinline__ void wait_ge(const void* p, unsigned target, void* tmo, unsigned call) {
  u64 t0 = __builtin_amdgcn_s_memrealtime();
  while ((int)(ld_sys(p) - target) < 0) {
    if (ld_agent(tmo) == call || __builtin_amdgcn_s_memrealtime() - t0 > DEADLINE) {
      st_agent(tmo, call);
      break;
    }
    __builtin_amdgcn_s_sleep(POLL_SLEEP);
  }
}

__device__ __forceinline__ float silu_dq(float acc, float s) {
  float y = acc * s;
  float c = fminf(fmaxf(y, -60.0f), 60.0f);
  return y * __builtin_amdgcn_rcpf(1.0f + __builtin_amdgcn_exp2f(c * -1.44269504088896f));
}

__device__ __forceinline__ unsigned pack_z(float a) {
  return __builtin_bit_cast(unsigned, __builtin_amdgcn_ldexpf(a, -131));
}
__device__ __forceinline__ unsigned pack_misfit(float a) {
  const float zf = __builtin_amdgcn_ldexpf(a, -131);
  const unsigned back = __builtin_bit_cast(unsigned, __builtin_amdgcn_ldexpf(zf, 131));
  return (__builtin_bit_cast(unsigned, zf) & 0x7E000000u) | (back ^ __builtin_bit_cast(unsigned, a));
}

__device__ __forceinline__ int blk_dest(int r, int o) { return o < 28 ? (r + 1 + o % 7) & 7 : r; }
__device__ __forceinline__ int blk_sub(int o) { return o < 28 ? o / 7 : o - 28; }

#define STAMP(i) do {} while (0)
#define STAMP2(i) do {} while (0)

struct Chunk { unsigned r[16]; };
constexpr int NCHUNK = 768;

__device__ __forceinline__ void chunk_decode(int ci, bool& isw, int& blk, int& e) {
  if (ci < 112) { isw = false; blk = ci >> 4; e = ci & 15; }
  else if (ci < 368) { isw = true; blk = (ci - 112) >> 5; e = (ci - 112) & 31; }
  else { isw = false; blk = 7 + ((ci - 368) >> 4); e = (ci - 368) & 15; }
}

__device__ __forceinline__ void chunk_load(const Params& p, int d, int ci, int lane, Chunk& c) {
  bool isw; int blk, e;
  chunk_decode(ci, isw, blk, e);
  if (isw) {
    const __amdgpu_buffer_rsrc_t rs = __builtin_amdgcn_make_buffer_rsrc((void*)p.w, 0, KSH * NFULL, 0x00020000);
    const int kb64 = e >> 2, G = e & 3;
    const unsigned vo = (lane >> 4) * 16 * NFULL + (lane & 15) * 4;
    const unsigned so = kb64 * 64 * NFULL + d * NSH + blk * 256 + G * 64;
#pragma unroll
    for (int r = 0; r < 16; ++r) c.r[r] = __builtin_amdgcn_raw_buffer_load_b32(rs, vo, so + r * NFULL, 0);
  } else {
    const __amdgpu_buffer_rsrc_t rs = __builtin_amdgcn_make_buffer_rsrc((void*)p.x, 0, 4096 * KSH, 0x00020000);
    const int mb = blk_dest(d, blk) * 4 + blk_sub(blk);
    const int mt = e >> 1, half = e & 1;
    const unsigned vo = (lane & 15) * KSH + (lane >> 4) * 16;
    const unsigned so = (mb * 128 + mt * 16) * KSH + half * 256;
#pragma unroll
    for (int u = 0; u < 4; ++u) {
      u32x4 v = __builtin_amdgcn_raw_buffer_load_b128(rs, vo, so + u * 64, 0);
      c.r[4 * u] = v[0]; c.r[4 * u + 1] = v[1]; c.r[4 * u + 2] = v[2]; c.r[4 * u + 3] = v[3];
    }
  }
}

constexpr int RING = 4;
constexpr int LSLOT = 16384 + 256;

__device__ __forceinline__ unsigned lds_peek(LAS unsigned* p) {
  unsigned v;
  asm volatile("ds_read_b32 %0, %1\n\ts_waitcnt lgkmcnt(0)" : "=&v"(v) : "v"(p) : "memory");
  return __builtin_amdgcn_readfirstlane(v);
}
__device__ __forceinline__ u64 uni64(u64 v) {
  return ((u64)(unsigned)__builtin_amdgcn_readfirstlane((unsigned)(v >> 32)) << 32) | (u64)(unsigned)__builtin_amdgcn_readfirstlane((unsigned)v);
}

__global__ void __launch_bounds__(512, 2) fused_gemm_rs(Params p) {
  __shared__ __attribute__((aligned(16))) char smem[LDS_BYTES];
  const int tid = threadIdx.x;
  const int lane = tid & 63;
  const int wave = __builtin_amdgcn_readfirstlane(tid >> 6);
  const int rank = p.rank;
  const int j = blockIdx.x & 7;
  const int grp = blockIdx.x >> 3;
  char* const myws = p.ws[rank];
  volatile LAS unsigned* sctl = (volatile LAS unsigned*)(LAS char*)(smem + 3 * STAGE);

  STAMP(0);
  Chunk cur;
  const bool sender = grp < 8 && wave < SEND_WAVES;
  const int peer = (rank + 1 + grp) & 7;
  constexpr int NSW = 8 * SEND_WAVES;
  const int swave = j * SEND_WAVES + wave;
  if (sender) chunk_load(p, peer, swave, lane, cur);
  if (tid == 0) {
    unsigned c = __hip_atomic_fetch_add((GAS unsigned*)(myws + OFF_CALL + (size_t)blockIdx.x * 128), 1u,
                                        __ATOMIC_RELAXED, __HIP_MEMORY_SCOPE_AGENT);
    sctl[0] = c + 1;
  }
  if (tid >= 16 && tid < 48) sctl[tid] = 0;
  __syncthreads();
  const unsigned call = __builtin_amdgcn_readfirstlane(sctl[0]);
  const unsigned par = call & 1;
  void* const tmo = myws + OFF_TMO;
  const float scale = p.sx[0] * p.sw[0];

  if (sender) {
    char* const dws = p.ws[peer];
    const __amdgpu_buffer_rsrc_t drs = __builtin_amdgcn_make_buffer_rsrc(dws, 0, (int)WS_NEED, 0x00020000);
    const void* tick = myws + OFF_SENT + (size_t)(par * 8 + peer) * 128;
    const char* pcnt = myws + OFF_DUMMY;
    char* pflag = nullptr;
    unsigned ptarget = 0;
    int ci = swave, cnext = swave + NSW;
    for (int guard = 0; guard < NCHUNK && ci < NCHUNK; ++guard) {
      bool isw; int blk, e;
      chunk_decode(ci, isw, blk, e);
      u32x4 v[4];
      unsigned dst, dstep01, dstep23;
      if (isw) {
#pragma unroll
        for (int u = 0; u < 4; ++u) {
          const unsigned r0 = cur.r[4 * u], r1 = cur.r[4 * u + 1], r2 = cur.r[4 * u + 2], r3 = cur.r[4 * u + 3];
          const unsigned a0 = __builtin_amdgcn_perm(r1, r0, 0x05010400u), a1 = __builtin_amdgcn_perm(r1, r0, 0x07030602u);
          const unsigned b0 = __builtin_amdgcn_perm(r3, r2, 0x05010400u), b1 = __builtin_amdgcn_perm(r3, r2, 0x07030602u);
          v[0][u] = __builtin_amdgcn_perm(b0, a0, 0x05040100u);
          v[1][u] = __builtin_amdgcn_perm(b0, a0, 0x07060302u);
          v[2][u] = __builtin_amdgcn_perm(b1, a1, 0x05040100u);
          v[3][u] = __builtin_amdgcn_perm(b1, a1, 0x07060302u);
        }
        const int kb64 = e >> 2, G = e & 3;
        dst = (unsigned)OFF_B + ((par * 8 + blk) * 32 + rank * 4 + (kb64 >> 1)) * 32768u + G * 8192 + (kb64 & 1) * 1024;
        dstep01 = 2048; dstep23 = 4096;
      } else {
#pragma unroll
        for (int u = 0; u < 4; ++u) v[u] = u32x4{cur.r[4 * u], cur.r[4 * u + 1], cur.r[4 * u + 2], cur.r[4 * u + 3]};
        const int mt = e >> 1, half = e & 1;
        dst = (unsigned)OFF_A + ((par * 32 + blk) * 32 + rank * 4 + half * 2) * 16384u + mt * 2048;
        dstep01 = 1024; dstep23 = 16384;
      }
      asm volatile("" : "+v"(v[0]), "+v"(v[1]), "+v"(v[2]), "+v"(v[3]) :: "memory");
      unsigned tk, ag;
      ticket_issue(tk, tick);
      if (cnext < NCHUNK) chunk_load(p, peer, cnext, lane, cur);
      asm volatile("" ::: "memory");
#define SEND_STEP(k, off) do { \
        __builtin_amdgcn_raw_buffer_store_b128(v[k], drs, lane * 16, (off), 17); \
        if ((k) == 2 * SEND_WIN - 3) { \
          asm volatile("s_waitcnt vmcnt(%1)" : "+v"(ag) : "i"(SEND_WIN - 1) : "memory"); \
          if (__builtin_amdgcn_readfirstlane(ag) + 1 == ptarget && pflag != nullptr && lane == 0) st_sys(pflag, call); \
          asm volatile("" ::: "memory"); \
        } else WAIT_VM(SEND_WIN - 1); \
        if ((k) == SEND_WIN - 2) ticket_issue(ag, pcnt); } while (0)
      SEND_STEP(0, dst);
      SEND_STEP(1, dst + dstep01);
      SEND_STEP(2, dst + dstep23);
      SEND_STEP(3, dst + dstep23 + dstep01);
      asm volatile("" : "+v"(tk) :: "memory");
      const int piece = isw ? 32 + blk : blk;
      pcnt = myws + OFF_AGG + (size_t)(peer * 40 + piece) * 64;
      pflag = dws + OFF_PIECE + (size_t)piece * 128 + rank * 4;
      ptarget = call * (isw ? 32u : 16u);
      ci = cnext;
      cnext = 2 * NSW + (int)__builtin_amdgcn_readfirstlane(tk);
    }
    WAIT_VM(0);
    {
      unsigned ag;
      ticket_issue(ag, pcnt);
      asm volatile("s_waitcnt vmcnt(0)" : "+v"(ag) :: "memory");
      if (__builtin_amdgcn_readfirstlane(ag) + 1 == ptarget && pflag != nullptr && lane == 0) st_sys(pflag, call);
    }
  }
  if (blockIdx.x == 64 && tid < 2 * NR)
    st_agent(myws + (tid < NR ? OFF_QCNT : OFF_SENT) + (size_t)((par ^ 1) * 8 + (tid & 7)) * 128, 0u);
  STAMP(1);

  if (grp >= 7) {
    const int wm = wave >> 2, G = wave & 3;
    const int npass = (grp >= 8 && grp < 15) ? 2 : 1;
    for (int pass = 0; pass < npass; ++pass) {
      const int o = grp == 7 ? 24 : (pass == 0 ? grp - 8 : grp - 8 + 25);
      if (tid < 16) wait_ge(myws + OFF_PIECE + (size_t)(tid < 8 ? o : 32 + j) * 128 + (tid & 7) * 4, call, tmo, call);
      __syncthreads();
      STAMP(2 + 3 * pass);
      f32x4 acc[4][4];
#pragma unroll
      for (int a = 0; a < 4; ++a)
#pragma unroll
        for (int b = 0; b < 4; ++b) acc[a][b] = f32x4{0.f, 0.f, 0.f, 0.f};
      {
        const char* ga = myws + OFF_A + (size_t)(par * 32 + o) * 32 * 16384 + wave * 2048 + lane * 16;
        const char* gb = myws + OFF_B + (size_t)(par * 8 + j) * 32 * 32768 + wave * 4096 + lane * 16;
        auto stage = [&](int ks, int buf) {
          LAS char* la = (LAS char*)smem + buf * STAGE + wave * 2048;
          LAS char* lb = (LAS char*)smem + buf * STAGE + 16384 + wave * 4096;
          const char* a = ga + (size_t)ks * 16384;
          const char* b = gb + (size_t)ks * 32768;
#pragma unroll
          for (int i = 0; i < 2; ++i)
            __builtin_amdgcn_global_load_lds((const GAS void*)(a + i * 1024), (LAS void*)(la + i * 1024), 16, 0, 17);
#pragma unroll
          for (int i = 0; i < 4; ++i)
            __builtin_amdgcn_global_load_lds((const GAS void*)(b + i * 1024), (LAS void*)(lb + i * 1024), 16, 0, 17);
        };
        auto comp = [&](int buf) {
          const LAS char* la = (const LAS char*)smem + buf * STAGE + wm * 8192 + lane * 16;
          const LAS char* lb = (const LAS char*)smem + buf * STAGE + 16384 + G * 8192 + lane * 16;
          i32x8 fa[4], fb[4];
#pragma unroll
          for (int t = 0; t < 4; ++t) {
            i32x4 lo = *(const LAS i32x4*)(lb + t * 2048), hi = *(const LAS i32x4*)(lb + t * 2048 + 1024);
            fb[t] = __builtin_shufflevector(lo, hi, 0, 1, 2, 3, 4, 5, 6, 7);
          }
#pragma unroll
          for (int m = 0; m < 4; ++m) {
            i32x4 lo = *(const LAS i32x4*)(la + m * 2048), hi = *(const LAS i32x4*)(la + m * 2048 + 1024);
            fa[m] = __builtin_shufflevector(lo, hi, 0, 1, 2, 3, 4, 5, 6, 7);
          }
#pragma unroll
          for (int m = 0; m < 4; ++m)
#pragma unroll
            for (int t = 0; t < 4; ++t)
              acc[m][t] = __builtin_amdgcn_mfma_scale_f32_16x16x128_f8f6f4(fa[m], fb[t], acc[m][t], 0, 0, 0, 127, 0, 127);
        };
        stage(0, 0);
        stage(1, 1);
        int buf = 0;
        for (int ks = 0; ks < 31; ++ks) {
          WAIT_VM(6);
          asm volatile("s_waitcnt lgkmcnt(0)" ::: "memory");
          __builtin_amdgcn_s_barrier();
          const int nb = buf == 0 ? 2 : buf - 1;
          if (ks + 2 < 32) stage(ks + 2, nb);
          comp(buf);
          buf = buf == 2 ? 0 : buf + 1;
        }
        WAIT_VM(0);
        asm volatile("s_waitcnt lgkmcnt(0)" ::: "memory");
        __builtin_amdgcn_s_barrier();
        comp(buf);
      }
      STAMP(3 + 3 * pass);

      const int dest = blk_dest(rank, o), sub = blk_sub(o);
      if (__builtin_amdgcn_readfirstlane(ld_agent(tmo)) == call) {
#pragma unroll
        for (int a = 0; a < 4; ++a)
#pragma unroll
          for (int b = 0; b < 4; ++b) acc[a][b] = acc[a][b] * __builtin_nanf("");
      }
      int ln = lane;
      asm volatile("" : "+v"(ln));
      const int q = ln >> 4, c = ln & 15;
      if (dest == rank) {
        const float* dst = p.out[rank] + (size_t)(sub * 128 + wm * 64) * NFULL + rank * NSH + j * 256 + G * 64;
        const unsigned vo = (4 * q * NFULL + 4 * c) * 4;
#pragma unroll
        for (int m = 0; m < 4; ++m)
#pragma unroll
          for (int r = 0; r < 4; ++r) {
            f32x4 v;
#pragma unroll
            for (int t = 0; t < 4; ++t) v[t] = silu_dq(acc[m][t][r], scale);
            st16_out(dst + (size_t)(m * 16 + r) * NFULL, vo, __builtin_bit_cast(u32x4, v));
          }
      } else {
        char* slot = myws + OFF_O + ((size_t)(o * 8 + j) * 8 + wave) * SLOT;
        unsigned misfit = 0;
#pragma unroll
        for (int m = 0; m < 4; ++m)
#pragma unroll
          for (int t = 0; t < 4; ++t) {
#pragma unroll
            for (int r = 0; r < 4; ++r) {
              misfit |= pack_misfit(acc[m][t][r]);
            }
            asm volatile("" : "+v"(acc[m][t]), "+v"(misfit));
          }
        unsigned nexc = 0;
        if (__builtin_amdgcn_ballot_w64(misfit != 0) != 0) {
#pragma unroll
          for (int m = 0; m < 4; ++m)
#pragma unroll
            for (int t = 0; t < 4; ++t) {
#pragma unroll
              for (int r = 0; r < 4; ++r) {
                const bool mis = pack_misfit(acc[m][t][r]) != 0;
                const u64 mask = __builtin_amdgcn_ballot_w64(mis);
                if (mask != 0) {
                  const unsigned idx = nexc + __builtin_amdgcn_mbcnt_hi((unsigned)(mask >> 32), __builtin_amdgcn_mbcnt_lo((unsigned)mask, 0u));
                  if (mis && idx < (unsigned)EXC_CAP) {
                    const u64 ent = ((u64)((m * 16 + 4 * q + r) * 64 + 4 * c + t) << 32) | as_u(acc[m][t][r]);
                    __hip_atomic_store((GAS u64*)(slot + 16384 + idx * 8), ent, __ATOMIC_RELAXED, __HIP_MEMORY_SCOPE_SYSTEM);
                  }
                  nexc += __builtin_popcountll(mask);
                }
              }
              asm volatile("" : "+v"(acc[m][t]));
            }
        }
        if (nexc > (unsigned)EXC_CAP) {
          nexc = 15;
#pragma unroll
          for (int m = 0; m < 4; ++m)
#pragma unroll
            for (int r = 0; r < 4; ++r) {
              f32x4 v = {acc[m][0][r], acc[m][1][r], acc[m][2][r], acc[m][3][r]};
              st16_sys(slot + (m * 4 + r) * 1024, ln * 16, __builtin_bit_cast(u32x4, v));
            }
        } else {
          unsigned zd[3][4];
#pragma unroll
          for (int a = 0; a < 3; ++a)
#pragma unroll
            for (int t = 0; t < 4; ++t) zd[a][t] = pack_z(acc[3][t][1 + a]);
#pragma unroll
          for (int i = 0; i < 13; ++i) {
            u32x4 v;
#pragma unroll
            for (int t = 0; t < 4; ++t) {
              unsigned x;
              if (i < 12) x = zd[i >> 2][t] << (25 - 6 * (i & 3));
              else x = ((zd[0][t] << 1) & 0x02000000u) | ((zd[1][t] << 2) & 0x04000000u) | ((zd[2][t] << 3) & 0x08000000u) |
                       ((zd[0][t] >> 3) & 0x10000000u) | ((zd[1][t] >> 2) & 0x20000000u) | ((zd[2][t] >> 1) & 0x40000000u);
              v[t] = (pack_z(acc[i >> 2][t][i & 3]) & 0x81FFFFFFu) | (x & 0x7E000000u);
            }
            st16_sys(slot + i * 1024, ln * 16, v);
          }
        }
        if (lane == 0) sctl[8 + wave] = nexc;
        WAIT_VM(0);
        __syncthreads();
        if (tid == 0) {
          unsigned cnts = 0;
#pragma unroll
          for (int w = 0; w < 8; ++w) cnts |= sctl[8 + w] << (4 * w);
          __hip_atomic_store((GAS u64*)(p.ws[dest & 7] + OFF_FLAG + (size_t)(rank * 32 + sub * 8 + j) * 128),
                             ((u64)call << 32) | cnts, __ATOMIC_RELAXED, __HIP_MEMORY_SCOPE_SYSTEM);
        }
      }
      STAMP(4 + 3 * pass);
    }
  } else {
    const int s = peer;
    const int ring = wave >= 4 ? (wave & 1) : (wave >> 1);
    LAS char* const rbase = (LAS char*)smem + ring * (RING * LSLOT);
    volatile LAS unsigned* const full = sctl + 16 + ring * 16;
    volatile LAS unsigned* const info = full + 4;
    LAS unsigned* const done = (LAS unsigned*)(full + 8);
    if (wave == 4 || wave == 5) {
      const int o0 = (rank - s - 1) & 7;
      const char* sws = p.ws[s];
      const void* tick = myws + OFF_QCNT + (size_t)(par * 8 + s) * 128;
      unsigned item, inext;
      ticket_issue(item, tick);
      ticket_issue(inext, tick);
      asm volatile("s_waitcnt vmcnt(0)" : "+v"(item), "+v"(inext) :: "memory");
      item = __builtin_amdgcn_readfirstlane(item);
      inext = __builtin_amdgcn_readfirstlane(inext);
      if (lane == 0) wait_ge(myws + OFF_PIECE + (size_t)GATE_O * 128 + s * 4, call, tmo, call);
      STAMP2(2);
      u64 fpre = 0;
      int pend = -1;
      unsigned pinfo = 0;
      unsigned seq = 0;
#define PUBLISH() do { if (pend >= 0) { if (lane == 0) { info[pend] = pinfo; asm volatile("s_waitcnt lgkmcnt(0)" ::: "memory"); full[pend] = seq; } pend = -1; } } while (0)
      for (; seq < 257u; ) {
        const int sl = seq % RING;
        const bool stop = item >= 256u;
        const int tl = (item >> 3) & 31, w = item & 7;
        bool bad = false;
        if (stop || (int)((unsigned)(fpre >> 32) - call) < 0 || (int)(lds_peek(done + sl) - 3 * (seq / RING)) < 0) {
          WAIT_VM(0);
          PUBLISH();
          u64 t0 = __builtin_amdgcn_s_memrealtime();
          if (!stop) {
            const char* f = myws + OFF_FLAG + (size_t)(s * 32 + tl) * 128;
            for (;;) {
              fpre = uni64(ld_sys64(f));
              if ((int)((unsigned)(fpre >> 32) - call) >= 0) break;
              if (ld_agent(tmo) == call || __builtin_amdgcn_s_memrealtime() - t0 > DEADLINE) { st_agent(tmo, call); break; }
              __builtin_amdgcn_s_sleep(2);
            }
          }
          while ((int)(lds_peek(done + sl) - 3 * (seq / RING)) < 0) {
            if (__builtin_amdgcn_s_memrealtime() - t0 > 2 * DEADLINE) { st_agent(tmo, call); break; }
            __builtin_amdgcn_s_sleep(1);
          }
          bad = __builtin_amdgcn_readfirstlane(ld_agent(tmo)) == call;
          if (stop) { pend = sl; pinfo = 1u << 13; ++seq; PUBLISH(); break; }
        }
        const unsigned nexc = ((unsigned)fpre >> (4 * w)) & 15u;
        unsigned tk; u64 fn;
        ticket_issue(tk, tick);
        asm volatile("global_load_dwordx2 %0, %1, %2 sc0 sc1" : "=&v"(fn)
                     : "v"(0u), "s"(myws + OFF_FLAG + (size_t)(s * 32 + ((inext >> 3) & 31)) * 128) : "memory");
        const char* src = sws + OFF_O + ((size_t)((o0 + 7 * (tl >> 3)) * 8 + (tl & 7)) * 8 + w) * SLOT + lane * 16;
        LAS char* dstl = rbase + sl * LSLOT;
        const int nld = bad ? 0 : (nexc == 15u ? 16 : 13);
        if (nexc != 0 && nexc != 15u && !bad)
          __builtin_amdgcn_global_load_lds((const GAS void*)(src - lane * 12 + 16384), (LAS void*)(dstl + 16384), 4, 0, 17);
        for (int i = 0; i < nld; ++i) {
          __builtin_amdgcn_global_load_lds((const GAS void*)(src + i * 1024), (LAS void*)(dstl + i * 1024), 16, 0, 17);
          if (i <= PULL_WIN - 2) WAIT_VM(PULL_WIN + 1);
          else WAIT_VM(PULL_WIN - 1);
          if (i == PULL_WIN - 2) PUBLISH();
        }
        if (nld == 0) { WAIT_VM(0); PUBLISH(); }
        asm volatile("" : "+v"(tk), "+v"(fn) :: "memory");
        ++seq;
        pend = sl;
        pinfo = item | (nexc << 8) | ((unsigned)bad << 12);
        item = inext;
        inext = __builtin_amdgcn_readfirstlane(tk);
        fpre = uni64(fn);
      }
      STAMP2(3);
    } else {
      const int role = wave >= 6 ? 2 : (wave & 1);
      const unsigned vo = ((lane >> 4) * 4 * NFULL + (lane & 15) * 4) * 4;
      for (unsigned seq = 0; seq < 257u; ++seq) {
        const int sl = seq % RING;
        u64 t0 = __builtin_amdgcn_s_memrealtime();
        bool lost = false;
        while (lds_peek((LAS unsigned*)(full + sl)) != seq + 1) {
          if (__builtin_amdgcn_s_memrealtime() - t0 > 4 * DEADLINE) { lost = true; break; }
          __builtin_amdgcn_s_sleep(1);
        }
        if (lost) { st_agent(tmo, call); if (lane == 0) p.out[rank][0] = __builtin_nanf(""); break; }
        const unsigned inf = lds_peek((LAS unsigned*)(info + sl));
        if (inf >> 13) break;
        const int w = inf & 7, tl = (inf >> 3) & 31;
        const unsigned nexc = (inf >> 8) & 15u;
        const bool bad = (inf >> 12) & 1u;
        const float sc = bad ? __builtin_nanf("") : scale;
        float* tbase = p.out[rank] + (size_t)((tl >> 3) * 128 + (w >> 2) * 64) * NFULL + s * NSH + (tl & 7) * 256 + (w & 3) * 64;
        const LAS char* sb = rbase + sl * LSLOT + lane * 16;
        u32x4 cw[6];
#pragma unroll
        for (int i = 0; i < 4; ++i) cw[i] = *(const LAS u32x4*)(sb + (role * 4 + i) * 1024);
        cw[4] = *(const LAS u32x4*)(sb + 12 * 1024);
        cw[5] = *(const LAS u32x4*)(sb + (13 + role) * 1024);
        u64 ent = 0;
        if (nexc != 0 && nexc != 15u) ent = *(const LAS u64*)(rbase + sl * LSLOT + 16384 + (lane & 15) * 8);
        asm volatile("s_waitcnt lgkmcnt(0)" : "+v"(cw[0]), "+v"(cw[1]), "+v"(cw[2]), "+v"(cw[3]), "+v"(cw[4]), "+v"(cw[5]), "+v"(ent) :: "memory");
        if (lane == 0) __hip_atomic_fetch_add(done + sl, 1u, __ATOMIC_RELAXED, __HIP_MEMORY_SCOPE_WORKGROUP);
        if (nexc != 15u) {
#pragma unroll
          for (int t = 0; t < 4; ++t) {
            unsigned d = ((cw[4][t] >> (1 + role)) & 0x01000000u) | ((cw[4][t] << (3 - role)) & 0x80000000u);
#pragma unroll
            for (int i = 0; i < 4; ++i) d |= ((cw[i][t] >> 25) & 63u) << (6 * i);
            cw[5][t] = as_u(__builtin_amdgcn_ldexpf(as_f(d), 131));
#pragma unroll
            for (int i = 0; i < 5; ++i) cw[i][t] = as_u(__builtin_amdgcn_ldexpf(as_f(cw[i][t] & 0x81FFFFFFu), 131));
          }
        }
#pragma unroll
        for (int i = 0; i < 6; ++i) {
          if (i == 4 && role != 0) continue;
          f32x4 y;
#pragma unroll
          for (int t = 0; t < 4; ++t) y[t] = silu_dq(as_f(cw[i][t]), sc);
          const int row = i < 4 ? role * 16 + i : (i == 4 ? 48 : 49 + role);
          st16_out(tbase + (size_t)row * NFULL, vo, __builtin_bit_cast(u32x4, y));
        }
        if (nexc != 0 && nexc != 15u && !bad) {
          const unsigned pos = (unsigned)(ent >> 32) & 4095u;
          const unsigned qi = (pos >> 10) * 4 + ((pos >> 6) & 3);
          const unsigned owner = qi < 12 ? qi >> 2 : (qi == 12 ? 0 : qi - 13);
          WAIT_VM(0);
          if ((unsigned)lane < nexc && owner == (unsigned)role)
            st_agent(tbase + (size_t)(pos >> 6) * NFULL + (pos & 63u), as_u(silu_dq(as_f((unsigned)ent), scale)));
        }
      }
      STAMP(4);
    }
  }
}

extern "C" void kernel_launch(int rank, int nranks, ncclComm_t comm,
                              hipStream_t stream,
                              void* const* d_in, const int* in_sizes, int n_in,
                              void* d_out, int out_size,
                              const int* in_dtypes, int out_dtype,
                              void* const* d_out_all,
                              void* const* d_ws_all, size_t ws_bytes) {
  (void)comm; (void)in_dtypes; (void)out_dtype; (void)d_out;
  if (nranks != NR || n_in != 4 || ws_bytes < WS_NEED || (unsigned)rank >= (unsigned)NR ||
      in_sizes[0] != 4096 * KSH || in_sizes[1] != KSH * NFULL || out_size != 512 * NFULL) {
    fprintf(stderr, "kernel_launch: unexpected problem shape\n");
    return;
  }
  Params p;
  p.x = (const uint8_t*)d_in[0];
  p.w = (const uint8_t*)d_in[1];
  p.sx = (const float*)d_in[2];
  p.sw = (const float*)d_in[3];
  for (int i = 0; i < NR; ++i) {
    p.out[i] = (float*)d_out_all[i];
    p.ws[i] = (char*)d_ws_all[i];
  }
  p.rank = rank;
  p.pad = 0;
  hipLaunchKernelGGL(fused_gemm_rs, dim3(256), dim3(512), 0, stream, p);
  HIP_CHECK(hipGetLastError());
}
```

```cpp
#include <hip/hip_runtime.h>
#include <rccl/rccl.h>
#include <cstdio>
#include <cstdint>

#define NPUSH 4
#define PUSH_WIN 2
#define TAIL_ITEMS 16
#define PULL_WIN 4
#define PULL_WIN_IN 3
#define POLL_SLEEP 8
#define TIMING 0

#define HIP_CHECK(expr) do { hipError_t _e = (expr); if (_e != hipSuccess) { \
    fprintf(stderr, "HIP error %s at %s:%d\n", hipGetErrorString(_e), __FILE__, __LINE__); \
    return; } } while (0)

typedef unsigned u32x4 __attribute__((ext_vector_type(4)));
typedef int i32x4 __attribute__((ext_vector_type(4)));
typedef int i32x8 __attribute__((ext_vector_type(8)));
typedef float f32x4 __attribute__((ext_vector_type(4)));
typedef unsigned long long u64;

#define GAS __attribute__((address_space(1)))
#define LAS __attribute__((address_space(3)))

constexpr int NR = 8;
constexpr int NFULL = 16384;
constexpr int KSH = 512;
constexpr int NSH = 2048;
constexpr u64 DEADLINE = 100000000ull;

constexpr size_t OFF_CALL  = 0;
constexpr size_t OFF_PUSHF = 32768;
constexpr size_t OFF_QCNT  = OFF_PUSHF + 1024;
constexpr size_t OFF_TMO   = OFF_QCNT + 2048;
constexpr size_t OFF_READY = OFF_TMO + 128;
constexpr size_t OFF_RFLAG = OFF_READY + 5248;
constexpr size_t OFF_PAGG  = OFF_RFLAG + 5120;
constexpr size_t OFF_FLAG  = 65536;
constexpr size_t OFF_EAGG  = OFF_FLAG + 16 * 72 * 128;
constexpr size_t OFF_TIME  = OFF_EAGG + 96 * 256;
constexpr size_t OFF_A     = 1u << 20;
constexpr size_t OFF_B     = OFF_A + (32u << 20);
constexpr size_t OFF_S     = OFF_B + (8u << 20);
constexpr size_t PAIR      = 9216;
constexpr size_t OFF_O     = OFF_S + 2560 * PAIR;
constexpr size_t SLOT      = 16384;
constexpr int EXC_OFF = 13312;
constexpr size_t WS_NEED   = OFF_O + (size_t)28 * 64 * SLOT;
static_assert(OFF_PAGG + 4096 <= OFF_FLAG && OFF_TIME + 32768 <= OFF_A, "layout");
constexpr int EXC_CAP = 14;

constexpr int STAGE = 49152;
constexpr int LDS_BYTES = 3 * STAGE + 256;

struct Params {
  const uint8_t* x;
  const uint8_t* w;
  const float* sx;
  const float* sw;
  float* out[NR];
  char* ws[NR];
  int rank;
  int pad;
};

__device__ __forceinline__ float as_f(unsigned u) { return __builtin_bit_cast(float, u); }
__device__ __forceinline__ unsigned as_u(float f) { return __builtin_bit_cast(unsigned, f); }
__device__ __forceinline__ void st16_sys(const void* base, unsigned off, u32x4 v) {
  asm volatile("global_store_dwordx4 %0, %1, %2 sc0 sc1\n\ts_nop 1" :: "v"(off), "v"(v), "s"(base) : "memory");
}
#define OUT_ST "sc1"
__device__ __forceinline__ void st16_out(const void* base, unsigned off, u32x4 v) {
  asm volatile("global_store_dwordx4 %0, %1, %2 " OUT_ST "\n\ts_nop 1" :: "v"(off), "v"(v), "s"(base) : "memory");
}
__device__ __forceinline__ void ticket_issue(unsigned& t, const void* ctr) {
  u64 sv;
  asm volatile("s_mov_b64 %1, exec\n\ts_mov_b64 exec, 1\n\tglobal_atomic_add %0, %2, %3, %4 sc0\n\ts_mov_b64 exec, %1"
               : "=&v"(t), "=&s"(sv) : "v"(0u), "v"(1u), "s"(ctr) : "memory");
}
#define WAIT_VM(n) asm volatile("s_waitcnt vmcnt(%0)" :: "i"(n) : "memory")

__device__ __forceinline__ unsigned ld_sys(const void* p) {
  return __hip_atomic_load((const GAS unsigned*)p, __ATOMIC_RELAXED, __HIP_MEMORY_SCOPE_SYSTEM);
}
__device__ __forceinline__ u64 ld_sys64(const void* p) {
  return __hip_atomic_load((const GAS u64*)p, __ATOMIC_RELAXED, __HIP_MEMORY_SCOPE_SYSTEM);
}
__device__ __forceinline__ unsigned ld_agent(const void* p) {
  return __hip_atomic_load((const GAS unsigned*)p, __ATOMIC_RELAXED, __HIP_MEMORY_SCOPE_AGENT);
}
__device__ __forceinline__ void st_agent(void* p, unsigned v) {
  __hip_atomic_store((GAS unsigned*)p, v, __ATOMIC_RELAXED, __HIP_MEMORY_SCOPE_AGENT);
}
__device__ __forceinline__ void st_sys(void* p, unsigned v) {
  __hip_atomic_store((GAS unsigned*)p, v, __ATOMIC_RELAXED, __HIP_MEMORY_SCOPE_SYSTEM);
}

__device__ __forceinline__ void wait_ge(const void* p, unsigned target, void* tmo, unsigned call) {
  u64 t0 = __builtin_amdgcn_s_memrealtime();
  while ((int)(ld_sys(p) - target) < 0) {
    if (ld_agent(tmo) == call || __builtin_amdgcn_s_memrealtime() - t0 > DEADLINE) {
      st_agent(tmo, call);
      break;
    }
    __builtin_amdgcn_s_sleep(POLL_SLEEP);
  }
}

__device__ __forceinline__ float silu_dq(float acc, float s) {
  float y = acc * s;
  float c = fminf(fmaxf(y, -60.0f), 60.0f);
  return y * __builtin_amdgcn_rcpf(1.0f + __builtin_amdgcn_exp2f(c * -1.44269504088896f));
}

__device__ __forceinline__ unsigned pack_z(float a) {
  return __builtin_bit_cast(unsigned, __builtin_amdgcn_ldexpf(a, -131));
}
__device__ __forceinline__ unsigned pack_misfit(float a) {
  const float zf = __builtin_amdgcn_ldexpf(a, -131);
  const unsigned back = __builtin_bit_cast(unsigned, __builtin_amdgcn_ldexpf(zf, 131));
  return (__builtin_bit_cast(unsigned, zf) & 0x7E000000u) | (back ^ __builtin_bit_cast(unsigned, a));
}

__device__ __forceinline__ int blk_dest(int r, int o) { return o < 28 ? (r + 1 + o % 7) & 7 : r; }
__device__ __forceinline__ int blk_sub(int o) { return o < 28 ? o / 7 : o - 28; }

#define STAMP(i) do {} while (0)
#define STAMP2(i) do {} while (0)

__device__ __forceinline__ int blk_order(int r, int mb) {
  const int d = mb >> 2, sub = mb & 3;
  return d == r ? 28 + sub : ((d - r - 1) & 7) + 7 * sub;
}

__device__ __forceinline__ void x_load(const Params& p, int mb, int e, int lane, unsigned (&w)[16]) {
  const __amdgpu_buffer_rsrc_t rs = __builtin_amdgcn_make_buffer_rsrc((void*)p.x, 0, 4096 * KSH, 0x00020000);
  const unsigned vo = (lane & 15) * KSH + (lane >> 4) * 16;
  const unsigned so = (mb * 128 + (e >> 1) * 16) * KSH + (e & 1) * 256;
#pragma unroll
  for (int u = 0; u < 4; ++u) {
    u32x4 v = __builtin_amdgcn_raw_buffer_load_b128(rs, vo, so + u * 64, 0);
    w[4 * u] = v[0]; w[4 * u + 1] = v[1]; w[4 * u + 2] = v[2]; w[4 * u + 3] = v[3];
  }
}
__device__ __forceinline__ void w_load(const Params& p, int col0, int e, int lane, unsigned (&r)[16]) {
  const __amdgpu_buffer_rsrc_t rs = __builtin_amdgcn_make_buffer_rsrc((void*)p.w, 0, KSH * NFULL, 0x00020000);
  const unsigned vo = (lane >> 4) * 16 * NFULL + (lane & 15) * 4;
  const unsigned so = (e >> 2) * 64 * NFULL + col0 + (e & 3) * 64;
#pragma unroll
  for (int i = 0; i < 16; ++i) r[i] = __builtin_amdgcn_raw_buffer_load_b32(rs, vo, so + i * NFULL, 0);
}
__device__ __forceinline__ void w_transpose(unsigned (&r)[16]) {
  unsigned w[16];
#pragma unroll
  for (int u = 0; u < 4; ++u) {
    const unsigned a0 = __builtin_amdgcn_perm(r[4 * u + 1], r[4 * u], 0x05010400u), a1 = __builtin_amdgcn_perm(r[4 * u + 1], r[4 * u], 0x07030602u);
    const unsigned b0 = __builtin_amdgcn_perm(r[4 * u + 3], r[4 * u + 2], 0x05010400u), b1 = __builtin_amdgcn_perm(r[4 * u + 3], r[4 * u + 2], 0x07030602u);
    w[u] = __builtin_amdgcn_perm(b0, a0, 0x05040100u);
    w[4 + u] = __builtin_amdgcn_perm(b0, a0, 0x07060302u);
    w[8 + u] = __builtin_amdgcn_perm(b1, a1, 0x05040100u);
    w[12 + u] = __builtin_amdgcn_perm(b1, a1, 0x07060302u);
  }
#pragma unroll
  for (int i = 0; i < 16; ++i) r[i] = w[i];
}
__device__ __forceinline__ void chunk_place(bool isw, int blk, int e, int src, unsigned par, unsigned& dst, unsigned& s01, unsigned& s23) {
  if (isw) {
    dst = (unsigned)OFF_B + (blk * 32 + src * 4 + (e >> 3)) * 32768u + (e & 3) * 8192 + ((e >> 2) & 1) * 1024;
    s01 = 2048; s23 = 4096;
  } else {
    dst = (unsigned)OFF_A + ((par * 32 + blk) * 32 + src * 4 + (e & 1) * 2) * 16384u + (e >> 1) * 2048;
    s01 = 1024; s23 = 16384;
  }
}
__device__ __forceinline__ void chunk_store(const char* base, unsigned dst, unsigned s01, unsigned s23, int lane, const unsigned (&w)[16]) {
  st16_sys(base + dst, lane * 16, u32x4{w[0], w[1], w[2], w[3]});
  st16_sys(base + dst + s01, lane * 16, u32x4{w[4], w[5], w[6], w[7]});
  st16_sys(base + dst + s23, lane * 16, u32x4{w[8], w[9], w[10], w[11]});
  st16_sys(base + dst + s23 + s01, lane * 16, u32x4{w[12], w[13], w[14], w[15]});
}

__device__ __forceinline__ unsigned wave_scan_incl(unsigned v) {
  v += __builtin_amdgcn_update_dpp(0u, v, 0x111, 0xf, 0xf, true);
  v += __builtin_amdgcn_update_dpp(0u, v, 0x112, 0xf, 0xf, true);
  v += __builtin_amdgcn_update_dpp(0u, v, 0x114, 0xf, 0xf, true);
  v += __builtin_amdgcn_update_dpp(0u, v, 0x118, 0xf, 0xf, true);
  v += __builtin_amdgcn_update_dpp(0u, v, 0x142, 0xa, 0xf, false);
  v += __builtin_amdgcn_update_dpp(0u, v, 0x143, 0xc, 0xf, false);
  return v;
}

constexpr int ESC_CAP = 1120;
constexpr int SCR = 1344;
__device__ __forceinline__ bool chunk_pack(const unsigned (&w)[16], const char* pair, int h, LAS char* scr, int lane) {
  const char* out = pair + h * 3072;
  unsigned m[16], P[4] = {0, 0, 0, 0}, big = 0, cnt = 0;
#pragma unroll
  for (int i = 0; i < 16; ++i) {
    const unsigned e = (w[i] >> 3) & 0x0F0F0F0Fu;
    const unsigned in47 = (e >> 2) & ~(e >> 3) & 0x01010101u;
    const unsigned c = e & (in47 | (in47 << 1));
    m[i] = ~(c | (c >> 1)) & 0x01010101u;
    P[i >> 2] |= c << (2 * (i & 3));
    big |= e + 0x05050505u;
    cnt += __builtin_popcount(m[i]);
  }
  const unsigned incl = wave_scan_incl(cnt);
  const unsigned total = __builtin_amdgcn_readlane(incl, 63);
  if (total > (unsigned)ESC_CAP || __builtin_amdgcn_ballot_w64((big & 0x10101010u) != 0) != 0) {
#pragma unroll
    for (int b = 0; b < 4; ++b)
      st16_sys(b < 3 ? out + b * 1024 : pair + 7168 + h * 1024, lane * 16, u32x4{w[4 * b], w[4 * b + 1], w[4 * b + 2], w[4 * b + 3]});
    return true;
  }
  unsigned pos = incl - cnt;
  const unsigned dummy = 1152 + lane;
#pragma unroll
  for (int i = 0; i < 16; ++i) {
#pragma unroll
    for (int k = 0; k < 4; ++k) {
      const unsigned mk = (m[i] >> (8 * k)) & 1u;
      scr[mk ? pos : dummy] = (char)(w[i] >> (8 * k + 3));
      pos += mk;
    }
  }
  u32x4 n0, n1;
#pragma unroll
  for (int i = 0; i < 4; ++i) {
    n0[i] = (w[2 * i] & 0x87878787u) | ((w[2 * i + 1] << 3) & 0x38383838u) | ((w[2 * i + 1] >> 1) & 0x40404040u);
    n1[i] = (w[8 + 2 * i] & 0x87878787u) | ((w[9 + 2 * i] << 3) & 0x38383838u) | ((w[9 + 2 * i] >> 1) & 0x40404040u);
  }
  st16_sys(out, lane * 16, n0);
  st16_sys(out + 1024, lane * 16, n1);
  st16_sys(out + 2048, lane * 16, u32x4{P[0], P[1], P[2], P[3]});
  asm volatile("s_waitcnt lgkmcnt(0)" ::: "memory");
  if (lane < 56) {
    unsigned b[5];
#pragma unroll
    for (int i = 0; i < 5; ++i) {
      const unsigned e = *(const LAS unsigned*)(scr + lane * 20 + i * 4) & 0x0F0F0F0Fu;
      b[i] = (e - ((e >> 3) & 0x01010101u) * 3u) & 0x07070707u;
    }
    const u64 o = (u64)(b[0] | (b[1] << 3) | ((b[4] & 0x03030303u) << 6)) |
                  ((u64)(b[2] | (b[3] << 3) | ((b[4] & 0x04040404u) << 4)) << 32);
    __hip_atomic_store((GAS u64*)(pair + 6144 + h * 448 + lane * 8), o, __ATOMIC_RELAXED, __HIP_MEMORY_SCOPE_SYSTEM);
  }
  asm volatile("s_waitcnt lgkmcnt(0)" ::: "memory");
  return false;
}

template <class Mid>
__device__ __forceinline__ void chunk_unpack(const LAS char* pair, int h, bool raw, LAS char* scr, int lane, unsigned (&w)[16], Mid&& mid) {
  const LAS char* in = pair + h * 3072;
  const u32x4 P = *(const LAS u32x4*)(in + 2048 + lane * 16);
  const u64 o = *(const LAS u64*)(pair + 6144 + h * 448 + lane * 8);
  const u32x4 n0 = *(const LAS u32x4*)(in + lane * 16), n1 = *(const LAS u32x4*)(in + 1024 + lane * 16);
  if (raw) {
    const u32x4 r3 = *(const LAS u32x4*)(pair + 7168 + h * 1024 + lane * 16);
#pragma unroll
    for (int i = 0; i < 4; ++i) { w[i] = n0[i]; w[4 + i] = n1[i]; w[8 + i] = P[i]; w[12 + i] = r3[i]; }
    mid();
    return;
  }
  {
    const unsigned o0 = (unsigned)o, o1 = (unsigned)(o >> 32);
    unsigned b[5] = {o0 & 0x07070707u, (o0 >> 3) & 0x07070707u, o1 & 0x07070707u, (o1 >> 3) & 0x07070707u,
                     ((o0 >> 6) & 0x03030303u) | ((o1 >> 4) & 0x04040404u)};
#pragma unroll
    for (int i = 0; i < 5; ++i) {
      const unsigned hi = ((b[i] + 0x03030303u) >> 3) & 0x01010101u;
      *(LAS unsigned*)(scr + lane * 20 + i * 4) = ((b[i] + hi + (hi << 1)) << 3) ^ 0x20202020u;
    }
  }
  unsigned M[4], cnt = 0;
#pragma unroll
  for (int k = 0; k < 4; ++k) { M[k] = ~(P[k] | (P[k] >> 1)) & 0x55555555u; cnt += __builtin_popcount(M[k]); }
  unsigned pos = ((wave_scan_incl(cnt) - cnt) & 0x7FFu) + (unsigned)(size_t)scr;
  asm volatile("s_waitcnt lgkmcnt(0)" ::: "memory");
  unsigned lo[16], hi[16], ps[16];
#pragma unroll
  for (int i = 0; i < 16; ++i) {
    const LAS unsigned* q = (const LAS unsigned*)(size_t)(pos & ~3u);
    lo[i] = q[0]; hi[i] = q[1]; ps[i] = pos;
    pos += __builtin_popcount(M[i >> 2] & (0x01010101u << (2 * (i & 3))));
  }
  __builtin_amdgcn_sched_barrier(0);
  mid();
  __builtin_amdgcn_sched_barrier(0);
#pragma unroll
  for (int i = 0; i < 16; ++i) {
    const int sh = 2 * (i & 3);
    const unsigned m = (M[i >> 2] >> sh) & 0x01010101u;
    unsigned t;
    asm("v_mul_u32_u24 %0, %1, %2" : "=v"(t) : "s"(0x010101u), "v"(m));
    const unsigned sel = (t << 8) | ((m << 2) ^ 0x04040404u);
    const unsigned esc = __builtin_amdgcn_perm(0u, __builtin_amdgcn_alignbyte(hi[i], lo[i], ps[i]), sel);
    const unsigned ex = ((sh <= 3 ? P[i >> 2] << (3 - sh) : P[i >> 2] >> (sh - 3)) & 0x18181818u) | 0x20202020u;
    const unsigned n = i < 8 ? n0[(i >> 1) & 3] : n1[(i >> 1) & 3];
    const unsigned sm = (i & 1) ? (((n >> 3) & 0x07070707u) | ((n << 1) & 0x80808080u)) : (n & 0x87878787u);
    w[i] = sm | (ex ^ esc);
  }
}

__device__ __forceinline__ void piece_flag(char* myws, int piece, unsigned t, unsigned call, int lane) {
  const unsigned need = piece >= 32 ? 256u : (piece < NPUSH ? 16u : 128u);
  if (t + 1 == call * need && lane == 0) st_agent(myws + OFF_RFLAG + (size_t)piece * 128, call);
}
__device__ __forceinline__ void piece_count(char* myws, int piece, unsigned call, int lane) {
  unsigned t;
  ticket_issue(t, myws + OFF_READY + (size_t)piece * 128);
  asm volatile("s_waitcnt vmcnt(0)" : "+v"(t) :: "memory");
  piece_flag(myws, piece, __builtin_amdgcn_readfirstlane(t), call, lane);
}

constexpr int NIN = (32 - NPUSH) * 8 + 128;
constexpr int NITEM = NIN + 256;

constexpr int RING = 4;
constexpr int LSLOT = 16384;

__device__ __forceinline__ unsigned lds_peek(LAS unsigned* p) {
  unsigned v;
  asm volatile("ds_read_b32 %0, %1\n\ts_waitcnt lgkmcnt(0)" : "=&v"(v) : "v"(p) : "memory");
  return __builtin_amdgcn_readfirstlane(v);
}
__device__ __forceinline__ u64 uni64(u64 v) {
  return ((u64)(unsigned)__builtin_amdgcn_readfirstlane((unsigned)(v >> 32)) << 32) | (u64)(unsigned)__builtin_amdgcn_readfirstlane((unsigned)v);
}

__global__ void __launch_bounds__(512, 2) fused_gemm_rs(Params p) {
  __shared__ __attribute__((aligned(16))) char smem[LDS_BYTES];
  const int tid = threadIdx.x;
  const int lane = tid & 63;
  const int wave = __builtin_amdgcn_readfirstlane(tid >> 6);
  const int rank = p.rank;
  const int j = blockIdx.x & 7;
  const int grp = blockIdx.x >> 3;
  char* const myws = p.ws[rank];
  volatile LAS unsigned* sctl = (volatile LAS unsigned*)(LAS char*)(smem + 3 * STAGE);

  const int peer = (rank + 1 + grp) & 7;
  unsigned w0[16], wp[NPUSH / 2][16];
  const int pw = j * 4 + wave;
  const int ew = (wave >> 1) * 400 + (blockIdx.x - 56) * 2 + (wave & 1);
  auto enc_id = [&](int n, bool& isw, int& a, int& b, int& e) {
    if (n < 128) { isw = false; a = (n >> 4) * 4; b = 0; e = n & 15; }
    else if (n < 2176) { isw = true; b = (n - 128) >> 8; a = ((n - 128) >> 5) & 7; e = n & 31; }
    else { isw = false; a = (((n - 2176) >> 4) & 7) * 4 + 1 + ((n - 2176) >> 7); b = 0; e = n & 15; }
  };
  auto enc_load = [&](int n, unsigned (&w)[16]) {
    bool isw; int a, b, e;
    enc_id(n, isw, a, b, e);
    if (isw) w_load(p, a * NSH + b * 256, e, lane, w); else x_load(p, a, e, lane, w);
  };
  unsigned cticket;
  ticket_issue(cticket, myws + OFF_CALL + (size_t)blockIdx.x * 128 + wave * 16);
  STAMP(0);
  if (grp >= 7) enc_load(ew, w0);
  else if (wave < 4) {
#pragma unroll
    for (int it = 0; it < NPUSH / 2; ++it) x_load(p, blk_dest(peer, (pw >> 4) + 2 * it) * 4, pw & 15, lane, wp[it]);
  }
  if (grp < 7) {
    if (tid >= 16 && tid < 48) sctl[tid] = 0;
    asm volatile("s_waitcnt lgkmcnt(0)" ::: "memory");
    __builtin_amdgcn_s_barrier();
  }
  asm volatile("s_waitcnt vmcnt(0)" : "+v"(cticket) :: "memory");
  const unsigned call = (unsigned)__builtin_amdgcn_readfirstlane(cticket) + 1;
  const unsigned par = call & 1;
  void* const tmo = myws + OFF_TMO;
  const float scale = p.sx[0] * p.sw[0];

  const bool dbg_skip = false;
  if (dbg_skip) {
  } else if (grp >= 7) {
    LAS char* const scr = (LAS char*)smem + wave * SCR;
    if (wave < 2) __builtin_amdgcn_s_setprio(3); else if (wave < 4) __builtin_amdgcn_s_setprio(2); else if (wave < 6) __builtin_amdgcn_s_setprio(1);
    for (int n = ew; n < 2560; n += 1600) {
      bool isw; int a, b, e;
      enc_id(n, isw, a, b, e);
      if (n != ew) enc_load(n, w0);
      if (isw) w_transpose(w0);
      const bool mine = !isw || a == rank;
      const bool theirs = !isw || a != rank;
      const int blk = isw ? b : blk_order(rank, a);
      if (theirs) {
        const int slot = isw ? 512 + (a * 8 + b) * 32 + e : a * 16 + e;
        const bool raw = chunk_pack(w0, myws + OFF_S + ((size_t)par * 1280 + (slot >> 1)) * PAIR, slot & 1, scr, lane);
        WAIT_VM(0);
        char* const agg = myws + OFF_EAGG + (size_t)(isw ? 32 + a * 8 + b : a) * 256;
        const u64 mine64 = (1ull << 32) | (raw ? 1u << e : 0u);
        u64 tot = 0;
        if (lane == 0) tot = __hip_atomic_fetch_add((GAS u64*)(agg + par * 128), mine64, __ATOMIC_RELAXED, __HIP_MEMORY_SCOPE_AGENT);
        tot = uni64(tot) + mine64;
        if ((unsigned)(tot >> 32) == (isw ? 32u : 16u)) {
          if (lane < NR && lane != rank && (!isw || lane == a))
            __hip_atomic_store((GAS u64*)(p.ws[lane] + OFF_FLAG + (size_t)((par * 8 + rank) * 72 + (isw ? 32 + b : a)) * 128),
                               ((u64)call << 32) | (unsigned)tot, __ATOMIC_RELAXED, __HIP_MEMORY_SCOPE_SYSTEM);
          if (lane == 0) __hip_atomic_store((GAS u64*)(agg + (par ^ 1) * 128), 0ull, __ATOMIC_RELAXED, __HIP_MEMORY_SCOPE_AGENT);
        }
      }
      if (mine) {
        unsigned dst, s01, s23;
        chunk_place(isw, blk, e, rank, par, dst, s01, s23);
        chunk_store(myws, dst, s01, s23, lane, w0);
        WAIT_VM(0);
        piece_count(myws, isw ? 32 + blk : blk, call, lane);
      }
    }
    __builtin_amdgcn_s_setprio(0);
  } else if (wave < 4) {
    char* const dws = p.ws[peer];
#pragma unroll
    for (int it = 0; it < NPUSH / 2; ++it) {
      unsigned dst, s01, s23;
      chunk_place(false, (pw >> 4) + 2 * it, pw & 15, rank, par, dst, s01, s23);
      const unsigned (&w)[16] = wp[it];
#pragma unroll
      for (int b = 0; b < 4; ++b) {
        st16_sys(dws + dst + (b & 1) * s01 + (b >> 1) * s23, lane * 16, u32x4{w[4 * b], w[4 * b + 1], w[4 * b + 2], w[4 * b + 3]});
        WAIT_VM(PUSH_WIN - 1);
      }
    }
    WAIT_VM(0);
    unsigned ag[NPUSH / 2];
#pragma unroll
    for (int it = 0; it < NPUSH / 2; ++it) ticket_issue(ag[it], myws + OFF_PAGG + (size_t)(peer * 8 + (pw >> 4) + 2 * it) * 64);
#pragma unroll
    for (int it = 0; it < NPUSH / 2; ++it) {
      asm volatile("s_waitcnt vmcnt(0)" : "+v"(ag[it]) :: "memory");
      if ((unsigned)__builtin_amdgcn_readfirstlane(ag[it]) + 1 == call * 16u && lane == 0)
        st_sys(dws + OFF_PUSHF + (size_t)((pw >> 4) + 2 * it) * 128 + rank * 4, call);
    }
  }
  if (blockIdx.x == 64 && tid < NR)
    st_agent(myws + OFF_QCNT + (size_t)((par ^ 1) * 8 + tid) * 128, 0u);
  STAMP(1);

  if (grp >= 7) {
    const int wm = wave >> 2, G = wave & 3;
    const int npass = (grp >= 8 && grp < 15) ? 2 : 1;
    for (int pass = 0; pass < npass; ++pass) {
      const int o = grp == 7 ? 24 : (pass == 0 ? grp - 8 : grp - 8 + 25);
      if (dbg_skip) {}
      else if (tid == 0) wait_ge(myws + OFF_RFLAG + (size_t)(32 + j) * 128, call, tmo, call);
      else if (tid == 1) wait_ge(myws + OFF_RFLAG + (size_t)o * 128, call, tmo, call);
      else if (tid < 10 && o < NPUSH && tid - 2 != rank) wait_ge(myws + OFF_PUSHF + (size_t)o * 128 + (tid - 2) * 4, call, tmo, call);
      if (wave == 0) {
        const unsigned t = ld_agent(tmo);
        if (lane == 0) sctl[1] = t == call ? 0x7FC00000u : 0u;
      }
      __syncthreads();
      STAMP(2 + 3 * pass);
      const float acc0 = as_f(sctl[1]);
      f32x4 acc[4][4];
#pragma unroll
      for (int a = 0; a < 4; ++a)
#pragma unroll
        for (int b = 0; b < 4; ++b) acc[a][b] = f32x4{acc0, acc0, acc0, acc0};
      {
        const __amdgpu_buffer_rsrc_t rs = __builtin_amdgcn_make_buffer_rsrc(myws, 0, (int)WS_NEED, 0x00020000);
        const unsigned ga = (unsigned)OFF_A + (par * 32 + o) * (32u * 16384u) + wave * 2048;
        const unsigned gb = (unsigned)OFF_B + j * (32u * 32768u) + wave * 4096;
        const unsigned l16 = lane * 16;
        auto dma = [&](int ks, int buf, int i) {
          if (i < 2)
            __builtin_amdgcn_raw_ptr_buffer_load_lds(rs, (LAS void*)((LAS char*)smem + buf * STAGE + wave * 2048 + i * 1024), 16,
                                                     l16, ga + ks * 16384 + i * 1024, 0, 17);
          else
            __builtin_amdgcn_raw_ptr_buffer_load_lds(rs, (LAS void*)((LAS char*)smem + buf * STAGE + 16384 + wave * 4096 + (i - 2) * 1024), 16,
                                                     l16, gb + ks * 32768 + (i - 2) * 1024, 0, 17);
        };
        const LAS char* const la = (const LAS char*)smem + wm * 8192 + lane * 16;
        const LAS char* const lb = (const LAS char*)smem + 16384 + G * 8192 + lane * 16;
        auto frag = [&](const LAS char* q) -> i32x8 {
          const i32x4 lo = *(const LAS i32x4*)q, hi = *(const LAS i32x4*)(q + 1024);
          return __builtin_shufflevector(lo, hi, 0, 1, 2, 3, 4, 5, 6, 7);
        };
#define MMA(m, t, CA, CB) acc[m][t] = __builtin_amdgcn_mfma_scale_f32_16x16x128_f8f6f4(CA[m], CB[t], acc[m][t], 0, 0, 0, 127, 0, 127)
#define GEMM_STEP(KS, VM, CA, CB, NA, NB, DO_STAGE) do { \
          WAIT_VM(VM); \
          asm volatile("s_waitcnt lgkmcnt(0)" ::: "memory"); \
          __builtin_amdgcn_s_barrier(); \
          __builtin_amdgcn_sched_barrier(0); \
          const int nbuf = buf == 2 ? 0 : buf + 1; \
          _Pragma("unroll") for (int g = 0; g < 16; ++g) { \
            MMA(g >> 2, g & 3, CA, CB); \
            __builtin_amdgcn_sched_barrier(0); \
            if (g < 8) { \
              if (g & 1) NB[g >> 1] = frag(lb + nbuf * STAGE + (g >> 1) * 2048); \
              else NA[g >> 1] = frag(la + nbuf * STAGE + (g >> 1) * 2048); \
            } else if (DO_STAGE && g < 14) dma((KS) + 3, buf, g - 8); \
            __builtin_amdgcn_sched_barrier(0); \
          } \
          buf = nbuf; } while (0)
        i32x8 a0[4], b0[4], a1[4], b1[4];
#pragma unroll
        for (int i = 0; i < 18; ++i) dma(i / 6, i / 6, i % 6);
        WAIT_VM(12);
        __builtin_amdgcn_s_barrier();
#pragma unroll
        for (int t = 0; t < 4; ++t) { a0[t] = frag(la + t * 2048); b0[t] = frag(lb + t * 2048); }
        int buf = 0;
        for (int ks = 0; ks < 28; ks += 2) {
          GEMM_STEP(ks, 6, a0, b0, a1, b1, true);
          GEMM_STEP(ks + 1, 6, a1, b1, a0, b0, true);
        }
#define PIN_ACC() _Pragma("unroll") for (int g = 0; g < 16; ++g) asm volatile("" : "+v"(acc[g >> 2][g & 3]))
        GEMM_STEP(28, 6, a0, b0, a1, b1, true);
        PIN_ACC();
        GEMM_STEP(29, 6, a1, b1, a0, b0, false);
        PIN_ACC();
        GEMM_STEP(30, 0, a0, b0, a1, b1, false);
        PIN_ACC();
#pragma unroll
        for (int g = 0; g < 16; ++g) MMA(g >> 2, g & 3, a1, b1);
        PIN_ACC();
      }
      STAMP(3 + 3 * pass);

      const int dest = blk_dest(rank, o), sub = blk_sub(o);
      int ln = lane;
      asm volatile("" : "+v"(ln));
      const int q = ln >> 4, c = ln & 15;
      if (dest == rank) {
        const float* dst = p.out[rank] + (size_t)(sub * 128 + wm * 64) * NFULL + rank * NSH + j * 256 + G * 64;
        const unsigned vo = (4 * q * NFULL + 4 * c) * 4;
#pragma unroll
        for (int m = 0; m < 4; ++m)
#pragma unroll
          for (int r = 0; r < 4; ++r) {
            f32x4 v;
#pragma unroll
            for (int t = 0; t < 4; ++t) v[t] = silu_dq(acc[m][t][r], scale);
            st16_out(dst + (size_t)(m * 16 + r) * NFULL, vo, __builtin_bit_cast(u32x4, v));
          }
      } else {
        char* slot = myws + OFF_O + ((size_t)(o * 8 + j) * 8 + wave) * SLOT;
        unsigned gmask = 0;
#pragma unroll
        for (int m = 0; m < 4; ++m)
#pragma unroll
          for (int t = 0; t < 4; ++t) {
            unsigned misfit = 0;
#pragma unroll
            for (int r = 0; r < 4; ++r) misfit |= pack_misfit(acc[m][t][r]);
            if (__builtin_amdgcn_ballot_w64(misfit != 0) != 0) gmask |= 1u << (m * 4 + t);
            asm volatile("" : "+v"(acc[m][t]));
          }
        unsigned nexc = 0;
        if (gmask != 0) {
#pragma unroll
          for (int m = 0; m < 4; ++m)
#pragma unroll
            for (int t = 0; t < 4; ++t) {
              if ((gmask >> (m * 4 + t)) & 1u) {
#pragma unroll
                for (int r = 0; r < 4; ++r) {
                  const bool mis = pack_misfit(acc[m][t][r]) != 0;
                  const u64 mask = __builtin_amdgcn_ballot_w64(mis);
                  const unsigned idx = nexc + __builtin_amdgcn_mbcnt_hi((unsigned)(mask >> 32), __builtin_amdgcn_mbcnt_lo((unsigned)mask, 0u));
                  if (mis && idx < (unsigned)EXC_CAP) {
                    const u64 ent = ((u64)((m * 16 + 4 * q + r) * 64 + 4 * c + t) << 32) | as_u(acc[m][t][r]);
                    __hip_atomic_store((GAS u64*)(slot + EXC_OFF + idx * 8), ent, __ATOMIC_RELAXED, __HIP_MEMORY_SCOPE_SYSTEM);
                  }
                  nexc += __builtin_popcountll(mask);
                }
              }
              asm volatile("" : "+v"(acc[m][t]));
            }
        }
        if (nexc > (unsigned)EXC_CAP) {
          nexc = 15;
          WAIT_VM(0);
#pragma unroll
          for (int m = 0; m < 4; ++m)
#pragma unroll
            for (int r = 0; r < 4; ++r) {
              f32x4 v = {acc[m][0][r], acc[m][1][r], acc[m][2][r], acc[m][3][r]};
              st16_sys(slot + (m * 4 + r) * 1024, ln * 16, __builtin_bit_cast(u32x4, v));
            }
        } else {
          unsigned zd[3][4];
#pragma unroll
          for (int a = 0; a < 3; ++a)
#pragma unroll
            for (int t = 0; t < 4; ++t) zd[a][t] = pack_z(acc[3][t][1 + a]);
#pragma unroll
          for (int i = 0; i < 13; ++i) {
            u32x4 v;
#pragma unroll
            for (int t = 0; t < 4; ++t) {
              unsigned x;
              if (i < 12) x = zd[i >> 2][t] << (25 - 6 * (i & 3));
              else x = ((zd[0][t] << 1) & 0x02000000u) | ((zd[1][t] << 2) & 0x04000000u) | ((zd[2][t] << 3) & 0x08000000u) |
                       ((zd[0][t] >> 3) & 0x10000000u) | ((zd[1][t] >> 2) & 0x20000000u) | ((zd[2][t] >> 1) & 0x40000000u);
              v[t] = (pack_z(acc[i >> 2][t][i & 3]) & 0x81FFFFFFu) | (x & 0x7E000000u);
            }
            st16_sys(slot + i * 1024, ln * 16, v);
          }
        }
        if (lane == 0) sctl[8 + wave] = nexc;
        WAIT_VM(0);
        __syncthreads();
        if (tid == 0) {
          unsigned cnts = 0;
#pragma unroll
          for (int w = 0; w < 8; ++w) cnts |= sctl[8 + w] << (4 * w);
          __hip_atomic_store((GAS u64*)(p.ws[dest & 7] + OFF_FLAG + (size_t)((par * 8 + rank) * 72 + 40 + sub * 8 + j) * 128),
                             ((u64)call << 32) | cnts, __ATOMIC_RELAXED, __HIP_MEMORY_SCOPE_SYSTEM);
        }
      }
      STAMP(4 + 3 * pass);
    }
  } else {
    const int s = peer;
    const int ring = wave >= 4 ? (wave & 1) : (wave >> 1);
    LAS char* const rbase = (LAS char*)smem + ring * (RING * LSLOT);
    volatile LAS unsigned* const full = sctl + 16 + ring * 16;
    volatile LAS unsigned* const info = full + 4;
    LAS unsigned* const done = (LAS unsigned*)(full + 8);
    auto in_id = [&](int t, bool& isw, int& blk, int& e0) {
      constexpr int N1 = (7 - NPUSH) * 8;
      if (t < N1) { isw = false; blk = NPUSH + (t >> 3); e0 = (t & 7) * 2; }
      else if (t < N1 + 128) { isw = true; blk = (t - N1) >> 4; e0 = ((t - N1) & 15) * 2; }
      else { isw = false; blk = 7 + ((t - N1 - 128) >> 3); e0 = ((t - N1) & 7) * 2; }
    };
    if (wave == 4 || wave == 5) {
      const int o0 = (rank - s - 1) & 7;
      const char* sws = p.ws[s];
      const void* tick = myws + OFF_QCNT + (size_t)(par * 8 + s) * 128;
      auto describe = [&](unsigned t, unsigned& off, unsigned& flag, unsigned& shift) {
        int f;
        if (t < (unsigned)NIN) {
          bool isw; int blk, e0;
          in_id((int)t, isw, blk, e0);
          f = isw ? 32 + blk : blk_dest(rank, blk) * 4 + blk_sub(blk);
          const int slot = isw ? 512 + (rank * 8 + blk) * 32 + e0 : f * 16 + e0;
          off = (unsigned)OFF_S + (par * 1280 + (slot >> 1)) * (unsigned)PAIR;
          shift = e0;
        } else {
          const int tl = ((t - NIN) >> 3) & 31, w = (t - NIN) & 7;
          f = 40 + tl;
          off = (unsigned)OFF_O + (((o0 + 7 * (tl >> 3)) * 8 + (tl & 7)) * 8 + w) * (unsigned)SLOT;
          shift = 4 * w;
        }
        flag = (unsigned)OFF_FLAG + ((par * 8 + s) * 72 + f) * 128;
      };
      unsigned item = j * 2 + ring, inext = item + 16;
      unsigned off, flag, shift;
      describe(item, off, flag, shift);
      u64 fpre = 0;
      bool sfree = true;
      int pend = -1;
      unsigned pinfo = 0;
      unsigned seq = 0;
      const unsigned l16 = lane * 16;
#define PUBLISH() do { if (pend >= 0) { if (lane == 0) { info[pend] = pinfo; asm volatile("s_waitcnt lgkmcnt(0)" ::: "memory"); full[pend] = seq; } pend = -1; } } while (0)
      for (; seq <= (unsigned)NITEM; ) {
        const int sl = seq % RING;
        const bool stop = item >= (unsigned)NITEM;
        bool bad = false;
        if (!sfree) sfree = (int)(lds_peek(done + sl) - 3 * (seq / RING)) >= 0;
        if (stop || (int)((unsigned)(fpre >> 32) - call) < 0 || !sfree) {
          WAIT_VM(0);
          PUBLISH();
          u64 t0 = __builtin_amdgcn_s_memrealtime();
          if (!stop) {
            for (;;) {
              fpre = uni64(ld_sys64(myws + flag));
              if ((int)((unsigned)(fpre >> 32) - call) >= 0) break;
              if (ld_agent(tmo) == call || __builtin_amdgcn_s_memrealtime() - t0 > DEADLINE) { st_agent(tmo, call); bad = true; break; }
              __builtin_amdgcn_s_sleep(2);
            }
          }
          while ((int)(lds_peek(done + sl) - 3 * (seq / RING)) < 0) {
            if (__builtin_amdgcn_s_memrealtime() - t0 > 2 * DEADLINE) { st_agent(tmo, call); bad = true; break; }
            __builtin_amdgcn_s_sleep(1);
          }
          bad = __builtin_amdgcn_readfirstlane((int)bad) != 0;
          if (stop) { pend = sl; pinfo = 1u << 15; ++seq; PUBLISH(); break; }
          if (seq == 0) STAMP2(2);
        }
        if (item < (unsigned)NIN && inext >= (unsigned)NIN) STAMP2(5);
        const bool isin = item < (unsigned)NIN;
        const unsigned bits = ((unsigned)fpre >> shift) & (isin ? 3u : 15u);
        int len = isin ? (bits == 0 ? 7040 : 9216) : (bits == 0 ? 13312 : (bits == 15u ? 16384 : 13440));
        const int nld = (len + 1023) >> 10;
        if (bad) len = 0;
        const __amdgpu_buffer_rsrc_t rs = __builtin_amdgcn_make_buffer_rsrc((void*)(sws + off), 0, len, 0x00020000);
        LAS char* dstl = rbase + sl * LSLOT;
        unsigned tk; u64 fn;
#define PULL_ITEM(W) do { \
          __builtin_amdgcn_raw_ptr_buffer_load_lds(rs, (LAS void*)dstl, 16, l16, 0, 0, 17); \
          describe(inext, off, flag, shift); \
          ticket_issue(tk, tick); \
          asm volatile("global_load_dwordx2 %0, %1, %2 sc0 sc1" : "=&v"(fn) : "v"(0u), "s"(myws + flag) : "memory"); \
          WAIT_VM((W) + 1); \
          _Pragma("unroll") for (int i = 1; i < (W); ++i) { \
            __builtin_amdgcn_raw_ptr_buffer_load_lds(rs, (LAS void*)(dstl + i * 1024), 16, l16 + i * 1024, 0, 0, 17); \
            WAIT_VM((W) + 1); \
            if (i == (W) - 2) {                              \
              PUBLISH(); \
              sfree = (int)(lds_peek(done + (seq + 1) % RING) - 3 * ((seq + 1) / RING)) >= 0; \
            } \
          } \
          for (int i = (W); i < nld; ++i) { \
            __builtin_amdgcn_raw_ptr_buffer_load_lds(rs, (LAS void*)(dstl + i * 1024), 16, l16 + i * 1024, 0, 0, 17); \
            if (item < (unsigned)(NITEM - TAIL_ITEMS) || i == nld - 1) WAIT_VM((W) - 1); \
          } \
          asm volatile("" : "+v"(tk), "+v"(fn) :: "memory");           \
          inext2 = 32 + (unsigned)__builtin_amdgcn_readfirstlane(tk); \
          fpre = uni64(fn); } while (0)
        unsigned inext2;
        if (isin) PULL_ITEM(PULL_WIN_IN); else PULL_ITEM(PULL_WIN);
        ++seq;
        pend = sl;
        pinfo = item | (bits << 10) | ((unsigned)bad << 14);
        item = inext;
        inext = inext2;
      }
      STAMP2(3);
    } else {
      const int role = wave >= 6 ? 2 : (wave & 1);
      LAS char* const scr = (LAS char*)smem + 2 * RING * LSLOT + (ring * 3 + role) * SCR;
      const unsigned vo = ((lane >> 4) * 4 * NFULL + (lane & 15) * 4) * 4;
      int pp = 40;
      for (unsigned seq = 0; seq <= (unsigned)NITEM; ++seq) {
        const int sl = seq % RING;
        u64 t0 = __builtin_amdgcn_s_memrealtime();
        bool lost = false;
        while (lds_peek((LAS unsigned*)(full + sl)) != seq + 1) {
          if (pp != 40) { WAIT_VM(0); piece_count(myws, pp, call, lane); pp = 40; continue; }
          if (__builtin_amdgcn_s_memrealtime() - t0 > 4 * DEADLINE) { lost = true; break; }
          __builtin_amdgcn_s_sleep(1);
        }
        if (lost) { st_agent(tmo, call); if (lane == 0) p.out[rank][0] = __builtin_nanf(""); break; }
        const unsigned inf = lds_peek((LAS unsigned*)(info + sl));
        if (inf >> 15) break;
        const unsigned item = inf & 1023u, bits = (inf >> 10) & 15u;
        const bool bad = (inf >> 14) & 1u;
        if (item < (unsigned)NIN) {
          bool isw; int blk, e0;
          in_id((int)item, isw, blk, e0);
          const int c = (role + (int)(seq % 3)) % 3;
          if (c < 2) {
            unsigned w[16], t;
            chunk_unpack(rbase + sl * LSLOT, c, ((bits >> c) & 1u) || bad, scr, lane, w, [&]() {
              WAIT_VM(0);
              ticket_issue(t, myws + (pp != 40 ? OFF_READY + (size_t)pp * 128 : OFF_CALL + (size_t)blockIdx.x * 128 + wave * 16 + 8));
            });
            asm volatile("s_waitcnt lgkmcnt(0)" ::: "memory");
            if (lane == 0) __hip_atomic_fetch_add(done + sl, 1u, __ATOMIC_RELAXED, __HIP_MEMORY_SCOPE_WORKGROUP);
            unsigned dst, s01, s23;
            chunk_place(isw, blk, e0 + c, s, par, dst, s01, s23);
            chunk_store(myws, dst, s01, s23, lane, w);
            asm volatile("s_waitcnt vmcnt(4)" : "+v"(t) :: "memory");
            if (pp != 40) piece_flag(myws, pp, __builtin_amdgcn_readfirstlane(t), call, lane);
            pp = isw ? 32 + blk : blk;
          } else if (lane == 0) __hip_atomic_fetch_add(done + sl, 1u, __ATOMIC_RELAXED, __HIP_MEMORY_SCOPE_WORKGROUP);
          continue;
        }
        if (pp != 40) { WAIT_VM(0); piece_count(myws, pp, call, lane); pp = 40; }
        const int w = (item - NIN) & 7, tl = ((item - NIN) >> 3) & 31;
        const unsigned nexc = bits;
        const float sc = bad ? __builtin_nanf("") : scale;
        float* tbase = p.out[rank] + (size_t)((tl >> 3) * 128 + (w >> 2) * 64) * NFULL + s * NSH + (tl & 7) * 256 + (w & 3) * 64;
        const LAS char* sb = rbase + sl * LSLOT + lane * 16;
        u32x4 cw[6];
#pragma unroll
        for (int i = 0; i < 4; ++i) cw[i] = *(const LAS u32x4*)(sb + (role * 4 + i) * 1024);
        cw[4] = *(const LAS u32x4*)(sb + 12 * 1024);
        cw[5] = *(const LAS u32x4*)(sb + (13 + role) * 1024);
        u64 ent = 0;
        if (nexc != 0 && nexc != 15u) ent = *(const LAS u64*)(rbase + sl * LSLOT + EXC_OFF + (lane & 15) * 8);
        asm volatile("s_waitcnt lgkmcnt(0)" : "+v"(cw[0]), "+v"(cw[1]), "+v"(cw[2]), "+v"(cw[3]), "+v"(cw[4]), "+v"(cw[5]), "+v"(ent) :: "memory");
        if (lane == 0) __hip_atomic_fetch_add(done + sl, 1u, __ATOMIC_RELAXED, __HIP_MEMORY_SCOPE_WORKGROUP);
        if (nexc != 15u) {
#pragma unroll
          for (int t = 0; t < 4; ++t) {
            unsigned d = ((cw[4][t] >> (1 + role)) & 0x01000000u) | ((cw[4][t] << (3 - role)) & 0x80000000u);
#pragma unroll
            for (int i = 0; i < 4; ++i) d |= ((cw[i][t] >> 25) & 63u) << (6 * i);
            cw[5][t] = as_u(__builtin_amdgcn_ldexpf(as_f(d), 131));
#pragma unroll
            for (int i = 0; i < 5; ++i) cw[i][t] = as_u(__builtin_amdgcn_ldexpf(as_f(cw[i][t] & 0x81FFFFFFu), 131));
          }
        }
#pragma unroll
        for (int i = 0; i < 6; ++i) {
          if (i == 4 && role != 0) continue;
          f32x4 y;
#pragma unroll
          for (int t = 0; t < 4; ++t) y[t] = silu_dq(as_f(cw[i][t]), sc);
          const int row = i < 4 ? role * 16 + i : (i == 4 ? 48 : 49 + role);
          st16_out(tbase + (size_t)row * NFULL, vo, __builtin_bit_cast(u32x4, y));
        }
        if (nexc != 0 && nexc != 15u && !bad) {
          const unsigned pos = (unsigned)(ent >> 32) & 4095u;
          const unsigned qi = (pos >> 10) * 4 + ((pos >> 6) & 3);
          const unsigned owner = qi < 12 ? qi >> 2 : (qi == 12 ? 0 : qi - 13);
          WAIT_VM(0);
          if ((unsigned)lane < nexc && owner == (unsigned)role)
            st_agent(tbase + (size_t)(pos >> 6) * NFULL + (pos & 63u), as_u(silu_dq(as_f((unsigned)ent), scale)));
        }
      }
      if (pp != 40) { WAIT_VM(0); piece_count(myws, pp, call, lane); }
      STAMP(4);
    }
  }
}

extern "C" void kernel_launch(int rank, int nranks, ncclComm_t comm,
                              hipStream_t stream,
                              void* const* d_in, const int* in_sizes, int n_in,
                              void* d_out, int out_size,
                              const int* in_dtypes, int out_dtype,
                              void* const* d_out_all,
                              void* const* d_ws_all, size_t ws_bytes) {
  (void)comm; (void)in_dtypes; (void)out_dtype; (void)d_out;
  if (nranks != NR || n_in != 4 || ws_bytes < WS_NEED || (unsigned)rank >= (unsigned)NR ||
      in_sizes[0] != 4096 * KSH || in_sizes[1] != KSH * NFULL || out_size != 512 * NFULL) {
    fprintf(stderr, "kernel_launch: unexpected problem shape\n");
    return;
  }
  Params p;
  p.x = (const uint8_t*)d_in[0];
  p.w = (const uint8_t*)d_in[1];
  p.sx = (const float*)d_in[2];
  p.sw = (const float*)d_in[3];
  for (int i = 0; i < NR; ++i) {
    p.out[i] = (float*)d_out_all[i];
    p.ws[i] = (char*)d_ws_all[i];
  }
  p.rank = rank;
  p.pad = 0;
  hipLaunchKernelGGL(fused_gemm_rs, dim3(256), dim3(512), 0, stream, p);
  HIP_CHECK(hipGetLastError());
}
```

```cpp
#include <hip/hip_runtime.h>
#include <rccl/rccl.h>
#include <cstdio>
#include <cstdint>

#define NPUSH 4
#define PUSH_WIN 2
#define TAIL_ITEMS 16
#define PULL_WIN 4
#define PULL_WIN_IN 3
#define POLL_SLEEP 8
#define TIMING 0

#define HIP_CHECK(expr) do { hipError_t _e = (expr); if (_e != hipSuccess) { \
    fprintf(stderr, "HIP error %s at %s:%d\n", hipGetErrorString(_e), __FILE__, __LINE__); \
    return; } } while (0)

typedef unsigned u32x4 __attribute__((ext_vector_type(4)));
typedef int i32x4 __attribute__((ext_vector_type(4)));
typedef int i32x8 __attribute__((ext_vector_type(8)));
typedef float f32x4 __attribute__((ext_vector_type(4)));
typedef unsigned long long u64;

#define GAS __attribute__((address_space(1)))
#define LAS __attribute__((address_space(3)))

constexpr int NR = 8;
constexpr int NFULL = 16384;
constexpr int KSH = 512;
constexpr u64 DEADLINE = 100000000ull;

constexpr int NFL = 96;
constexpr size_t OFF_CALL  = 0;
constexpr size_t OFF_PUSHF = 32768;
constexpr size_t OFF_QCNT  = OFF_PUSHF + 1024;
constexpr size_t OFF_TMO   = OFF_QCNT + 2048;
constexpr size_t OFF_READY = OFF_TMO + 128;
constexpr size_t OFF_RFLAG = OFF_READY + 7168;
constexpr size_t OFF_PAGG  = OFF_RFLAG + 7168;
constexpr size_t OFF_FLAG  = 65536;
constexpr size_t OFF_EAGG  = OFF_FLAG + 16 * NFL * 128;
constexpr size_t OFF_TIME  = OFF_EAGG + 96 * 256;
constexpr size_t OFF_A     = 1u << 20;
constexpr size_t OFF_B     = OFF_A + (32u << 20);
constexpr size_t OFF_S     = OFF_B + (16u << 20);
constexpr size_t PAIR      = 9216;
constexpr size_t OFF_O     = OFF_S + 2560 * PAIR;
constexpr size_t SLOT      = 16384;
constexpr size_t WS_NEED   = OFF_O + (size_t)16 * 12 * 8 * SLOT;
static_assert(OFF_PAGG + 4096 <= OFF_FLAG && OFF_TIME + 32768 <= OFF_A, "layout");
constexpr int EXC_CAP = 14;
constexpr int NOPIECE = 63;

constexpr int STAGE = 49152;
constexpr int LDS_BYTES = 3 * STAGE + 256;

struct Params {
  const uint8_t* x;
  const uint8_t* w;
  const float* sx;
  const float* sw;
  float* out[NR];
  char* ws[NR];
  int rank;
  int pad;
};

__device__ __forceinline__ float as_f(unsigned u) { return __builtin_bit_cast(float, u); }
__device__ __forceinline__ unsigned as_u(float f) { return __builtin_bit_cast(unsigned, f); }
__device__ __forceinline__ void st16_sys(const void* base, unsigned off, u32x4 v) {
  asm volatile("global_store_dwordx4 %0, %1, %2 sc0 sc1\n\ts_nop 1" :: "v"(off), "v"(v), "s"(base) : "memory");
}
#define OUT_ST "sc1"
__device__ __forceinline__ void st16_out(const void* base, unsigned off, u32x4 v) {
  asm volatile("global_store_dwordx4 %0, %1, %2 " OUT_ST "\n\ts_nop 1" :: "v"(off), "v"(v), "s"(base) : "memory");
}
__device__ __forceinline__ void ticket_issue(unsigned& t, const void* ctr) {
  u64 sv;
  asm volatile("s_mov_b64 %1, exec\n\ts_mov_b64 exec, 1\n\tglobal_atomic_add %0, %2, %3, %4 sc0\n\ts_mov_b64 exec, %1"
               : "=&v"(t), "=&s"(sv) : "v"(0u), "v"(1u), "s"(ctr) : "memory");
}
#define WAIT_VM(n) asm volatile("s_waitcnt vmcnt(%0)" :: "i"(n) : "memory")

__device__ __forceinline__ unsigned ld_sys(const void* p) {
  return __hip_atomic_load((const GAS unsigned*)p, __ATOMIC_RELAXED, __HIP_MEMORY_SCOPE_SYSTEM);
}
__device__ __forceinline__ u64 ld_sys64(const void* p) {
  return __hip_atomic_load((const GAS u64*)p, __ATOMIC_RELAXED, __HIP_MEMORY_SCOPE_SYSTEM);
}
__device__ __forceinline__ unsigned ld_agent(const void* p) {
  return __hip_atomic_load((const GAS unsigned*)p, __ATOMIC_RELAXED, __HIP_MEMORY_SCOPE_AGENT);
}
__device__ __forceinline__ void st_agent(void* p, unsigned v) {
  __hip_atomic_store((GAS unsigned*)p, v, __ATOMIC_RELAXED, __HIP_MEMORY_SCOPE_AGENT);
}
__device__ __forceinline__ void st_sys(void* p, unsigned v) {
  __hip_atomic_store((GAS unsigned*)p, v, __ATOMIC_RELAXED, __HIP_MEMORY_SCOPE_SYSTEM);
}

__device__ __forceinline__ void wait_ge(const void* p, unsigned target, void* tmo, unsigned call) {
  u64 t0 = __builtin_amdgcn_s_memrealtime();
  for (unsigned n = 1; (int)(ld_sys(p) - target) < 0; ++n) {
    if ((n & 15u) == 0 && (ld_agent(tmo) == call || __builtin_amdgcn_s_memrealtime() - t0 > DEADLINE)) {
      st_agent(tmo, call);
      break;
    }
    __builtin_amdgcn_s_sleep(POLL_SLEEP);
  }
}

__device__ __forceinline__ float silu_dq(float acc23, float s) {
  float y = (acc23 * 1.1920928955078125e-07f) * s;
  float c = fminf(fmaxf(y, -60.0f), 60.0f);
  return y * __builtin_amdgcn_rcpf(1.0f + __builtin_amdgcn_exp2f(c * -1.44269504088896f));
}
__device__ __forceinline__ unsigned to_i32(float a) {
  int r;
  asm("v_cvt_i32_f32 %0, %1" : "=v"(r) : "v"(a));
  return (unsigned)r;
}
__device__ __forceinline__ bool too_big(float a) { return !(__builtin_fabsf(a) < 2147483648.0f); }

__device__ __forceinline__ int same_peer(int r, int i) { return (r & 4) | ((r + 1 + i) & 3); }
__device__ __forceinline__ int partner(int r, int q) { return r < 4 ? 4 | ((r + q) & 3) : (r - q) & 3; }
__device__ __forceinline__ int wblock(int r, int n) {
  const int t = partner(r, n & 3);
  return (r < 4 ? r * 4 + (t & 3) : t * 4 + (r & 3)) * 4 + (n >> 2);
}
__device__ __forceinline__ int x1_block(int r, int k) { return k < 3 ? same_peer(r, k) * 4 : partner(r, (k - 2) & 3) * 4 + 2; }
__device__ __forceinline__ int push_index(int r, int mb) {
  int k = -1;
#pragma unroll
  for (int i = 0; i < NPUSH; ++i) if (x1_block(r, i) == mb) k = i;
  return k;
}
__device__ __forceinline__ int tile_sub(int t) { return t < 3 ? 0 : (t < 6 ? 2 : (t < 9 ? 3 : (t < 12 ? 1 : t - 12))); }
__device__ __forceinline__ void x_piece(int r, int mb, int mt, int& piece, int& need) {
  const bool tail = (mb & 3) == 1 && mt >= 6;
  piece = tail ? 32 + (mb >> 2) : mb;
  need = tail ? 32 : ((mb & 3) == 1 ? 96 : (push_index(r, mb) >= 0 ? 16 : 128));
}

#define STAMP(i) do {} while (0)
#define STAMP2(i) do {} while (0)

__device__ __forceinline__ void x_load(const Params& p, int mb, int e, int lane, unsigned (&w)[16]) {
  const __amdgpu_buffer_rsrc_t rs = __builtin_amdgcn_make_buffer_rsrc((void*)p.x, 0, 4096 * KSH, 0x00020000);
  const unsigned vo = (lane & 15) * KSH + (lane >> 4) * 16;
  const unsigned so = (mb * 128 + (e >> 1) * 16) * KSH + (e & 1) * 256;
#pragma unroll
  for (int u = 0; u < 4; ++u) {
    u32x4 v = __builtin_amdgcn_raw_buffer_load_b128(rs, vo, so + u * 64, 0);
    w[4 * u] = v[0]; w[4 * u + 1] = v[1]; w[4 * u + 2] = v[2]; w[4 * u + 3] = v[3];
  }
}
__device__ __forceinline__ void w_load(const Params& p, int col0, int e, int lane, unsigned (&r)[16]) {
  const __amdgpu_buffer_rsrc_t rs = __builtin_amdgcn_make_buffer_rsrc((void*)p.w, 0, KSH * NFULL, 0x00020000);
  const unsigned vo = (lane >> 4) * 16 * NFULL + (lane & 15) * 4;
  const unsigned so = (e >> 2) * 64 * NFULL + col0 + (e & 3) * 64;
#pragma unroll
  for (int i = 0; i < 16; ++i) r[i] = __builtin_amdgcn_raw_buffer_load_b32(rs, vo, so + i * NFULL, 0);
}
__device__ __forceinline__ void w_transpose(unsigned (&r)[16]) {
  unsigned w[16];
#pragma unroll
  for (int u = 0; u < 4; ++u) {
    const unsigned a0 = __builtin_amdgcn_perm(r[4 * u + 1], r[4 * u], 0x05010400u), a1 = __builtin_amdgcn_perm(r[4 * u + 1], r[4 * u], 0x07030602u);
    const unsigned b0 = __builtin_amdgcn_perm(r[4 * u + 3], r[4 * u + 2], 0x05010400u), b1 = __builtin_amdgcn_perm(r[4 * u + 3], r[4 * u + 2], 0x07030602u);
    w[u] = __builtin_amdgcn_perm(b0, a0, 0x05040100u);
    w[4 + u] = __builtin_amdgcn_perm(b0, a0, 0x07060302u);
    w[8 + u] = __builtin_amdgcn_perm(b1, a1, 0x05040100u);
    w[12 + u] = __builtin_amdgcn_perm(b1, a1, 0x07060302u);
  }
#pragma unroll
  for (int i = 0; i < 16; ++i) r[i] = w[i];
}
__device__ __forceinline__ void chunk_place(bool isw, int blk, int e, int src, unsigned par, unsigned& dst, unsigned& s01, unsigned& s23) {
  if (isw) {
    dst = (unsigned)OFF_B + (blk * 32 + src * 4 + (e >> 3)) * 32768u + (e & 3) * 8192 + ((e >> 2) & 1) * 1024;
    s01 = 2048; s23 = 4096;
  } else {
    dst = (unsigned)OFF_A + ((par * 32 + blk) * 32 + src * 4 + (e & 1) * 2) * 16384u + (e >> 1) * 2048;
    s01 = 1024; s23 = 16384;
  }
}
__device__ __forceinline__ void chunk_store(const char* base, unsigned dst, unsigned s01, unsigned s23, int lane, const unsigned (&w)[16]) {
  st16_sys(base + dst, lane * 16, u32x4{w[0], w[1], w[2], w[3]});
  st16_sys(base + dst + s01, lane * 16, u32x4{w[4], w[5], w[6], w[7]});
  st16_sys(base + dst + s23, lane * 16, u32x4{w[8], w[9], w[10], w[11]});
  st16_sys(base + dst + s23 + s01, lane * 16, u32x4{w[12], w[13], w[14], w[15]});
}

__device__ __forceinline__ unsigned wave_scan_incl(unsigned v) {
  v += __builtin_amdgcn_update_dpp(0u, v, 0x111, 0xf, 0xf, true);
  v += __builtin_amdgcn_update_dpp(0u, v, 0x112, 0xf, 0xf, true);
  v += __builtin_amdgcn_update_dpp(0u, v, 0x114, 0xf, 0xf, true);
  v += __builtin_amdgcn_update_dpp(0u, v, 0x118, 0xf, 0xf, true);
  v += __builtin_amdgcn_update_dpp(0u, v, 0x142, 0xa, 0xf, false);
  v += __builtin_amdgcn_update_dpp(0u, v, 0x143, 0xc, 0xf, false);
  return v;
}

constexpr int ESC_CAP = 1120;
constexpr int SCR = 1344;
__device__ __forceinline__ bool chunk_pack(const unsigned (&w)[16], const char* pair, int h, LAS char* scr, int lane) {
  const char* out = pair + h * 3072;
  unsigned m[16], P[4] = {0, 0, 0, 0}, big = 0, cnt = 0;
#pragma unroll
  for (int i = 0; i < 16; ++i) {
    const unsigned e = (w[i] >> 3) & 0x0F0F0F0Fu;
    const unsigned in47 = (e >> 2) & ~(e >> 3) & 0x01010101u;
    const unsigned c = e & (in47 | (in47 << 1));
    m[i] = ~(c | (c >> 1)) & 0x01010101u;
    P[i >> 2] |= c << (2 * (i & 3));
    big |= e + 0x05050505u;
    cnt += __builtin_popcount(m[i]);
  }
  const unsigned incl = wave_scan_incl(cnt);
  const unsigned total = __builtin_amdgcn_readlane(incl, 63);
  if (total > (unsigned)ESC_CAP || __builtin_amdgcn_ballot_w64((big & 0x10101010u) != 0) != 0) {
#pragma unroll
    for (int b = 0; b < 4; ++b)
      st16_sys(b < 3 ? out + b * 1024 : pair + 7168 + h * 1024, lane * 16, u32x4{w[4 * b], w[4 * b + 1], w[4 * b + 2], w[4 * b + 3]});
    return true;
  }
  unsigned pos = incl - cnt;
  const unsigned dummy = 1152 + lane;
#pragma unroll
  for (int i = 0; i < 16; ++i) {
#pragma unroll
    for (int k = 0; k < 4; ++k) {
      const unsigned mk = (m[i] >> (8 * k)) & 1u;
      scr[mk ? pos : dummy] = (char)(w[i] >> (8 * k + 3));
      pos += mk;
    }
  }
  u32x4 n0, n1;
#pragma unroll
  for (int i = 0; i < 4; ++i) {
    n0[i] = (w[2 * i] & 0x87878787u) | ((w[2 * i + 1] << 3) & 0x38383838u) | ((w[2 * i + 1] >> 1) & 0x40404040u);
    n1[i] = (w[8 + 2 * i] & 0x87878787u) | ((w[9 + 2 * i] << 3) & 0x38383838u) | ((w[9 + 2 * i] >> 1) & 0x40404040u);
  }
  st16_sys(out, lane * 16, n0);
  st16_sys(out + 1024, lane * 16, n1);
  st16_sys(out + 2048, lane * 16, u32x4{P[0], P[1], P[2], P[3]});
  asm volatile("s_waitcnt lgkmcnt(0)" ::: "memory");
  if (lane < 56) {
    unsigned b[5];
#pragma unroll
    for (int i = 0; i < 5; ++i) {
      const unsigned e = *(const LAS unsigned*)(scr + lane * 20 + i * 4) & 0x0F0F0F0Fu;
      b[i] = (e - ((e >> 3) & 0x01010101u) * 3u) & 0x07070707u;
    }
    const u64 o = (u64)(b[0] | (b[1] << 3) | ((b[4] & 0x03030303u) << 6)) |
                  ((u64)(b[2] | (b[3] << 3) | ((b[4] & 0x04040404u) << 4)) << 32);
    __hip_atomic_store((GAS u64*)(pair + 6144 + h * 448 + lane * 8), o, __ATOMIC_RELAXED, __HIP_MEMORY_SCOPE_SYSTEM);
  }
  asm volatile("s_waitcnt lgkmcnt(0)" ::: "memory");
  return false;
}

template <class Mid>
__device__ __forceinline__ void chunk_unpack(const LAS char* pair, int h, bool raw, LAS char* scr, int lane, unsigned (&w)[16], Mid&& mid) {
  const LAS char* in = pair + h * 3072;
  const u32x4 P = *(const LAS u32x4*)(in + 2048 + lane * 16);
  const u64 o = *(const LAS u64*)(pair + 6144 + h * 448 + lane * 8);
  const u32x4 n0 = *(const LAS u32x4*)(in + lane * 16), n1 = *(const LAS u32x4*)(in + 1024 + lane * 16);
  if (raw) {
    const u32x4 r3 = *(const LAS u32x4*)(pair + 7168 + h * 1024 + lane * 16);
#pragma unroll
    for (int i = 0; i < 4; ++i) { w[i] = n0[i]; w[4 + i] = n1[i]; w[8 + i] = P[i]; w[12 + i] = r3[i]; }
    mid();
    return;
  }
  {
    const unsigned o0 = (unsigned)o, o1 = (unsigned)(o >> 32);
    unsigned b[5] = {o0 & 0x07070707u, (o0 >> 3) & 0x07070707u, o1 & 0x07070707u, (o1 >> 3) & 0x07070707u,
                     ((o0 >> 6) & 0x03030303u) | ((o1 >> 4) & 0x04040404u)};
#pragma unroll
    for (int i = 0; i < 5; ++i) {
      const unsigned hi = ((b[i] + 0x03030303u) >> 3) & 0x01010101u;
      *(LAS unsigned*)(scr + lane * 20 + i * 4) = ((b[i] + hi + (hi << 1)) << 3) ^ 0x20202020u;
    }
  }
  unsigned M[4], cnt = 0;
#pragma unroll
  for (int k = 0; k < 4; ++k) { M[k] = ~(P[k] | (P[k] >> 1)) & 0x55555555u; cnt += __builtin_popcount(M[k]); }
  unsigned pos = ((wave_scan_incl(cnt) - cnt) & 0x7FFu) + (unsigned)(size_t)scr;
  asm volatile("s_waitcnt lgkmcnt(0)" ::: "memory");
  unsigned lo[16], hi[16], ps[16];
#pragma unroll
  for (int i = 0; i < 16; ++i) {
    const LAS unsigned* q = (const LAS unsigned*)(size_t)(pos & ~3u);
    lo[i] = q[0]; hi[i] = q[1]; ps[i] = pos;
    pos += __builtin_popcount(M[i >> 2] & (0x01010101u << (2 * (i & 3))));
  }
  __builtin_amdgcn_sched_barrier(0);
  mid();
  __builtin_amdgcn_sched_barrier(0);
#pragma unroll
  for (int i = 0; i < 16; ++i) {
    const int sh = 2 * (i & 3);
    const unsigned m = (M[i >> 2] >> sh) & 0x01010101u;
    unsigned t;
    asm("v_mul_u32_u24 %0, %1, %2" : "=v"(t) : "s"(0x010101u), "v"(m));
    const unsigned sel = (t << 8) | ((m << 2) ^ 0x04040404u);
    const unsigned esc = __builtin_amdgcn_perm(0u, __builtin_amdgcn_alignbyte(hi[i], lo[i], ps[i]), sel);
    const unsigned ex = ((sh <= 3 ? P[i >> 2] << (3 - sh) : P[i >> 2] >> (sh - 3)) & 0x18181818u) | 0x20202020u;
    const unsigned n = i < 8 ? n0[(i >> 1) & 3] : n1[(i >> 1) & 3];
    const unsigned sm = (i & 1) ? (((n >> 3) & 0x07070707u) | ((n << 1) & 0x80808080u)) : (n & 0x87878787u);
    w[i] = sm | (ex ^ esc);
  }
}

__device__ __forceinline__ void piece_flag(char* myws, int piece, int need, unsigned t, unsigned call, int lane) {
  if (t + 1 == call * (unsigned)need && lane == 0) st_agent(myws + OFF_RFLAG + (size_t)piece * 128, call);
}
__device__ __forceinline__ void piece_count(char* myws, int piece, int need, unsigned call, int lane) {
  unsigned t;
  ticket_issue(t, myws + OFF_READY + (size_t)piece * 128);
  asm volatile("s_waitcnt vmcnt(0)" : "+v"(t) :: "memory");
  piece_flag(myws, piece, need, __builtin_amdgcn_readfirstlane(t), call, lane);
}

constexpr int NIN = (7 - NPUSH) * 8 + 346;
constexpr int NITEM = NIN + 256;
__device__ __forceinline__ void item_describe(int r, int s, int t, unsigned& d0, unsigned& d1, unsigned& d2) {
  if (t < NIN) {
    constexpr int N1 = (7 - NPUSH) * 8;
    int blk, h;
    bool isw = false;
    int u = t - N1;
    if (u < 0) { blk = x1_block(r, NPUSH + (t >> 3)); h = t & 7; }
    else if (u < 64) { isw = true; blk = u >> 4; h = u & 15; }
    else if (u < 96) { u -= 64; blk = partner(r, ((u >> 3) + 1) & 3) * 4 + 3; h = u & 7; }
    else if (u < 114) { u -= 96; blk = same_peer(r, u / 6) * 4 + 1; h = u % 6; }
    else if (u < 122) { u -= 114; blk = partner(r, ((u >> 1) + 1) & 3) * 4 + 1; h = 6 + (u & 1); }
    else if (u < 314) { u -= 122; isw = true; blk = 4 + (u >> 4); h = u & 15; }
    else { u -= 314; blk = r * 4 + (u >> 3); h = u & 7; }
    int piece = 40 + blk, need = 256;
    if (!isw) x_piece(r, blk, h, piece, need);
    unsigned dst, s01, s23;
    chunk_place(isw, blk, 2 * h, s, 0, dst, s01, s23);
    d0 = (unsigned)OFF_S + (isw ? 256 + wblock(r, blk) * 16 + h : blk * 8 + h) * (unsigned)PAIR;
    d1 = (isw ? 32 + blk : blk) * 128 | (2 * h) << 16;
    d2 = dst >> 10 | (unsigned)isw << 17 | piece << 18 | (need >> 4) << 24;
  } else {
    const int u = t - NIN;
    int n, tl, w;
    if (((r ^ s) & 4) == 0) {
      const int i = (r - s - 1) & 3;
      if (u < 64) { n = (u >> 3) & 3; tl = u < 32 ? i : 9 + i; w = u & 7; }
      else { n = u >> 4; tl = (u & 1) ? 9 + i : i; w = (u >> 1) & 7; }
    } else {
      const int qr = (r < 4 ? s - r : r - s) & 3;
      int qq, c = 0, kind;
      if (u < 24) { qq = u >> 3; kind = 0; w = u & 7; }
      else if (u < 60) { const int x = (u - 24) % 12; qq = (u - 24) / 12; kind = x < 8 ? 1 : 2; w = x < 8 ? x : x - 4; }
      else { const int b = (u - 60) / 20, x = (u - 60) % 20; c = (1 + b / 3) & 3; qq = b % 3; kind = x < 16 ? x & 1 : 2; w = x < 16 ? x >> 1 : x - 12; }
      const int q = qq + (qq >= qr ? 1 : 0);
      n = c * 4 + q;
      tl = 3 + 3 * kind + ((qr - q - 1) & 3);
    }
    const bool half = tl >= 9 && w >= 4;
    const int hi = half && ((r ^ s) & 4) != 0;
    d0 = (unsigned)OFF_O + ((n * 12 + tl) * 8 + w) * (unsigned)SLOT + hi * 8192;
    d1 = ((48 + n * 3 + (tl < 6 ? 0 : (tl < 9 ? 1 : 2))) * 128 + w * 16) | (half ? 2u : 1u) << 24;
    d2 = ((tile_sub(tl) * 128 + (w >> 2) * 64) * NFULL + wblock(s, n) * 256 + (w & 3) * 64) | (hi * 2) << 24;
  }
}

constexpr int RING = 4;
constexpr int LSLOT = 16384;
static_assert(2 * RING * LSLOT + 6 * SCR + NITEM * 12 <= 3 * STAGE, "LDS");

__device__ __forceinline__ unsigned lds_peek(LAS unsigned* p) {
  unsigned v;
  asm volatile("ds_read_b32 %0, %1\n\ts_waitcnt lgkmcnt(0)" : "=&v"(v) : "v"(p) : "memory");
  return __builtin_amdgcn_readfirstlane(v);
}
__device__ __forceinline__ u64 uni64(u64 v) {
  return ((u64)(unsigned)__builtin_amdgcn_readfirstlane((unsigned)(v >> 32)) << 32) | (u64)(unsigned)__builtin_amdgcn_readfirstlane((unsigned)v);
}

__global__ void __launch_bounds__(512, 2) fused_gemm_rs(Params p) {
  __shared__ __attribute__((aligned(16))) char smem[LDS_BYTES];
  const int tid = threadIdx.x;
  const int lane = tid & 63;
  const int wave = __builtin_amdgcn_readfirstlane(tid >> 6);
  const int rank = p.rank;
  const int j = blockIdx.x & 7;
  const int grp = blockIdx.x >> 3;
  char* const myws = p.ws[rank];
  volatile LAS unsigned* sctl = (volatile LAS unsigned*)(LAS char*)(smem + 3 * STAGE);

  const int peer = (rank + 1 + grp) & 7;
  unsigned w0[16], wp[NPUSH / 2][16];
  const int pw = j * 4 + wave;
  const int ew = (wave >> 1) * 400 + (blockIdx.x - 56) * 2 + (wave & 1);
  auto enc_id = [&](int n, bool& isw, int& a, int& b, int& e) {
    isw = (n >= 256 && n < 768) || n >= 1024;
    if (isw) { const int i = (n < 768 ? n - 256 : n - 512) >> 5; a = i & 3; b = i >> 2; e = n & 31; }
    else { a = ((n >> 4) & 7) * 4 + ((n & 128) ? 0 : 2) + (n >= 768 ? 1 : 0); b = 0; e = n & 15; }
  };
  auto enc_load = [&](int n, unsigned (&w)[16]) {
    bool isw; int a, b, e;
    enc_id(n, isw, a, b, e);
    if (isw) w_load(p, wblock(a, b) * 256, e, lane, w); else x_load(p, a, e, lane, w);
  };
  LAS unsigned* const table = (LAS unsigned*)((LAS char*)smem + 2 * RING * LSLOT + 6 * SCR);
  unsigned cticket;
  ticket_issue(cticket, myws + OFF_CALL + (size_t)blockIdx.x * 128 + wave * 16);
  STAMP(0);
  if (grp >= 7) enc_load(ew, w0);
  else if (wave < 4) {
#pragma unroll
    for (int it = 0; it < NPUSH / 2; ++it) x_load(p, x1_block(peer, (pw >> 4) + 2 * it), pw & 15, lane, wp[it]);
  }
  if (grp < 7) {
    if (tid >= 16 && tid < 48) sctl[tid] = 0;
    if (wave >= 4)
      for (int t = tid - 256; t < NITEM; t += 256) {
        unsigned d0, d1, d2;
        item_describe(rank, peer, t, d0, d1, d2);
        table[t * 3] = d0; table[t * 3 + 1] = d1; table[t * 3 + 2] = d2;
      }
    asm volatile("s_waitcnt lgkmcnt(0)" ::: "memory");
    __builtin_amdgcn_s_barrier();
  }
  asm volatile("s_waitcnt vmcnt(0)" : "+v"(cticket) :: "memory");
  const unsigned call = (unsigned)__builtin_amdgcn_readfirstlane(cticket) + 1;
  const unsigned par = call & 1;
  void* const tmo = myws + OFF_TMO;
  const float scale = p.sx[0] * p.sw[0];

  const bool dbg_skip = false;
  if (dbg_skip) {
  } else if (grp >= 7) {
    LAS char* const scr = (LAS char*)smem + wave * SCR;
    if (wave < 2) __builtin_amdgcn_s_setprio(3); else if (wave < 4) __builtin_amdgcn_s_setprio(2); else if (wave < 6) __builtin_amdgcn_s_setprio(1);
    for (int n = ew; n < 2560; n += 1600) {
      bool isw; int a, b, e;
      enc_id(n, isw, a, b, e);
      if (n != ew) enc_load(n, w0);
      if (isw) w_transpose(w0);
      const int other = isw ? partner(a, b & 3) : 0;
      const bool low = (a & 3) == 0 || ((a & 3) == 1 && e < 12);
      const bool mine = isw ? (a == rank || other == rank) : ((a >> 2) == rank || low == ((((a >> 2) ^ rank) & 4) == 0));
      const int sp = isw ? 32 + wblock(a, b) : a;
      {
        const int slot = isw ? 512 + (sp - 32) * 32 + e : a * 16 + e;
        const bool raw = chunk_pack(w0, myws + OFF_S + ((size_t)par * 1280 + (slot >> 1)) * PAIR, slot & 1, scr, lane);
        WAIT_VM(0);
        char* const agg = myws + OFF_EAGG + (size_t)sp * 256;
        const u64 mine64 = (1ull << 32) | (raw ? 1u << e : 0u);
        u64 tot = 0;
        if (lane == 0) tot = __hip_atomic_fetch_add((GAS u64*)(agg + par * 128), mine64, __ATOMIC_RELAXED, __HIP_MEMORY_SCOPE_AGENT);
        tot = uni64(tot) + mine64;
        if ((unsigned)(tot >> 32) == (isw ? 32u : 16u)) {
          if (lane < NR && lane != rank && (!isw || lane == a || lane == other))
            __hip_atomic_store((GAS u64*)(p.ws[lane] + OFF_FLAG + (size_t)((par * 8 + rank) * NFL + (isw ? 32 + b : a)) * 128),
                               ((u64)call << 32) | (unsigned)tot, __ATOMIC_RELAXED, __HIP_MEMORY_SCOPE_SYSTEM);
          if (lane == 0) __hip_atomic_store((GAS u64*)(agg + (par ^ 1) * 128), 0ull, __ATOMIC_RELAXED, __HIP_MEMORY_SCOPE_AGENT);
        }
      }
      if (mine) {
        unsigned dst, s01, s23;
        chunk_place(isw, isw ? b : a, e, rank, par, dst, s01, s23);
        chunk_store(myws, dst, s01, s23, lane, w0);
        int piece = 40 + b, need = 256;
        if (!isw) x_piece(rank, a, e >> 1, piece, need);
        WAIT_VM(0);
        piece_count(myws, piece, need, call, lane);
      }
    }
    __builtin_amdgcn_s_setprio(0);
  } else if (wave < 4) {
    char* const dws = p.ws[peer];
#pragma unroll
    for (int it = 0; it < NPUSH / 2; ++it) {
      unsigned dst, s01, s23;
      chunk_place(false, x1_block(peer, (pw >> 4) + 2 * it), pw & 15, rank, par, dst, s01, s23);
      const unsigned (&w)[16] = wp[it];
#pragma unroll
      for (int b = 0; b < 4; ++b) {
        st16_sys(dws + dst + (b & 1) * s01 + (b >> 1) * s23, lane * 16, u32x4{w[4 * b], w[4 * b + 1], w[4 * b + 2], w[4 * b + 3]});
        WAIT_VM(PUSH_WIN - 1);
      }
    }
    WAIT_VM(0);
    unsigned ag[NPUSH / 2];
#pragma unroll
    for (int it = 0; it < NPUSH / 2; ++it) ticket_issue(ag[it], myws + OFF_PAGG + (size_t)(peer * 8 + (pw >> 4) + 2 * it) * 64);
#pragma unroll
    for (int it = 0; it < NPUSH / 2; ++it) {
      asm volatile("s_waitcnt vmcnt(0)" : "+v"(ag[it]) :: "memory");
      if ((unsigned)__builtin_amdgcn_readfirstlane(ag[it]) + 1 == call * 16u && lane == 0)
        st_sys(dws + OFF_PUSHF + (size_t)((pw >> 4) + 2 * it) * 128 + rank * 4, call);
    }
  }
  if (blockIdx.x == 64 && tid < NR)
    st_agent(myws + OFF_QCNT + (size_t)((par ^ 1) * 8 + tid) * 128, 0u);
  STAMP(1);

  if (grp >= 7) {
    const int wm = wave >> 2, G = wave & 3;
    const int npass = grp < 14 ? 2 : 1;
    for (int pass = 0; pass < npass; ++pass) {
      const int pos = grp - 7 + 25 * pass;
      const int n = pos < 24 ? j + (pos >= 12 ? 8 : 0) : j + 8 * (pos & 1);
      const int tl = pos < 12 ? pos : (pos < 24 ? pos - 12 : 12 + ((pos - 24) >> 1));
      const int sub = tile_sub(tl);
      const int cp = partner(rank, (n + 1 + tl % 3) & 3);
      const int dA = tl >= 12 ? rank : ((tl < 3 || tl >= 9) ? same_peer(rank, tl % 3) : cp);
      const int dB = (tl >= 9 && tl < 12) ? cp : dA;
      const int pk = push_index(rank, dA * 4 + sub);
      if (!dbg_skip && (tid < 2 || (tid == 2 && sub == 1) || (tid >= 3 && tid < 11 && pk >= 0 && tid - 3 != rank)))
        wait_ge(myws + (tid < 3 ? OFF_RFLAG + (size_t)(tid == 0 ? 40 + n : (tid == 1 ? dA * 4 + sub : 32 + dB)) * 128
                                : OFF_PUSHF + (size_t)pk * 128 + (tid - 3) * 4), call, tmo, call);
      __syncthreads();
      STAMP(2 + 3 * pass);
      unsigned tmov;
      asm volatile("global_load_dword %0, %1, %2 sc1" : "=&v"(tmov) : "v"(0u), "s"(tmo) : "memory");
      f32x4 acc[4][4];
      {
        const __amdgpu_buffer_rsrc_t rs = __builtin_amdgcn_make_buffer_rsrc(myws, 0, (int)WS_NEED, 0x00020000);
        const unsigned ga = (unsigned)OFF_A + (par * 32 + (wave < 6 ? dA : dB) * 4 + sub) * (32u * 16384u) + wave * 2048;
        const unsigned gb = (unsigned)OFF_B + n * (32u * 32768u) + wave * 4096;
        const unsigned l16 = lane * 16;
        auto dma = [&](int ks, int buf, int i) {
          if (i < 2)
            __builtin_amdgcn_raw_ptr_buffer_load_lds(rs, (LAS void*)((LAS char*)smem + buf * STAGE + wave * 2048 + i * 1024), 16,
                                                     l16, ga + ks * 16384 + i * 1024, 0, 17);
          else
            __builtin_amdgcn_raw_ptr_buffer_load_lds(rs, (LAS void*)((LAS char*)smem + buf * STAGE + 16384 + wave * 4096 + (i - 2) * 1024), 16,
                                                     l16, gb + ks * 32768 + (i - 2) * 1024, 0, 17);
        };
        const LAS char* const la = (const LAS char*)smem + wm * 8192 + lane * 16;
        const LAS char* const lb = (const LAS char*)smem + 16384 + G * 8192 + lane * 16;
        auto frag = [&](const LAS char* q) -> i32x8 {
          const i32x4 lo = *(const LAS i32x4*)q, hi = *(const LAS i32x4*)(q + 1024);
          return __builtin_shufflevector(lo, hi, 0, 1, 2, 3, 4, 5, 6, 7);
        };
#define MMA(m, t, CA, CB) acc[m][t] = __builtin_amdgcn_mfma_scale_f32_16x16x128_f8f6f4(CA[m], CB[t], acc[m][t], 0, 0, 0, 150, 0, 127)
#define GEMM_STEP(KS, VM, CA, CB, NA, NB, DO_STAGE) do { \
          WAIT_VM(VM); \
          asm volatile("s_waitcnt lgkmcnt(0)" ::: "memory"); \
          __builtin_amdgcn_s_barrier(); \
          __builtin_amdgcn_sched_barrier(0); \
          const int nbuf = buf == 2 ? 0 : buf + 1; \
          _Pragma("unroll") for (int g = 0; g < 16; ++g) { \
            MMA(g >> 2, g & 3, CA, CB); \
            __builtin_amdgcn_sched_barrier(0); \
            if (g < 8) { \
              if (g & 1) NB[g >> 1] = frag(lb + nbuf * STAGE + (g >> 1) * 2048); \
              else NA[g >> 1] = frag(la + nbuf * STAGE + (g >> 1) * 2048); \
            } else if (DO_STAGE && g < 14) dma((KS) + 3, buf, g - 8); \
            __builtin_amdgcn_sched_barrier(0); \
          } \
          buf = nbuf; } while (0)
        i32x8 a0[4], b0[4], a1[4], b1[4];
#pragma unroll
        for (int i = 0; i < 18; ++i) dma(i / 6, i / 6, i % 6);
        asm volatile("s_waitcnt vmcnt(12)" : "+v"(tmov) :: "memory");
        __builtin_amdgcn_s_barrier();
        const float acc0 = tmov == call ? __builtin_nanf("") : 0.0f;
#pragma unroll
        for (int g = 0; g < 16; ++g) acc[g >> 2][g & 3] = f32x4{acc0, acc0, acc0, acc0};
#pragma unroll
        for (int t = 0; t < 4; ++t) { a0[t] = frag(la + t * 2048); b0[t] = frag(lb + t * 2048); }
        int buf = 0;
        for (int ks = 0; ks < 28; ks += 2) {
          GEMM_STEP(ks, 6, a0, b0, a1, b1, true);
          GEMM_STEP(ks + 1, 6, a1, b1, a0, b0, true);
        }
#define PIN_ACC() _Pragma("unroll") for (int g = 0; g < 16; ++g) asm volatile("" : "+v"(acc[g >> 2][g & 3]))
        GEMM_STEP(28, 6, a0, b0, a1, b1, true);
        PIN_ACC();
        GEMM_STEP(29, 6, a1, b1, a0, b0, false);
        PIN_ACC();
        GEMM_STEP(30, 0, a0, b0, a1, b1, false);
        PIN_ACC();
#pragma unroll
        for (int g = 0; g < 16; ++g) MMA(g >> 2, g & 3, a1, b1);
        PIN_ACC();
      }
      STAMP(3 + 3 * pass);

      int ln = lane;
      asm volatile("" : "+v"(ln));
      const int q = ln >> 4, c = ln & 15;
      if (tl >= 12) {
        const float* dst = p.out[rank] + (size_t)(sub * 128 + wm * 64) * NFULL + wblock(rank, n) * 256 + G * 64;
        const unsigned vo = (4 * q * NFULL + 4 * c) * 4;
#pragma unroll
        for (int m = 0; m < 4; ++m)
#pragma unroll
          for (int r = 0; r < 4; ++r) {
            f32x4 v;
#pragma unroll
            for (int t = 0; t < 4; ++t) v[t] = silu_dq(acc[m][t][r], scale);
            st16_out(dst + (size_t)(m * 16 + r) * NFULL, vo, __builtin_bit_cast(u32x4, v));
          }
      } else {
        char* slot = myws + OFF_O + ((size_t)(n * 12 + tl) * 8 + wave) * SLOT;
        const bool mixed = dA != dB && wm == 1;
        char* const meta = slot + (mixed ? 6144 : 12288);
        const int g2 = mixed ? 8192 : 6144;
        if (wave < 2) __builtin_amdgcn_s_setprio(3); else if (wave < 4) __builtin_amdgcn_s_setprio(2); else if (wave < 6) __builtin_amdgcn_s_setprio(1);
        unsigned gmask = 0;
        unsigned lowm[2] = {0, 0};
        unsigned nexc = 0;
        auto emit = [&](u64 mask, bool on, unsigned pos, unsigned val) {
          const unsigned idx = nexc + __builtin_amdgcn_mbcnt_hi((unsigned)(mask >> 32), __builtin_amdgcn_mbcnt_lo((unsigned)mask, 0u));
          if (on && idx < (unsigned)EXC_CAP) {
            __hip_atomic_store((GAS u64*)(meta + 256 + idx * 8), ((u64)pos << 32) | val, __ATOMIC_RELAXED, __HIP_MEMORY_SCOPE_SYSTEM);
            if (mixed) __hip_atomic_store((GAS u64*)(meta + 8192 + 256 + idx * 8), ((u64)pos << 32) | val, __ATOMIC_RELAXED, __HIP_MEMORY_SCOPE_SYSTEM);
          }
          nexc += __builtin_popcountll(mask);
        };
#pragma unroll
        for (int m = 0; m < 4; ++m) {
          u32x4 v[3];
#pragma unroll
          for (int t = 0; t < 4; ++t) {
            unsigned I[4];
            u64 mis = 0;
#pragma unroll
            for (int r = 0; r < 4; ++r) {
              I[r] = to_i32(acc[m][t][r]);
              mis |= __builtin_amdgcn_ballot_w64(too_big(acc[m][t][r]));
            }
            asm volatile("s_cmp_lg_u64 %1, 0\n\ts_addc_u32 %0, %0, %0" : "+s"(gmask) : "s"(mis) : "scc");
            const unsigned L = __builtin_amdgcn_perm(I[1], I[0], 0x0C0C0400u) | __builtin_amdgcn_perm(I[3], I[2], 0x04000C0Cu);
            lowm[m >> 1] = (lowm[m >> 1] >> 1) | (L & 0x80808080u);
#pragma unroll
            for (int r = 0; r < 3; ++r) v[r][t] = __builtin_amdgcn_perm(I[3], I[r], 0x03020105u + r);
            const bool fine = (L & 0x7F7F7F7Fu) != 0;
            const u64 fm = __builtin_amdgcn_ballot_w64(fine);
            if (__builtin_expect(fm != 0, 0)) emit(fm, fine, 0x1000 + m * 1024 + ln * 16 + t, L);
          }
#pragma unroll
          for (int r = 0; r < 3; ++r) st16_sys(slot + (m >> 1) * g2 + ((m & 1) * 3 + r) * 1024, ln * 16, v[r]);
        }
        u64 rem = ((u64)lowm[1] << 32) | lowm[0];
        unsigned loww = rem != 0 ? 0x80000000u : 0u, last = 0;
#pragma unroll
        for (int k = 0; k < 5; ++k) {
          if (rem != 0) last = __builtin_ctzll(rem);
          loww |= last << (6 * k);
          rem &= rem - 1;
        }
        st_sys(meta + ln * 4, loww);
        if (mixed) st_sys(meta + 8192 + ln * 4, loww);
        for (u64 rm; (rm = __builtin_amdgcn_ballot_w64(rem != 0)) != 0 && nexc <= (unsigned)EXC_CAP; rem &= rem - 1) {
          const unsigned b = __builtin_ctzll(rem | (1ull << 63));
          emit(rm, rem != 0, 0x1000 + ((b >> 5) * 2 + ((b >> 2) & 1)) * 1024 + ln * 16 + (b & 3), 0x80u << (b & 24));
        }
        while (gmask != 0 && nexc <= (unsigned)EXC_CAP) {
          const int g = 15 - __builtin_ctz(gmask);
          gmask &= gmask - 1;
          f32x4 v = acc[0][0];
#pragma unroll
          for (int gg = 1; gg < 16; ++gg) v = g == gg ? acc[gg >> 2][gg & 3] : v;
#pragma unroll
          for (int r = 0; r < 4; ++r) {
            const float a = v[r];
            emit(__builtin_amdgcn_ballot_w64(too_big(a)), too_big(a), (g >> 2) * 1024 + ln * 16 + r * 4 + (g & 3), as_u(a));
          }
        }
        if (nexc > (unsigned)EXC_CAP) {
          nexc = 15;
          WAIT_VM(0);
#pragma unroll
          for (int m = 0; m < 4; ++m)
#pragma unroll
            for (int r = 0; r < 4; ++r) {
              f32x4 v = {acc[m][0][r], acc[m][1][r], acc[m][2][r], acc[m][3][r]};
              st16_sys(slot + (m * 4 + r) * 1024, ln * 16, __builtin_bit_cast(u32x4, v));
            }
        }
        WAIT_VM(0);
        const size_t fo = OFF_FLAG + (size_t)((par * 8 + rank) * NFL + 48 + n * 3 + (tl < 6 ? 0 : (tl < 9 ? 1 : 2))) * 128 + wave * 16;
        if (lane == 0) {
          __hip_atomic_store((GAS u64*)(p.ws[dA & 7] + fo), ((u64)call << 32) | nexc, __ATOMIC_RELAXED, __HIP_MEMORY_SCOPE_SYSTEM);
          if (mixed) __hip_atomic_store((GAS u64*)(p.ws[dB & 7] + fo), ((u64)call << 32) | nexc, __ATOMIC_RELAXED, __HIP_MEMORY_SCOPE_SYSTEM);
        }
        __builtin_amdgcn_s_setprio(0);
      }
      STAMP(4 + 3 * pass);
    }
  } else {
    const int s = peer;
    const int ring = wave >= 4 ? (wave & 1) : (wave >> 1);
    LAS char* const rbase = (LAS char*)smem + ring * (RING * LSLOT);
    volatile LAS unsigned* const full = sctl + 16 + ring * 16;
    volatile LAS unsigned* const info = full + 4;
    LAS unsigned* const done = (LAS unsigned*)(full + 8);
    const unsigned nitem = ((rank ^ s) & 4) == 0 ? NITEM : NITEM - 16;
    if (wave == 4 || wave == 5) {
      const char* sws = p.ws[s];
      const void* tick = myws + OFF_QCNT + (size_t)(par * 8 + s) * 128;
      const char* const fbase = myws + OFF_FLAG + (size_t)(par * 8 + s) * NFL * 128;
      auto describe = [&](unsigned t, unsigned& off, unsigned& d1) {
        const unsigned tt = t < nitem ? t : 0;
        off = lds_peek(table + tt * 3) + (tt < (unsigned)NIN ? par * 1280 * (unsigned)PAIR : 0u);
        d1 = lds_peek(table + tt * 3 + 1);
      };
      unsigned item = j * 2 + ring, inext = item + 16;
      unsigned off, d1;
      describe(item, off, d1);
      u64 fpre = 0;
      bool sfree = true;
      int pend = -1;
      unsigned pinfo = 0;
      unsigned seq = 0;
      const unsigned l16 = lane * 16;
#define PUBLISH() do { if (pend >= 0) { if (lane == 0) { info[pend] = pinfo; asm volatile("s_waitcnt lgkmcnt(0)" ::: "memory"); full[pend] = seq; } pend = -1; } } while (0)
      for (; seq <= (unsigned)NITEM; ) {
        const int sl = seq % RING;
        const bool stop = item >= nitem;
        bool bad = false;
        if (!sfree) sfree = (int)(lds_peek(done + sl) - 3 * (seq / RING)) >= 0;
        if (stop || (int)((unsigned)(fpre >> 32) - call) < 0 || !sfree) {
          WAIT_VM(0);
          PUBLISH();
          u64 t0 = __builtin_amdgcn_s_memrealtime();
          if (!stop) {
            for (;;) {
              fpre = uni64(ld_sys64(fbase + (d1 & 0xFFFFu)));
              if ((int)((unsigned)(fpre >> 32) - call) >= 0) break;
              if (ld_agent(tmo) == call || __builtin_amdgcn_s_memrealtime() - t0 > DEADLINE) { st_agent(tmo, call); bad = true; break; }
              __builtin_amdgcn_s_sleep(2);
            }
          }
          while ((int)(lds_peek(done + sl) - 3 * (seq / RING)) < 0) {
            if (__builtin_amdgcn_s_memrealtime() - t0 > 2 * DEADLINE) { st_agent(tmo, call); bad = true; break; }
            __builtin_amdgcn_s_sleep(1);
          }
          bad = __builtin_amdgcn_readfirstlane((int)bad) != 0;
          if (stop) { pend = sl; pinfo = 1u << 15; ++seq; PUBLISH(); break; }
          if (seq == 0) STAMP2(2);
        }
        if (item < (unsigned)NIN && inext >= (unsigned)NIN) STAMP2(5);
        const bool isin = item < (unsigned)NIN;
        const unsigned bits = ((unsigned)fpre >> ((d1 >> 16) & 31u)) & (isin ? 3u : 15u);
        const int hs = (d1 >> 25) & 1;
        int len = isin ? (bits == 0 ? 7040 : 9216) : (bits == 15u ? 16384 >> hs : (12288 >> hs) + 256 + (int)((bits + 7) >> 3) * 64);
        const int nld = (len + 1023) >> 10;
        if (bad) len = 0;
        const __amdgpu_buffer_rsrc_t rs = __builtin_amdgcn_make_buffer_rsrc((void*)(sws + off), 0, len, 0x00020000);
        LAS char* dstl = rbase + sl * LSLOT;
        unsigned tk; u64 fn;
#define PULL_ITEM(W) do { \
          __builtin_amdgcn_raw_ptr_buffer_load_lds(rs, (LAS void*)dstl, 16, l16, 0, 0, 17); \
          describe(inext, off, d1); \
          ticket_issue(tk, tick); \
          asm volatile("global_load_dwordx2 %0, %1, %2 sc0 sc1" : "=&v"(fn) : "v"(0u), "s"(fbase + (d1 & 0xFFFFu)) : "memory"); \
          WAIT_VM((W) + 1); \
          _Pragma("unroll") for (int i = 1; i < (W); ++i) { \
            __builtin_amdgcn_raw_ptr_buffer_load_lds(rs, (LAS void*)(dstl + i * 1024), 16, l16 + i * 1024, 0, 0, 17); \
            WAIT_VM((W) + 1); \
            if (i == (W) - 2) {                              \
              PUBLISH(); \
              sfree = (int)(lds_peek(done + (seq + 1) % RING) - 3 * ((seq + 1) / RING)) >= 0; \
            } \
          } \
          for (int i = (W); i < nld; ++i) { \
            __builtin_amdgcn_raw_ptr_buffer_load_lds(rs, (LAS void*)(dstl + i * 1024), 16, l16 + i * 1024, 0, 0, 17); \
            if (item + TAIL_ITEMS < nitem || i == nld - 1) WAIT_VM((W) - 1); \
          } \
          asm volatile("" : "+v"(tk), "+v"(fn) :: "memory");           \
          inext2 = 32 + (unsigned)__builtin_amdgcn_readfirstlane(tk); \
          fpre = uni64(fn); } while (0)
        unsigned inext2;
        if (isin) PULL_ITEM(PULL_WIN_IN); else PULL_ITEM(PULL_WIN);
        ++seq;
        pend = sl;
        pinfo = item | (bits << 10) | ((unsigned)bad << 14);
        item = inext;
        inext = inext2;
      }
      STAMP2(3);
    } else {
      const int role = wave >= 6 ? 2 : (wave & 1);
      LAS char* const scr = (LAS char*)smem + 2 * RING * LSLOT + (ring * 3 + role) * SCR;
      const unsigned vo = ((lane >> 4) * 4 * NFULL + (lane & 15) * 4) * 4;
      int pp = NOPIECE, pneed = 0;
      for (unsigned seq = 0; seq <= (unsigned)NITEM; ++seq) {
        const int sl = seq % RING;
        u64 t0 = __builtin_amdgcn_s_memrealtime();
        bool lost = false;
        while (lds_peek((LAS unsigned*)(full + sl)) != seq + 1) {
          if (pp != NOPIECE) { WAIT_VM(0); piece_count(myws, pp, pneed, call, lane); pp = NOPIECE; continue; }
          if (__builtin_amdgcn_s_memrealtime() - t0 > 4 * DEADLINE) { lost = true; break; }
          __builtin_amdgcn_s_sleep(1);
        }
        if (lost) { st_agent(tmo, call); if (lane == 0) p.out[rank][0] = __builtin_nanf(""); break; }
        const unsigned inf = lds_peek((LAS unsigned*)(info + sl));
        if (inf >> 15) break;
        const unsigned item = inf & 1023u, bits = (inf >> 10) & 15u;
        const bool bad = (inf >> 14) & 1u;
        const unsigned d2 = lds_peek(table + item * 3 + 2);
        if (item < (unsigned)NIN) {
          const int c = (role + (int)(seq % 3)) % 3;
          if (c < 2) {
            unsigned w[16], t;
            chunk_unpack(rbase + sl * LSLOT, c, ((bits >> c) & 1u) || bad, scr, lane, w, [&]() {
              WAIT_VM(0);
              ticket_issue(t, myws + (pp != NOPIECE ? OFF_READY + (size_t)pp * 128 : OFF_CALL + (size_t)blockIdx.x * 128 + wave * 16 + 8));
            });
            asm volatile("s_waitcnt lgkmcnt(0)" ::: "memory");
            if (lane == 0) __hip_atomic_fetch_add(done + sl, 1u, __ATOMIC_RELAXED, __HIP_MEMORY_SCOPE_WORKGROUP);
            const bool isw = (d2 >> 17) & 1u;
            chunk_store(myws, ((d2 & 0x1FFFFu) << 10) + (isw ? c * 8192u : c * 32768u + par * (16u << 20)),
                        isw ? 2048 : 1024, isw ? 4096 : 16384, lane, w);
            asm volatile("s_waitcnt vmcnt(4)" : "+v"(t) :: "memory");
            if (pp != NOPIECE) piece_flag(myws, pp, pneed, __builtin_amdgcn_readfirstlane(t), call, lane);
            pp = (d2 >> 18) & 63u;
            pneed = (d2 >> 24) * 16;
          } else if (lane == 0) __hip_atomic_fetch_add(done + sl, 1u, __ATOMIC_RELAXED, __HIP_MEMORY_SCOPE_WORKGROUP);
          continue;
        }
        if (pp != NOPIECE) { WAIT_VM(0); piece_count(myws, pp, pneed, call, lane); pp = NOPIECE; }
        const int ng = 4 >> ((lds_peek(table + item * 3 + 1) >> 25) & 1u), mlo = d2 >> 24;
        const unsigned nexc = bits;
        const float sc = bad ? __builtin_nanf("") : scale;
        float* tbase = p.out[rank] + (d2 & 0x7FFFFFu);
        const LAS char* sb = rbase + sl * LSLOT;
        const unsigned loww = *(const LAS unsigned*)(sb + ng * 3072 + lane * 4);
        const u64 ent = *(const LAS u64*)(sb + ng * 3072 + 256 + (lane & 15) * 8);
        u64 lowm = 0;
#pragma unroll
        for (int k = 0; k < 5; ++k) lowm |= 1ull << ((loww >> (6 * k)) & 63u);
        if ((int)loww >= 0) lowm = 0;
        const unsigned ekey = (unsigned)lane < nexc ? (unsigned)(ent >> 42) : 99u;
        const int k0 = (role + 3 - (int)(seq % 3)) % 3;
        if (k0 >= ng && lane == 0) __hip_atomic_fetch_add(done + sl, 1u, __ATOMIC_RELAXED, __HIP_MEMORY_SCOPE_WORKGROUP);
        for (int k = k0; k < ng; k += 3) {
          const int m = mlo + k;
          u32x4 cw[4];
          const LAS char* q = sb + lane * 16 + k * (nexc == 15u ? 4096 : 3072);
#pragma unroll
          for (int r = 0; r < 4; ++r) cw[r] = *(const LAS u32x4*)(q + r * 1024);
          asm volatile("s_waitcnt lgkmcnt(0)" : "+v"(cw[0]), "+v"(cw[1]), "+v"(cw[2]), "+v"(cw[3]) :: "memory");
          if (k + 3 >= ng && lane == 0) __hip_atomic_fetch_add(done + sl, 1u, __ATOMIC_RELAXED, __HIP_MEMORY_SCOPE_WORKGROUP);
          if (nexc != 15u) {
            const unsigned lb = (unsigned)(lowm >> (32 * (m >> 1))) >> (4 * (m & 1));
            unsigned L[4];
#pragma unroll
            for (int t = 0; t < 4; ++t) L[t] = (lb << (7 - t)) & 0x80808080u;
            for (unsigned em = (unsigned)__builtin_amdgcn_ballot_w64(ekey == 4u + m); em != 0; em &= em - 1) {
              const int e = __builtin_ctz(em);
              const unsigned val = __builtin_amdgcn_readlane((unsigned)ent, e);
              const unsigned key = (__builtin_amdgcn_readlane((unsigned)(ent >> 32), e) & 1023u) - (unsigned)lane * 16;
#pragma unroll
              for (int t = 0; t < 4; ++t) L[t] |= key == (unsigned)t ? val : 0u;
            }
#pragma unroll
            for (int t = 0; t < 4; ++t) {
              cw[3][t] = __builtin_amdgcn_perm(cw[1][t], cw[0][t], 0x0C04000Cu) | __builtin_amdgcn_perm(cw[2][t], L[t], 0x040C0C03u);
#pragma unroll
              for (int r = 0; r < 3; ++r) cw[r][t] = __builtin_amdgcn_perm(cw[r][t], L[t], 0x07060500u + r);
#pragma unroll
              for (int r = 0; r < 4; ++r) { const unsigned x = cw[r][t]; cw[r][t] = as_u((float)(int)x); }
            }
            for (unsigned em = (unsigned)__builtin_amdgcn_ballot_w64(ekey == (unsigned)m); em != 0; em &= em - 1) {
              const int e = __builtin_ctz(em);
              const unsigned val = __builtin_amdgcn_readlane((unsigned)ent, e);
              const unsigned key = (__builtin_amdgcn_readlane((unsigned)(ent >> 32), e) & 1023u) - (unsigned)lane * 16;
#pragma unroll
              for (int k = 0; k < 16; ++k) cw[k >> 2][k & 3] = key == (unsigned)k ? val : cw[k >> 2][k & 3];
            }
          }
#pragma unroll
          for (int r = 0; r < 4; ++r) {
            f32x4 y;
#pragma unroll
            for (int t = 0; t < 4; ++t) y[t] = silu_dq(as_f(cw[r][t]), sc);
            st16_out(tbase + (size_t)(m * 16 + r) * NFULL, vo, __builtin_bit_cast(u32x4, y));
          }
        }
      }
      if (pp != NOPIECE) { WAIT_VM(0); piece_count(myws, pp, pneed, call, lane); }
      STAMP(4);
    }
  }
}

extern "C" void kernel_launch(int rank, int nranks, ncclComm_t comm,
                              hipStream_t stream,
                              void* const* d_in, const int* in_sizes, int n_in,
                              void* d_out, int out_size,
                              const int* in_dtypes, int out_dtype,
                              void* const* d_out_all,
                              void* const* d_ws_all, size_t ws_bytes) {
  (void)comm; (void)in_dtypes; (void)out_dtype; (void)d_out;
  if (nranks != NR || n_in != 4 || ws_bytes < WS_NEED || (unsigned)rank >= (unsigned)NR ||
      in_sizes[0] != 4096 * KSH || in_sizes[1] != KSH * NFULL || out_size != 512 * NFULL) {
    fprintf(stderr, "kernel_launch: unexpected problem shape\n");
    return;
  }
  Params p;
  p.x = (const uint8_t*)d_in[0];
  p.w = (const uint8_t*)d_in[1];
  p.sx = (const float*)d_in[2];
  p.sw = (const float*)d_in[3];
  for (int i = 0; i < NR; ++i) {
    p.out[i] = (float*)d_out_all[i];
    p.ws[i] = (char*)d_ws_all[i];
  }
  p.rank = rank;
  p.pad = 0;
  hipLaunchKernelGGL(fused_gemm_rs, dim3(256), dim3(512), 0, stream, p);
  HIP_CHECK(hipGetLastError());
}
```

```cpp
#include <hip/hip_runtime.h>
#include <rccl/rccl.h>
#include <cstdio>
#include <cstdint>

#define NPUSH 4
#define PUSH_WIN 2
#define TAIL_ITEMS 16
#define PULL_WIN 4
#define PULL_WIN_IN 3
#define PART_STAGGER 4
#define POLL_SLEEP 8
#define TIMING 0

#define HIP_CHECK(expr) do { hipError_t _e = (expr); if (_e != hipSuccess) { \
    fprintf(stderr, "HIP error %s at %s:%d\n", hipGetErrorString(_e), __FILE__, __LINE__); \
    return; } } while (0)

typedef unsigned u32x4 __attribute__((ext_vector_type(4)));
typedef int i32x4 __attribute__((ext_vector_type(4)));
typedef int i32x8 __attribute__((ext_vector_type(8)));
typedef float f32x4 __attribute__((ext_vector_type(4)));
typedef unsigned long long u64;

#define GAS __attribute__((address_space(1)))
#define LAS __attribute__((address_space(3)))

constexpr int NR = 8;
constexpr int NFULL = 16384;
constexpr int KSH = 512;
constexpr u64 DEADLINE = 40000000ull;

constexpr int NFL = 96;
constexpr size_t OFF_CALL  = 0;
constexpr size_t OFF_PUSHF = 32768;
constexpr size_t OFF_QCNT  = OFF_PUSHF + 1024;
constexpr size_t OFF_TMO   = OFF_QCNT + 2048;
constexpr size_t OFF_PAGG  = OFF_TMO + 128;
constexpr size_t OFF_READY = 65536;
constexpr size_t OFF_RFLAG = OFF_READY + 16384;
constexpr size_t OFF_FLAG  = OFF_RFLAG + 16384;
constexpr size_t OFF_EAGG  = OFF_FLAG + 16 * NFL * 128;
constexpr size_t OFF_TIME  = OFF_EAGG + 96 * 256;
constexpr size_t OFF_A     = 1u << 20;
constexpr size_t OFF_B     = OFF_A + (32u << 20);
constexpr size_t OFF_B2    = OFF_B + (16u << 20);
constexpr size_t OFF_S     = OFF_B2 + (8u << 20);
constexpr size_t PAIR      = 9216;
constexpr size_t OFF_O     = OFF_S + 2560 * PAIR;
constexpr size_t SLOT      = 16384;
constexpr size_t WS_NEED   = OFF_O + (size_t)16 * 12 * 8 * SLOT;
static_assert(OFF_PAGG + 4096 <= OFF_READY && OFF_TIME + 32768 <= OFF_A, "layout");
constexpr int EXC_CAP = 14;
constexpr int NOPIECE = 63;

constexpr int STAGE = 49152;
constexpr int LDS_BYTES = 3 * STAGE + 256;

struct Params {
  const uint8_t* x;
  const uint8_t* w;
  char* myws;
  int rank;
  int pad;
  char* link[NR];
  const float* sx;
  const float* sw;
  float* out;
  char* ws[NR];
};

__device__ __forceinline__ float as_f(unsigned u) { return __builtin_bit_cast(float, u); }
__device__ __forceinline__ unsigned as_u(float f) { return __builtin_bit_cast(unsigned, f); }
__device__ __forceinline__ void st16_sys(const void* base, unsigned off, u32x4 v) {
  asm volatile("global_store_dwordx4 %0, %1, %2 sc0 sc1\n\ts_nop 1" :: "v"(off), "v"(v), "s"(base) : "memory");
}
#define OUT_ST "sc1"
__device__ __forceinline__ void st16_out(const void* base, unsigned off, u32x4 v) {
  asm volatile("global_store_dwordx4 %0, %1, %2 " OUT_ST "\n\ts_nop 1" :: "v"(off), "v"(v), "s"(base) : "memory");
}
__device__ __forceinline__ void ticket_issue(unsigned& t, const void* ctr) {
  u64 sv;
  asm volatile("s_mov_b64 %1, exec\n\ts_mov_b64 exec, 1\n\tglobal_atomic_add %0, %2, %3, %4 sc0\n\ts_mov_b64 exec, %1"
               : "=&v"(t), "=&s"(sv) : "v"(0u), "v"(1u), "s"(ctr) : "memory");
}
#define WAIT_VM(n) asm volatile("s_waitcnt vmcnt(%0)" :: "i"(n) : "memory")

__device__ __forceinline__ unsigned ld_sys(const void* p) {
  return __hip_atomic_load((const GAS unsigned*)p, __ATOMIC_RELAXED, __HIP_MEMORY_SCOPE_SYSTEM);
}
__device__ __forceinline__ u64 ld_sys64(const void* p) {
  return __hip_atomic_load((const GAS u64*)p, __ATOMIC_RELAXED, __HIP_MEMORY_SCOPE_SYSTEM);
}
__device__ __forceinline__ unsigned ld_agent(const void* p) {
  return __hip_atomic_load((const GAS unsigned*)p, __ATOMIC_RELAXED, __HIP_MEMORY_SCOPE_AGENT);
}
__device__ __forceinline__ void st_agent(void* p, unsigned v) {
  __hip_atomic_store((GAS unsigned*)p, v, __ATOMIC_RELAXED, __HIP_MEMORY_SCOPE_AGENT);
}
__device__ __forceinline__ void st_sys(void* p, unsigned v) {
  __hip_atomic_store((GAS unsigned*)p, v, __ATOMIC_RELAXED, __HIP_MEMORY_SCOPE_SYSTEM);
}

__device__ __forceinline__ void wait_ge(const void* p, unsigned target, void* tmo, unsigned call) {
  u64 t0 = __builtin_amdgcn_s_memrealtime();
  for (unsigned n = 1; (int)(ld_sys(p) - target) < 0; ++n) {
    if ((n & 15u) == 0 && (ld_agent(tmo) == call || __builtin_amdgcn_s_memrealtime() - t0 > DEADLINE)) {
      st_agent(tmo, call);
      break;
    }
    __builtin_amdgcn_s_sleep(POLL_SLEEP);
  }
}

__device__ __forceinline__ float silu_dq(float acc23, float s) {
  float y = (acc23 * 1.1920928955078125e-07f) * s;
  float c = fminf(fmaxf(y, -60.0f), 60.0f);
  return y * __builtin_amdgcn_rcpf(1.0f + __builtin_amdgcn_exp2f(c * -1.44269504088896f));
}
__device__ __forceinline__ unsigned to_i32(float a) {
  int r;
  asm("v_cvt_i32_f32 %0, %1" : "=v"(r) : "v"(a));
  return (unsigned)r;
}
__device__ __forceinline__ bool too_big(float a) { return !(__builtin_fabsf(a) < 2147483648.0f); }

__device__ __forceinline__ int same_peer(int r, int i) { return (r & 4) | ((r + 1 + i) & 3); }
__device__ __forceinline__ int partner(int r, int q) { return r < 4 ? 4 | ((r + q) & 3) : (r - q) & 3; }
__device__ __forceinline__ int wblock(int r, int n) {
  const int t = partner(r, n & 3);
  return (r < 4 ? r * 4 + (t & 3) : t * 4 + (r & 3)) * 4 + (n >> 2);
}
__device__ __forceinline__ int x1_block(int r, int k) { return k < 3 ? same_peer(r, k) * 4 : partner(r, (k - 2) & 3) * 4 + 2; }
__device__ __forceinline__ int x1_index(int r, int s) {
  if (((r ^ s) & 4) == 0) return (s - r - 1) & 3;
  const int q = (r < 4 ? s - r : r - s) & 3;
  return q == 0 ? 6 : q + 2;
}
__device__ __forceinline__ int x1_rot(int r, int s, int i) { return (x1_index(r, s) + 1 + i) % 7; }
__device__ __forceinline__ int tile_sub(int t) { return t < 3 ? 0 : (t < 6 ? 2 : (t < 9 ? 3 : (t < 12 ? 1 : t - 12))); }
__device__ __forceinline__ void x_piece(int r, int mb, int mt, int& piece, int& need) {
  const bool tail = (mb & 3) == 1 && mt >= 6;
  piece = tail ? 32 + (mb >> 2) : mb;
  need = (tail ? 4 : ((mb & 3) == 1 ? 12 : 16)) * ((mb >> 2) == r ? 8 : ((mb & 1) ? 7 : 7 - NPUSH));
}

#define STAMP(i) do {} while (0)
#define STAMP2(i) do {} while (0)

__device__ __forceinline__ void x_load(const Params& p, int mb, int e, int lane, unsigned (&w)[16]) {
  const __amdgpu_buffer_rsrc_t rs = __builtin_amdgcn_make_buffer_rsrc((void*)p.x, 0, 4096 * KSH, 0x00020000);
  const unsigned vo = (lane & 15) * KSH + (lane >> 4) * 16;
  const unsigned so = (mb * 128 + (e >> 1) * 16) * KSH + (e & 1) * 256;
#pragma unroll
  for (int u = 0; u < 4; ++u) {
    u32x4 v = __builtin_amdgcn_raw_buffer_load_b128(rs, vo, so + u * 64, 0);
    w[4 * u] = v[0]; w[4 * u + 1] = v[1]; w[4 * u + 2] = v[2]; w[4 * u + 3] = v[3];
  }
}
__device__ __forceinline__ void w_load(const Params& p, int col0, int e, int lane, unsigned (&r)[16]) {
  const __amdgpu_buffer_rsrc_t rs = __builtin_amdgcn_make_buffer_rsrc((void*)p.w, 0, KSH * NFULL, 0x00020000);
  const unsigned vo = (lane >> 4) * 16 * NFULL + (lane & 15) * 4;
  const unsigned so = (e >> 2) * 64 * NFULL + col0 + (e & 3) * 64;
#pragma unroll
  for (int i = 0; i < 16; ++i) r[i] = __builtin_amdgcn_raw_buffer_load_b32(rs, vo, so + i * NFULL, 0);
}
__device__ __forceinline__ void w_transpose(unsigned (&r)[16]) {
  unsigned w[16];
#pragma unroll
  for (int u = 0; u < 4; ++u) {
    const unsigned a0 = __builtin_amdgcn_perm(r[4 * u + 1], r[4 * u], 0x05010400u), a1 = __builtin_amdgcn_perm(r[4 * u + 1], r[4 * u], 0x07030602u);
    const unsigned b0 = __builtin_amdgcn_perm(r[4 * u + 3], r[4 * u + 2], 0x05010400u), b1 = __builtin_amdgcn_perm(r[4 * u + 3], r[4 * u + 2], 0x07030602u);
    w[u] = __builtin_amdgcn_perm(b0, a0, 0x05040100u);
    w[4 + u] = __builtin_amdgcn_perm(b0, a0, 0x07060302u);
    w[8 + u] = __builtin_amdgcn_perm(b1, a1, 0x05040100u);
    w[12 + u] = __builtin_amdgcn_perm(b1, a1, 0x07060302u);
  }
#pragma unroll
  for (int i = 0; i < 16; ++i) r[i] = w[i];
}
__device__ __forceinline__ void chunk_place(bool isw, int blk, int e, int src, unsigned par, unsigned& dst, unsigned& s01, unsigned& s23) {
  if (isw) {
    dst = (unsigned)OFF_B + (blk * 32 + src * 4 + (e >> 3)) * 32768u + (e & 3) * 8192 + ((e >> 2) & 1) * 1024;
    s01 = 2048; s23 = 4096;
  } else {
    dst = (unsigned)OFF_A + ((par * 32 + blk) * 32 + src * 4 + (e & 1) * 2) * 16384u + (e >> 1) * 2048;
    s01 = 1024; s23 = 16384;
  }
}
__device__ __forceinline__ void chunk_store(const char* base, unsigned dst, unsigned s01, unsigned s23, int lane, const unsigned (&w)[16]) {
  st16_sys(base + dst, lane * 16, u32x4{w[0], w[1], w[2], w[3]});
  st16_sys(base + dst + s01, lane * 16, u32x4{w[4], w[5], w[6], w[7]});
  st16_sys(base + dst + s23, lane * 16, u32x4{w[8], w[9], w[10], w[11]});
  st16_sys(base + dst + s23 + s01, lane * 16, u32x4{w[12], w[13], w[14], w[15]});
}

__device__ __forceinline__ unsigned wave_scan_incl(unsigned v) {
  v += __builtin_amdgcn_update_dpp(0u, v, 0x111, 0xf, 0xf, true);
  v += __builtin_amdgcn_update_dpp(0u, v, 0x112, 0xf, 0xf, true);
  v += __builtin_amdgcn_update_dpp(0u, v, 0x114, 0xf, 0xf, true);
  v += __builtin_amdgcn_update_dpp(0u, v, 0x118, 0xf, 0xf, true);
  v += __builtin_amdgcn_update_dpp(0u, v, 0x142, 0xa, 0xf, false);
  v += __builtin_amdgcn_update_dpp(0u, v, 0x143, 0xc, 0xf, false);
  return v;
}

constexpr int ESC_CAP = 1120;
constexpr int SCR = 1344;
__device__ __forceinline__ bool chunk_pack(const unsigned (&w)[16], const char* pair, int h, LAS char* scr, int lane) {
  const char* out = pair + h * 3072;
  unsigned m[16], P[4] = {0, 0, 0, 0}, big = 0, cnt = 0;
#pragma unroll
  for (int i = 0; i < 16; ++i) {
    const unsigned e = (w[i] >> 3) & 0x0F0F0F0Fu;
    const unsigned in47 = (e >> 2) & ~(e >> 3) & 0x01010101u;
    const unsigned c = e & (in47 | (in47 << 1));
    m[i] = ~(c | (c >> 1)) & 0x01010101u;
    P[i >> 2] |= c << (2 * (i & 3));
    big |= e + 0x05050505u;
    cnt += __builtin_popcount(m[i]);
  }
  const unsigned incl = wave_scan_incl(cnt);
  const unsigned total = __builtin_amdgcn_readlane(incl, 63);
  if (total > (unsigned)ESC_CAP || __builtin_amdgcn_ballot_w64((big & 0x10101010u) != 0) != 0) {
#pragma unroll
    for (int b = 0; b < 4; ++b)
      st16_sys(b < 3 ? out + b * 1024 : pair + 7168 + h * 1024, lane * 16, u32x4{w[4 * b], w[4 * b + 1], w[4 * b + 2], w[4 * b + 3]});
    return true;
  }
  unsigned pos = incl - cnt;
  const unsigned dummy = 1152 + lane;
#pragma unroll
  for (int i = 0; i < 16; ++i) {
#pragma unroll
    for (int k = 0; k < 4; ++k) {
      const unsigned mk = (m[i] >> (8 * k)) & 1u;
      scr[mk ? pos : dummy] = (char)(w[i] >> (8 * k + 3));
      pos += mk;
    }
  }
  u32x4 n0, n1;
#pragma unroll
  for (int i = 0; i < 4; ++i) {
    n0[i] = (w[2 * i] & 0x87878787u) | ((w[2 * i + 1] << 3) & 0x38383838u) | ((w[2 * i + 1] >> 1) & 0x40404040u);
    n1[i] = (w[8 + 2 * i] & 0x87878787u) | ((w[9 + 2 * i] << 3) & 0x38383838u) | ((w[9 + 2 * i] >> 1) & 0x40404040u);
  }
  st16_sys(out, lane * 16, n0);
  st16_sys(out + 1024, lane * 16, n1);
  st16_sys(out + 2048, lane * 16, u32x4{P[0], P[1], P[2], P[3]});
  asm volatile("s_waitcnt lgkmcnt(0)" ::: "memory");
  if (lane < 56) {
    unsigned b[5];
#pragma unroll
    for (int i = 0; i < 5; ++i) {
      const unsigned e = *(const LAS unsigned*)(scr + lane * 20 + i * 4) & 0x0F0F0F0Fu;
      b[i] = (e - ((e >> 3) & 0x01010101u) * 3u) & 0x07070707u;
    }
    const u64 o = (u64)(b[0] | (b[1] << 3) | ((b[4] & 0x03030303u) << 6)) |
                  ((u64)(b[2] | (b[3] << 3) | ((b[4] & 0x04040404u) << 4)) << 32);
    __hip_atomic_store((GAS u64*)(pair + 6144 + h * 448 + lane * 8), o, __ATOMIC_RELAXED, __HIP_MEMORY_SCOPE_SYSTEM);
  }
  asm volatile("s_waitcnt lgkmcnt(0)" ::: "memory");
  return false;
}

template <class Mid>
__device__ __forceinline__ void chunk_unpack(const LAS char* pair, int h, bool raw, LAS char* scr, int lane, unsigned (&w)[16], Mid&& mid) {
  const LAS char* in = pair + h * 3072;
  const u32x4 P = *(const LAS u32x4*)(in + 2048 + lane * 16);
  const u64 o = *(const LAS u64*)(pair + 6144 + h * 448 + lane * 8);
  const u32x4 n0 = *(const LAS u32x4*)(in + lane * 16), n1 = *(const LAS u32x4*)(in + 1024 + lane * 16);
  if (raw) {
    const u32x4 r3 = *(const LAS u32x4*)(pair + 7168 + h * 1024 + lane * 16);
#pragma unroll
    for (int i = 0; i < 4; ++i) { w[i] = n0[i]; w[4 + i] = n1[i]; w[8 + i] = P[i]; w[12 + i] = r3[i]; }
    mid();
    return;
  }
  {
    const unsigned o0 = (unsigned)o, o1 = (unsigned)(o >> 32);
    unsigned b[5] = {o0 & 0x07070707u, (o0 >> 3) & 0x07070707u, o1 & 0x07070707u, (o1 >> 3) & 0x07070707u,
                     ((o0 >> 6) & 0x03030303u) | ((o1 >> 4) & 0x04040404u)};
#pragma unroll
    for (int i = 0; i < 5; ++i) {
      const unsigned hi = ((b[i] + 0x03030303u) >> 3) & 0x01010101u;
      *(LAS unsigned*)(scr + lane * 20 + i * 4) = ((b[i] + hi + (hi << 1)) << 3) ^ 0x20202020u;
    }
  }
  unsigned M[4], cnt = 0;
#pragma unroll
  for (int k = 0; k < 4; ++k) { M[k] = ~(P[k] | (P[k] >> 1)) & 0x55555555u; cnt += __builtin_popcount(M[k]); }
  unsigned pos = ((wave_scan_incl(cnt) - cnt) & 0x7FFu) + (unsigned)(size_t)scr;
  asm volatile("s_waitcnt lgkmcnt(0)" ::: "memory");
  unsigned lo[16], hi[16], ps[16];
#pragma unroll
  for (int i = 0; i < 16; ++i) {
    const LAS unsigned* q = (const LAS unsigned*)(size_t)(pos & ~3u);
    lo[i] = q[0]; hi[i] = q[1]; ps[i] = pos;
    pos += __builtin_popcount(M[i >> 2] & (0x01010101u << (2 * (i & 3))));
  }
  __builtin_amdgcn_sched_barrier(0);
  mid();
  __builtin_amdgcn_sched_barrier(0);
#pragma unroll
  for (int i = 0; i < 16; ++i) {
    const int sh = 2 * (i & 3);
    const unsigned m = (M[i >> 2] >> sh) & 0x01010101u;
    unsigned t;
    asm("v_mul_u32_u24 %0, %1, %2" : "=v"(t) : "s"(0x010101u), "v"(m));
    const unsigned sel = (t << 8) | ((m << 2) ^ 0x04040404u);
    const unsigned esc = __builtin_amdgcn_perm(0u, __builtin_amdgcn_alignbyte(hi[i], lo[i], ps[i]), sel);
    const unsigned ex = ((sh <= 3 ? P[i >> 2] << (3 - sh) : P[i >> 2] >> (sh - 3)) & 0x18181818u) | 0x20202020u;
    const unsigned n = i < 8 ? n0[(i >> 1) & 3] : n1[(i >> 1) & 3];
    const unsigned sm = (i & 1) ? (((n >> 3) & 0x07070707u) | ((n << 1) & 0x80808080u)) : (n & 0x87878787u);
    w[i] = sm | (ex ^ esc);
  }
}

__device__ __forceinline__ void piece_flag(char* myws, int piece, int need, unsigned t, unsigned call, int lane) {
  if (t + 1 == call * (unsigned)need && lane == 0) st_agent(myws + OFF_RFLAG + (size_t)piece * 128, call);
}
__device__ __forceinline__ void piece_count(char* myws, int piece, int need, unsigned call, int lane) {
  unsigned t;
  ticket_issue(t, myws + OFF_READY + (size_t)piece * 128);
  asm volatile("s_waitcnt vmcnt(0)" : "+v"(t) :: "memory");
  piece_flag(myws, piece, need, __builtin_amdgcn_readfirstlane(t), call, lane);
}

constexpr int NIN = (6 - NPUSH) * 8 + 340;
constexpr int NITEM = NIN + 256;
__device__ __forceinline__ void item_describe(int r, int s, int t, unsigned& d0, unsigned& d1, unsigned& d2) {
  const bool same = ((r ^ s) & 4) == 0;
  const int nin = same ? NIN : NIN - 4;
  if (t < nin) {
    constexpr int N1 = (6 - NPUSH) * 8;
    const int ks = x1_index(r, s);
    const int nc = same ? 4 : 3, ns = same ? 2 : 3;
    auto cross = [&](int i) { return partner(r, (i + (!same && i + 3 >= ks ? 2 : 1)) & 3); };
    int blk, h;
    bool isw = false;
    int u = t - N1;
    if (u < 0) { blk = x1_block(r, x1_rot(r, s, NPUSH + (t >> 3))); h = t & 7; }
    else if (u < 64) { isw = true; blk = u >> 4; h = u & 15; }
    else if ((u -= 64) < nc * 8) { blk = cross(u >> 3) * 4 + 3; h = u & 7; }
    else if ((u -= nc * 8) < ns * 6) { const int i = u / 6; blk = same_peer(r, i + (same && i >= ks ? 1 : 0)) * 4 + 1; h = u % 6; }
    else if ((u -= ns * 6) < nc * 2) { blk = cross(u >> 1) * 4 + 1; h = 6 + (u & 1); }
    else if ((u -= nc * 2) < 192) { isw = true; blk = 4 + (u >> 4); h = u & 15; }
    else { u -= 192; blk = r * 4 + (u >> 3); h = u & 7; }
    int piece = 40 + blk, need = 256;
    if (!isw) x_piece(r, blk, h, piece, need);
    unsigned dst, s01, s23;
    chunk_place(isw, blk, 2 * h, s, 0, dst, s01, s23);
    d0 = (unsigned)OFF_S + (isw ? 256 + wblock(r, blk) * 16 + h : blk * 8 + h) * (unsigned)PAIR;
    d1 = (isw ? 32 + blk : blk) * 128 | (2 * h) << 16;
    d2 = dst >> 10 | (unsigned)isw << 17 | piece << 18 | (need >> 2) << 24;
  } else {
    const int u = t - nin;
    int n, tl, w;
    if (same) {
      const int i = (r - s - 1) & 3;
      if (u < 64) { n = (u >> 3) & 3; tl = u < 32 ? i : 9 + i; w = u & 7; }
      else if (u < 208) { const int x = (u - 64) % 12; n = 4 + (u - 64) / 12; tl = x < 8 ? i : 9 + i; w = x & 7; }
      else { n = 4 + ((u - 208) >> 2); tl = 9 + i; w = 4 + (u & 3); }
    } else {
      const int qr = (r < 4 ? s - r : r - s) & 3;
      int qq, c = 0, kind;
      if (u < 24) { qq = u >> 3; kind = 0; w = u & 7; }
      else if (u < 60) { const int x = (u - 24) % 12; qq = (u - 24) / 12; kind = x < 8 ? 1 : 2; w = x < 8 ? x : x - 4; }
      else {
        const int b = u < 204 ? (u - 60) >> 4 : (u - 204) >> 2;
        c = (1 + b / 3) & 3; qq = b % 3;
        kind = u < 204 ? u & 1 : 2;
        w = u < 204 ? ((u - 60) >> 1) & 7 : 4 + (u & 3);
      }
      const int q = qq + (qq >= qr ? 1 : 0);
      n = c * 4 + q;
      tl = 3 + 3 * kind + ((qr - q - 1) & 3);
    }
    const bool half = tl >= 9 && w >= 4;
    const int hi = half && !same;
    d0 = (unsigned)OFF_O + ((n * 12 + tl) * 8 + w) * (unsigned)SLOT + hi * 8192;
    d1 = ((48 + n * 3 + (tl < 6 ? 0 : (tl < 9 ? 1 : 2))) * 128 + w * 16) | (half ? 2u : 1u) << 24;
    d2 = ((tile_sub(tl) * 128 + (w >> 2) * 64) * NFULL + wblock(s, n) * 256 + (w & 3) * 64) | (hi * 2) << 24;
  }
}

constexpr int RING = 4;
constexpr int LSLOT = 16384;
static_assert(2 * RING * LSLOT + 6 * SCR + NITEM * 12 <= 3 * STAGE, "LDS");

__device__ __forceinline__ unsigned lds_peek(LAS unsigned* p) {
  unsigned v;
  asm volatile("ds_read_b32 %0, %1\n\ts_waitcnt lgkmcnt(0)" : "=&v"(v) : "v"(p) : "memory");
  return __builtin_amdgcn_readfirstlane(v);
}
__device__ __forceinline__ u64 uni64(u64 v) {
  return ((u64)(unsigned)__builtin_amdgcn_readfirstlane((unsigned)(v >> 32)) << 32) | (u64)(unsigned)__builtin_amdgcn_readfirstlane((unsigned)v);
}

__global__ void __launch_bounds__(512, 2) fused_gemm_rs(Params p) {
  __shared__ __attribute__((aligned(16))) char smem[LDS_BYTES];
  const int tid = threadIdx.x;
  const int lane = tid & 63;
  const int wave = __builtin_amdgcn_readfirstlane(tid >> 6);
  const int rank = p.rank;
  const int j = blockIdx.x & 7;
  const int grp = blockIdx.x >> 3;
  char* const myws = p.myws;
  char* const pws = p.link[grp & 7];
  asm volatile("" :: "s"(p.x), "s"(p.w), "s"(myws), "s"(pws), "s"(rank));
  volatile LAS unsigned* sctl = (volatile LAS unsigned*)(LAS char*)(smem + 3 * STAGE);

  const int peer = (rank + 1 + grp) & 7;
  unsigned w0[16], wp[NPUSH / 2][16];
  const int pw = j * 4 + wave;
  const int ew = (wave >> 1) * 400 + (blockIdx.x - 56) * 2 + (wave & 1);
  auto enc_id = [&](int n, bool& isw, int& a, int& b, int& e) {
    isw = (n >= 256 && n < 768) || n >= 1024;
    if (isw) { const int i = (n < 768 ? n - 256 : n - 512) >> 5; a = i & 3; b = i >> 2; e = n & 31; }
    else { a = ((n >> 4) & 7) * 4 + ((n & 128) ? 0 : 2) + (n >= 768 ? 1 : 0); b = 0; e = n & 15; }
  };
  auto enc_load = [&](int n, unsigned (&w)[16]) {
    bool isw; int a, b, e;
    enc_id(n, isw, a, b, e);
    if (isw) w_load(p, wblock(a, b) * 256, e, lane, w); else x_load(p, a, e, lane, w);
  };
  LAS unsigned* const table = (LAS unsigned*)((LAS char*)smem + 2 * RING * LSLOT + 6 * SCR);
  unsigned cticket;
  ticket_issue(cticket, myws + OFF_CALL + (size_t)blockIdx.x * 128 + wave * 16);
  STAMP(0);
  if (grp >= 7) enc_load(ew, w0);
  else if (wave < 4) {
#pragma unroll
    for (int it = 0; it < NPUSH / 2; ++it) x_load(p, x1_block(peer, x1_rot(peer, rank, (pw >> 4) + 2 * it)), pw & 15, lane, wp[it]);
  }
  if (grp < 7) {
    if (tid >= 15 && tid < 48) sctl[tid] = 0;
    asm volatile("s_waitcnt lgkmcnt(0)" ::: "memory");
    __builtin_amdgcn_s_barrier();
    if (wave >= 4) {
      for (int t = tid - 256; t < NITEM; t += 256) {
        unsigned d0, d1, d2;
        item_describe(rank, peer, t, d0, d1, d2);
        table[t * 3] = d0; table[t * 3 + 1] = d1; table[t * 3 + 2] = d2;
      }
      asm volatile("s_waitcnt lgkmcnt(0)" ::: "memory");
      if (lane == 0) __hip_atomic_fetch_add((LAS unsigned*)(sctl + 15), 1u, __ATOMIC_RELAXED, __HIP_MEMORY_SCOPE_WORKGROUP);
    }
  }
  if (grp < 7 && wave < 4) WAIT_VM(NPUSH * 2); else WAIT_VM(0);
  asm volatile("" : "+v"(cticket) :: "memory");
  const unsigned call = (unsigned)__builtin_amdgcn_readfirstlane(cticket) + 1;
  const unsigned par = call & 1;
  void* const tmo = myws + OFF_TMO;

  const bool dbg_skip = false;
  if (dbg_skip) {
  } else if (grp >= 7) {
    LAS char* const scr = (LAS char*)smem + wave * SCR;
    if (wave < 2) __builtin_amdgcn_s_setprio(3); else if (wave < 4) __builtin_amdgcn_s_setprio(2); else if (wave < 6) __builtin_amdgcn_s_setprio(1);
    for (int n = ew; n < 2560; n += 1600) {
      bool isw; int a, b, e;
      enc_id(n, isw, a, b, e);
      if (n != ew) enc_load(n, w0);
      if (isw) w_transpose(w0);
      const int other = isw ? partner(a, b & 3) : 0;
      const bool low = (a & 3) == 0 || ((a & 3) == 1 && e < 12);
      const bool mine = isw ? (a == rank || other == rank) : ((a >> 2) == rank || low == ((((a >> 2) ^ rank) & 4) == 0));
      const int sp = isw ? 32 + wblock(a, b) : a;
      if (isw || (a >> 2) != rank) {
        const int slot = isw ? 512 + (sp - 32) * 32 + e : a * 16 + e;
        const bool raw = chunk_pack(w0, myws + OFF_S + ((size_t)par * 1280 + (slot >> 1)) * PAIR, slot & 1, scr, lane);
        WAIT_VM(0);
        char* const agg = myws + OFF_EAGG + (size_t)sp * 256;
        const u64 mine64 = (1ull << 32) | (raw ? 1u << e : 0u);
        u64 tot = 0;
        if (lane == 0) tot = __hip_atomic_fetch_add((GAS u64*)(agg + par * 128), mine64, __ATOMIC_RELAXED, __HIP_MEMORY_SCOPE_AGENT);
        tot = uni64(tot) + mine64;
        if ((unsigned)(tot >> 32) == (isw ? 32u : 16u)) {
          if (lane < NR && lane != rank && (!isw || lane == a || lane == other))
            __hip_atomic_store((GAS u64*)(p.ws[lane] + OFF_FLAG + (size_t)((par * 8 + rank) * NFL + (isw ? 32 + b : a)) * 128),
                               ((u64)call << 32) | (unsigned)tot, __ATOMIC_RELAXED, __HIP_MEMORY_SCOPE_SYSTEM);
          if (lane == 0) __hip_atomic_store((GAS u64*)(agg + (par ^ 1) * 128), 0ull, __ATOMIC_RELAXED, __HIP_MEMORY_SCOPE_AGENT);
        }
      }
      if (mine || isw) {
        unsigned dst, s01, s23;
        chunk_place(isw, isw ? b : a, e, rank, par, dst, s01, s23);
        if (!mine) dst += (unsigned)(OFF_B2 - OFF_B) + ((sp - 32) * 4 - (b * 32 + rank * 4)) * 32768u;
        chunk_store(myws, dst, s01, s23, lane, w0);
        int piece = mine ? 40 + b : 32 + sp, need = mine ? 256 : 32;
        if (!isw) x_piece(rank, a, e >> 1, piece, need);
        WAIT_VM(0);
        piece_count(myws, piece, need, call, lane);
        if (!isw && (a >> 2) == rank) piece_count(myws, 56 + (a & 3), 16, call, lane);
      }
    }
    __builtin_amdgcn_s_setprio(0);
  } else if (wave < 4) {
    char* const dws = pws;
    const __amdgpu_buffer_rsrc_t drs = __builtin_amdgcn_make_buffer_rsrc(dws, 0, (int)WS_NEED, 0x00020000);
    STAMP(6);
#pragma unroll
    for (int it = 0; it < NPUSH / 2; ++it) {
      unsigned dst, s01, s23;
      chunk_place(false, x1_block(peer, x1_rot(peer, rank, (pw >> 4) + 2 * it)), pw & 15, rank, par, dst, s01, s23);
      const unsigned (&w)[16] = wp[it];
#pragma unroll
      for (int b = 0; b < 4; ++b) {
        __builtin_amdgcn_raw_buffer_store_b128(u32x4{w[4 * b], w[4 * b + 1], w[4 * b + 2], w[4 * b + 3]}, drs, lane * 16,
                                               dst + (b & 1) * s01 + (b >> 1) * s23, 17);
        if (it > 0 || b >= PUSH_WIN - 1) WAIT_VM(PUSH_WIN - 1);
      }
    }
    WAIT_VM(0);
    STAMP(7);
    unsigned ag[NPUSH / 2];
#pragma unroll
    for (int it = 0; it < NPUSH / 2; ++it) ticket_issue(ag[it], myws + OFF_PAGG + (size_t)(peer * 8 + (pw >> 4) + 2 * it) * 64);
#pragma unroll
    for (int it = 0; it < NPUSH / 2; ++it) {
      asm volatile("s_waitcnt vmcnt(0)" : "+v"(ag[it]) :: "memory");
      if ((unsigned)__builtin_amdgcn_readfirstlane(ag[it]) + 1 == call * 16u && lane == 0)
        st_sys(dws + OFF_PUSHF + (size_t)x1_rot(peer, rank, (pw >> 4) + 2 * it) * 128 + rank * 4, call);
    }
  }
  if (blockIdx.x == 64 && tid < NR)
    st_agent(myws + OFF_QCNT + (size_t)((par ^ 1) * 8 + tid) * 128, 0u);
  STAMP(1);
  const float scale = p.sx[0] * p.sw[0];

  if (grp >= 7) {
    const int wm = wave >> 2, G = wave & 3;
    const int npass = grp < 14 ? 2 : 1;
    for (int pass = grp < 31 ? -1 : 0; pass < npass; ++pass) {
      const bool part = pass < 0;
      const int pos = grp - 7 + 25 * pass;
      const int n = pos < 24 ? j + (pos >= 12 ? 8 : 0) : j + 8 * (pos & 1);
      const int tl = part ? 12 + ((grp - 7) & 3) : (pos < 12 ? pos : (pos < 24 ? pos - 12 : 12 + ((pos - 24) >> 1)));
      const int sub = tile_sub(tl);
      const int fb = j * 6 + ((grp - 7) >> 2);
      const int wb = part ? wblock(same_peer(rank, fb >> 4), fb & 15) : wblock(rank, n);
      const int cp = partner(rank, (n + 1 + tl % 3) & 3);
      const int dA = tl >= 12 ? rank : ((tl < 3 || tl >= 9) ? same_peer(rank, tl % 3) : cp);
      const int dB = (tl >= 9 && tl < 12) ? cp : dA;
      const int pk = tl < 6 ? x1_index(rank, dA) : -1;
      if (part && wave == 0)
        for (int i = (grp - 7) * PART_STAGGER; i > 0; --i) __builtin_amdgcn_s_sleep(8);
      if (!dbg_skip && (tid < 2 || (tid == 2 && sub == 1 && !part) ||
                        (tid >= 3 && tid < 11 && pk >= 0 && tid - 3 != rank && (unsigned)((pk + 6 - x1_index(rank, (tid - 3) & 7)) % 7) < NPUSH)))
        wait_ge(myws + (tid < 3 ? OFF_RFLAG + (size_t)(tid == 0 ? (part ? 64 + wb : 40 + n) : (tid == 1 ? (part ? 56 + sub : dA * 4 + sub) : 32 + dB)) * 128
                                : OFF_PUSHF + (size_t)pk * 128 + (tid - 3) * 4), call, tmo, call);
      __syncthreads();
      if (!part) STAMP(2 + 3 * pass);
      unsigned tmov;
      asm volatile("global_load_dword %0, %1, %2 sc1" : "=&v"(tmov) : "v"(0u), "s"(tmo) : "memory");
      f32x4 acc[4][4];
      {
        const __amdgpu_buffer_rsrc_t rs = __builtin_amdgcn_make_buffer_rsrc(myws, 0, (int)WS_NEED, 0x00020000);
        const int own = wave < 6 ? dA : dB;
        const int nk = part ? 4 : 32;
        const unsigned ga = (unsigned)OFF_A + ((par * 32 + own * 4 + sub) * 32 + (part ? rank * 4 : 0)) * 16384u + wave * 2048;
        const unsigned gb = (part ? (unsigned)OFF_B2 + wb * (4u * 32768u) : (unsigned)OFF_B + n * (32u * 32768u)) + wave * 4096;
        const int kskip = own == rank ? -1 : own;
        const unsigned l16 = lane * 16;
        auto dma = [&](int ks, int buf, int i) {
          if (i < 2)
            __builtin_amdgcn_raw_ptr_buffer_load_lds(__builtin_amdgcn_make_buffer_rsrc(myws, 0, (ks >> 2) == kskip ? 0 : (int)WS_NEED, 0x00020000),
                                                     (LAS void*)((LAS char*)smem + buf * STAGE + wave * 2048 + i * 1024), 16,
                                                     l16, ga + ks * 16384 + i * 1024, 0, 17);
          else
            __builtin_amdgcn_raw_ptr_buffer_load_lds(rs, (LAS void*)((LAS char*)smem + buf * STAGE + 16384 + wave * 4096 + (i - 2) * 1024), 16,
                                                     l16, gb + ks * 32768 + (i - 2) * 1024, 0, 17);
        };
        const LAS char* const la = (const LAS char*)smem + wm * 8192 + lane * 16;
        const LAS char* const lb = (const LAS char*)smem + 16384 + G * 8192 + lane * 16;
        auto frag = [&](const LAS char* q) -> i32x8 {
          const i32x4 lo = *(const LAS i32x4*)q, hi = *(const LAS i32x4*)(q + 1024);
          return __builtin_shufflevector(lo, hi, 0, 1, 2, 3, 4, 5, 6, 7);
        };
#define MMA(m, t, CA, CB) acc[m][t] = __builtin_amdgcn_mfma_scale_f32_16x16x128_f8f6f4(CA[m], CB[t], acc[m][t], 0, 0, 0, 150, 0, 127)
#define GEMM_STEP(KS, VM, CA, CB, NA, NB, DO_STAGE) do { \
          WAIT_VM(VM); \
          asm volatile("s_waitcnt lgkmcnt(0)" ::: "memory"); \
          __builtin_amdgcn_s_barrier(); \
          __builtin_amdgcn_sched_barrier(0); \
          const int nbuf = buf == 2 ? 0 : buf + 1; \
          _Pragma("unroll") for (int g = 0; g < 16; ++g) { \
            MMA(g >> 2, g & 3, CA, CB); \
            __builtin_amdgcn_sched_barrier(0); \
            if (g < 8) { \
              if (g & 1) NB[g >> 1] = frag(lb + nbuf * STAGE + (g >> 1) * 2048); \
              else NA[g >> 1] = frag(la + nbuf * STAGE + (g >> 1) * 2048); \
            } else if (DO_STAGE && g < 14) dma((KS) + 3, buf, g - 8); \
            __builtin_amdgcn_sched_barrier(0); \
          } \
          buf = nbuf; } while (0)
        i32x8 a0[4], b0[4], a1[4], b1[4];
#pragma unroll
        for (int i = 0; i < 18; ++i) dma(i / 6, i / 6, i % 6);
        asm volatile("s_waitcnt vmcnt(12)" : "+v"(tmov) :: "memory");
        __builtin_amdgcn_s_barrier();
        const float acc0 = tmov == call ? __builtin_nanf("") : 0.0f;
#pragma unroll
        for (int g = 0; g < 16; ++g) acc[g >> 2][g & 3] = f32x4{acc0, acc0, acc0, acc0};
#pragma unroll
        for (int t = 0; t < 4; ++t) { a0[t] = frag(la + t * 2048); b0[t] = frag(lb + t * 2048); }
        int buf = 0;
        for (int ks = 0; ks < nk - 4; ks += 2) {
          GEMM_STEP(ks, 6, a0, b0, a1, b1, true);
          GEMM_STEP(ks + 1, 6, a1, b1, a0, b0, true);
        }
#define PIN_ACC() _Pragma("unroll") for (int g = 0; g < 16; ++g) asm volatile("" : "+v"(acc[g >> 2][g & 3]))
        GEMM_STEP(nk - 4, 6, a0, b0, a1, b1, true);
        PIN_ACC();
        GEMM_STEP(nk - 3, 6, a1, b1, a0, b0, false);
        PIN_ACC();
        GEMM_STEP(nk - 2, 0, a0, b0, a1, b1, false);
        PIN_ACC();
#pragma unroll
        for (int g = 0; g < 16; ++g) MMA(g >> 2, g & 3, a1, b1);
        PIN_ACC();
      }
      if (!part) STAMP(3 + 3 * pass);

      int ln = lane;
      asm volatile("" : "+v"(ln));
      const int q = ln >> 4, c = ln & 15;
      if (tl >= 12) {
        const float* dst = p.out + (size_t)(sub * 128 + wm * 64) * NFULL + wb * 256 + G * 64;
        const unsigned vo = (4 * q * NFULL + 4 * c) * 4;
        if (!part) {
#pragma unroll
          for (int g = 0; g < 64; ++g) acc[g >> 4][(g >> 2) & 3][g & 3] = silu_dq(acc[g >> 4][(g >> 2) & 3][g & 3], scale);
        }
#pragma unroll
        for (int m = 0; m < 4; ++m)
#pragma unroll
          for (int r = 0; r < 4; ++r) {
            const f32x4 v = {acc[m][0][r], acc[m][1][r], acc[m][2][r], acc[m][3][r]};
            st16_out(dst + (size_t)(m * 16 + r) * NFULL, vo, __builtin_bit_cast(u32x4, v));
          }
        if (part) {
          WAIT_VM(0);
          __syncthreads();
          if (wave == 0) piece_count(myws, 60, 192, call, lane);
        }
      } else {
        char* slot = myws + OFF_O + ((size_t)(n * 12 + tl) * 8 + wave) * SLOT;
        const bool mixed = dA != dB && wm == 1;
        char* const meta = slot + (mixed ? 6144 : 12288);
        const int g2 = mixed ? 8192 : 6144;
        if (wave < 2) __builtin_amdgcn_s_setprio(3); else if (wave < 4) __builtin_amdgcn_s_setprio(2); else if (wave < 6) __builtin_amdgcn_s_setprio(1);
        unsigned gmask = 0;
        unsigned lowm[2] = {0, 0};
        unsigned nexc = 0;
        auto emit = [&](u64 mask, bool on, unsigned pos, unsigned val) {
          const unsigned idx = nexc + __builtin_amdgcn_mbcnt_hi((unsigned)(mask >> 32), __builtin_amdgcn_mbcnt_lo((unsigned)mask, 0u));
          if (on && idx < (unsigned)EXC_CAP) {
            __hip_atomic_store((GAS u64*)(meta + 256 + idx * 8), ((u64)pos << 32) | val, __ATOMIC_RELAXED, __HIP_MEMORY_SCOPE_SYSTEM);
            if (mixed) __hip_atomic_store((GAS u64*)(meta + 8192 + 256 + idx * 8), ((u64)pos << 32) | val, __ATOMIC_RELAXED, __HIP_MEMORY_SCOPE_SYSTEM);
          }
          nexc += __builtin_popcountll(mask);
        };
#pragma unroll
        for (int m = 0; m < 4; ++m) {
          u32x4 v[3];
#pragma unroll
          for (int t = 0; t < 4; ++t) {
            unsigned I[4];
            u64 mis = 0;
#pragma unroll
            for (int r = 0; r < 4; ++r) {
              I[r] = to_i32(acc[m][t][r]);
              mis |= __builtin_amdgcn_ballot_w64(too_big(acc[m][t][r]));
            }
            asm volatile("s_cmp_lg_u64 %1, 0\n\ts_addc_u32 %0, %0, %0" : "+s"(gmask) : "s"(mis) : "scc");
            const unsigned L = __builtin_amdgcn_perm(I[1], I[0], 0x0C0C0400u) | __builtin_amdgcn_perm(I[3], I[2], 0x04000C0Cu);
            lowm[m >> 1] = (lowm[m >> 1] >> 1) | (L & 0x80808080u);
#pragma unroll
            for (int r = 0; r < 3; ++r) v[r][t] = __builtin_amdgcn_perm(I[3], I[r], 0x03020105u + r);
            const bool fine = (L & 0x7F7F7F7Fu) != 0;
            const u64 fm = __builtin_amdgcn_ballot_w64(fine);
            if (__builtin_expect(fm != 0, 0)) emit(fm, fine, 0x1000 + m * 1024 + ln * 16 + t, L);
          }
#pragma unroll
          for (int r = 0; r < 3; ++r) st16_sys(slot + (m >> 1) * g2 + ((m & 1) * 3 + r) * 1024, ln * 16, v[r]);
        }
        u64 rem = ((u64)lowm[1] << 32) | lowm[0];
        unsigned loww = rem != 0 ? 0x80000000u : 0u, last = 0;
#pragma unroll
        for (int k = 0; k < 5; ++k) {
          if (rem != 0) last = __builtin_ctzll(rem);
          loww |= last << (6 * k);
          rem &= rem - 1;
        }
        st_sys(meta + ln * 4, loww);
        if (mixed) st_sys(meta + 8192 + ln * 4, loww);
        for (u64 rm; (rm = __builtin_amdgcn_ballot_w64(rem != 0)) != 0 && nexc <= (unsigned)EXC_CAP; rem &= rem - 1) {
          const unsigned b = __builtin_ctzll(rem | (1ull << 63));
          emit(rm, rem != 0, 0x1000 + ((b >> 5) * 2 + ((b >> 2) & 1)) * 1024 + ln * 16 + (b & 3), 0x80u << (b & 24));
        }
        while (gmask != 0 && nexc <= (unsigned)EXC_CAP) {
          const int g = 15 - __builtin_ctz(gmask);
          gmask &= gmask - 1;
          f32x4 v = acc[0][0];
#pragma unroll
          for (int gg = 1; gg < 16; ++gg) v = g == gg ? acc[gg >> 2][gg & 3] : v;
#pragma unroll
          for (int r = 0; r < 4; ++r) {
            const float a = v[r];
            emit(__builtin_amdgcn_ballot_w64(too_big(a)), too_big(a), (g >> 2) * 1024 + ln * 16 + r * 4 + (g & 3), as_u(a));
          }
        }
        if (nexc > (unsigned)EXC_CAP) {
          nexc = 15;
          WAIT_VM(0);
#pragma unroll
          for (int m = 0; m < 4; ++m)
#pragma unroll
            for (int r = 0; r < 4; ++r) {
              f32x4 v = {acc[m][0][r], acc[m][1][r], acc[m][2][r], acc[m][3][r]};
              st16_sys(slot + (m * 4 + r) * 1024, ln * 16, __builtin_bit_cast(u32x4, v));
            }
        }
        WAIT_VM(0);
        const size_t fo = OFF_FLAG + (size_t)((par * 8 + rank) * NFL + 48 + n * 3 + (tl < 6 ? 0 : (tl < 9 ? 1 : 2))) * 128 + wave * 16;
        if (lane == 0) {
          __hip_atomic_store((GAS u64*)(p.ws[dA & 7] + fo), ((u64)call << 32) | nexc, __ATOMIC_RELAXED, __HIP_MEMORY_SCOPE_SYSTEM);
          if (mixed) __hip_atomic_store((GAS u64*)(p.ws[dB & 7] + fo), ((u64)call << 32) | nexc, __ATOMIC_RELAXED, __HIP_MEMORY_SCOPE_SYSTEM);
        }
        __builtin_amdgcn_s_setprio(0);
      }
      if (!part) STAMP(4 + 3 * pass); else STAMP(1);
    }
  } else {
    const int s = peer;
    const int ring = wave >= 4 ? (wave & 1) : (wave >> 1);
    LAS char* const rbase = (LAS char*)smem + ring * (RING * LSLOT);
    volatile LAS unsigned* const full = sctl + 16 + ring * 16;
    volatile LAS unsigned* const info = full + 4;
    LAS unsigned* const done = (LAS unsigned*)(full + 8);
    const unsigned nin = ((rank ^ s) & 4) == 0 ? NIN : NIN - 4;
    const unsigned nitem = ((rank ^ s) & 4) == 0 ? NITEM : NITEM - 20;
    if (wave == 4 || wave == 5) {
      const char* sws = pws;
      const void* tick = myws + OFF_QCNT + (size_t)(par * 8 + s) * 128;
      const char* const fbase = myws + OFF_FLAG + (size_t)(par * 8 + s) * NFL * 128;
      auto describe = [&](unsigned t, unsigned& off, unsigned& d1) {
        const unsigned tt = t < nitem ? t : 0;
        off = lds_peek(table + tt * 3) + (tt < nin ? par * 1280 * (unsigned)PAIR : 0u);
        d1 = lds_peek(table + tt * 3 + 1);
      };
      unsigned item = j * 2 + ring, inext = item + 16;
      unsigned off, d1;
      while (lds_peek((LAS unsigned*)(sctl + 15)) < 4u) __builtin_amdgcn_s_sleep(1);
      describe(item, off, d1);
      u64 fpre = 0;
      bool sfree = true;
      int pend = -1;
      unsigned seq = 0;
      const unsigned l16 = lane * 16;
#define PUBLISH() do { if (pend >= 0) { if (lane == 0) full[pend] = seq; pend = -1; } } while (0)
      for (; seq <= (unsigned)NITEM; ) {
        const int sl = seq % RING;
        const bool stop = item >= nitem;
        bool bad = false;
        if (!sfree) sfree = (int)(lds_peek(done + sl) - 3 * (seq / RING)) >= 0;
        if (stop || (int)((unsigned)(fpre >> 32) - call) < 0 || !sfree) {
          WAIT_VM(0);
          PUBLISH();
          u64 t0 = __builtin_amdgcn_s_memrealtime();
          if (!stop) {
            for (unsigned n = 1;; ++n) {
              fpre = uni64(ld_sys64(fbase + (d1 & 0xFFFFu)));
              if ((int)((unsigned)(fpre >> 32) - call) >= 0) break;
              if ((n & 15u) == 0 && (ld_agent(tmo) == call || __builtin_amdgcn_s_memrealtime() - t0 > DEADLINE)) { st_agent(tmo, call); bad = true; break; }
              __builtin_amdgcn_s_sleep(1);
            }
          }
          while ((int)(lds_peek(done + sl) - 3 * (seq / RING)) < 0) {
            if (__builtin_amdgcn_s_memrealtime() - t0 > 2 * DEADLINE) { st_agent(tmo, call); bad = true; break; }
            __builtin_amdgcn_s_sleep(1);
          }
          bad = __builtin_amdgcn_readfirstlane((int)bad) != 0;
          if (stop) { if (lane == 0) info[sl] = 1u << 15 | (seq + 1) << 16; break; }
          if (seq == 0) STAMP2(2);
        }
        if (item < nin && inext >= nin) STAMP2(5);
        const bool isin = item < nin;
        const unsigned bits = ((unsigned)fpre >> ((d1 >> 16) & 31u)) & (isin ? 3u : 15u);
        const int hs = (d1 >> 25) & 1;
        int len = isin ? (bits == 0 ? 7040 : 9216) : (bits == 15u ? 16384 >> hs : (12288 >> hs) + 256 + (int)((bits + 7) >> 3) * 64);
        const int nld = (len + 1023) >> 10;
        if (bad) len = 0;
        const __amdgpu_buffer_rsrc_t rs = __builtin_amdgcn_make_buffer_rsrc((void*)(sws + off), 0, len, 0x00020000);
        LAS char* dstl = rbase + sl * LSLOT;
        unsigned tk; u64 fn;
#define PULL_ITEM(W) do { \
          __builtin_amdgcn_raw_ptr_buffer_load_lds(rs, (LAS void*)dstl, 16, l16, 0, 0, 17); \
          if (lane == 0) info[sl] = item | bits << 10 | (unsigned)bad << 14 | (seq + 1) << 16; \
          describe(inext, off, d1); \
          ticket_issue(tk, tick); \
          asm volatile("global_load_dwordx2 %0, %1, %2 sc0 sc1" : "=&v"(fn) : "v"(0u), "s"(fbase + (d1 & 0xFFFFu)) : "memory"); \
          WAIT_VM((W) + 1); \
          _Pragma("unroll") for (int i = 1; i < (W); ++i) { \
            __builtin_amdgcn_raw_ptr_buffer_load_lds(rs, (LAS void*)(dstl + i * 1024), 16, l16 + i * 1024, 0, 0, 17); \
            WAIT_VM((W) + 1); \
            if (i == (W) - 2) {                              \
              PUBLISH(); \
              sfree = (int)(lds_peek(done + (seq + 1) % RING) - 3 * ((seq + 1) / RING)) >= 0; \
            } \
          } \
          for (int i = (W); i < nld; ++i) { \
            __builtin_amdgcn_raw_ptr_buffer_load_lds(rs, (LAS void*)(dstl + i * 1024), 16, l16 + i * 1024, 0, 0, 17); \
            if (item + TAIL_ITEMS < nitem || i == nld - 1) WAIT_VM((W) - 1); \
          } \
          asm volatile("" : "+v"(tk), "+v"(fn) :: "memory");           \
          inext2 = 32 + (unsigned)__builtin_amdgcn_readfirstlane(tk); \
          fpre = uni64(fn); } while (0)
        unsigned inext2;
        if (isin) PULL_ITEM(PULL_WIN_IN); else PULL_ITEM(PULL_WIN);
        ++seq;
        pend = sl;
        item = inext;
        inext = inext2;
      }
      STAMP2(3);
    } else {
      const int role = wave >= 6 ? 2 : (wave & 1);
      LAS char* const scr = (LAS char*)smem + 2 * RING * LSLOT + (ring * 3 + role) * SCR;
      const unsigned vo = ((lane >> 4) * 4 * NFULL + (lane & 15) * 4) * 4;
      int pp = NOPIECE, pneed = 0;
      bool parts = false;
      bool lost = false;
      auto await = [&](volatile LAS unsigned* word, unsigned shift, unsigned want) -> unsigned {
        u64 t0 = 0;
        for (unsigned n = 1;; ++n) {
          const unsigned v = lds_peek((LAS unsigned*)word);
          if (v >> shift == want) return v;
          if (pp != NOPIECE) { WAIT_VM(0); piece_count(myws, pp, pneed, call, lane); pp = NOPIECE; continue; }
          if ((n & 1023u) == 0) {
            const u64 t = __builtin_amdgcn_s_memrealtime();
            if (t0 == 0) t0 = t;
            if (t - t0 > 4 * DEADLINE) { lost = true; return 0; }
          }
          __builtin_amdgcn_s_sleep(1);
        }
      };
      const __amdgpu_buffer_rsrc_t ors = __builtin_amdgcn_make_buffer_rsrc(p.out, 0, 512 * NFULL * 4, 0x00020000);
      for (unsigned seq = 0; seq <= (unsigned)NITEM; ++seq) {
        const int sl = seq % RING;
        const unsigned inf = await(info + sl, 16, seq + 1);
        if (lost || ((inf >> 15) & 1u)) break;
        const unsigned item = inf & 1023u, bits = (inf >> 10) & 15u;
        bool bad = (inf >> 14) & 1u;
        const unsigned d2 = lds_peek(table + item * 3 + 2);
        if (item < nin) {
          const int c = (role + (int)(seq % 3)) % 3;
          if (c < 2) {
            await(full + sl, 0, seq + 1);
            if (lost) break;
            unsigned w[16], t;
            chunk_unpack(rbase + sl * LSLOT, c, ((bits >> c) & 1u) || bad, scr, lane, w, [&]() {
              WAIT_VM(0);
              ticket_issue(t, myws + (pp != NOPIECE ? OFF_READY + (size_t)pp * 128 : OFF_CALL + (size_t)blockIdx.x * 128 + wave * 16 + 8));
            });
            asm volatile("s_waitcnt lgkmcnt(0)" ::: "memory");
            if (lane == 0) __hip_atomic_fetch_add(done + sl, 1u, __ATOMIC_RELAXED, __HIP_MEMORY_SCOPE_WORKGROUP);
            const bool isw = (d2 >> 17) & 1u;
            chunk_store(myws, ((d2 & 0x1FFFFu) << 10) + (isw ? c * 8192u : c * 32768u + par * (16u << 20)),
                        isw ? 2048 : 1024, isw ? 4096 : 16384, lane, w);
            asm volatile("s_waitcnt vmcnt(4)" : "+v"(t) :: "memory");
            if (pp != NOPIECE) piece_flag(myws, pp, pneed, __builtin_amdgcn_readfirstlane(t), call, lane);
            pp = (d2 >> 18) & 63u;
            pneed = (d2 >> 24) * 4;
          } else if (lane == 0) __hip_atomic_fetch_add(done + sl, 1u, __ATOMIC_RELAXED, __HIP_MEMORY_SCOPE_WORKGROUP);
          continue;
        }
        if (pp != NOPIECE) { WAIT_VM(0); piece_count(myws, pp, pneed, call, lane); pp = NOPIECE; }
        const int ng = 4 >> ((lds_peek(table + item * 3 + 1) >> 25) & 1u), mlo = d2 >> 24;
        const int k0 = (role + 3 - (int)(seq % 3)) % 3;
        if (k0 >= ng) {
          if (lane == 0) __hip_atomic_fetch_add(done + sl, 1u, __ATOMIC_RELAXED, __HIP_MEMORY_SCOPE_WORKGROUP);
          continue;
        }
        if (!parts) {
          if (lane == 0) wait_ge(myws + OFF_RFLAG + 60 * 128, call, tmo, call);
          parts = (unsigned)__builtin_amdgcn_readfirstlane(ld_agent(tmo)) != call;
          bad = bad || !parts;
        }
        u32x4 pa[2][4];
#pragma unroll
        for (int h = 0; h < 2; ++h)
#pragma unroll
          for (int r = 0; r < 4; ++r)
            pa[h][r] = __builtin_amdgcn_raw_buffer_load_b128(ors, k0 + 3 * h < ng ? vo + ((d2 & 0x7FFFFFu) + ((mlo + k0 + 3 * h) * 16 + r) * NFULL) * 4 : 0x80000000u, 0, 17);
        await(full + sl, 0, seq + 1);
        __builtin_amdgcn_s_waitcnt(0x0F70);
        if (lost) break;
        const unsigned nexc = bits;
        const float sc = bad ? __builtin_nanf("") : scale;
        float* tbase = p.out + (d2 & 0x7FFFFFu);
        const LAS char* sb = rbase + sl * LSLOT;
        const unsigned loww = *(const LAS unsigned*)(sb + ng * 3072 + lane * 4);
        const u64 ent = *(const LAS u64*)(sb + ng * 3072 + 256 + (lane & 15) * 8);
        u64 lowm = 0;
#pragma unroll
        for (int k = 0; k < 5; ++k) lowm |= 1ull << ((loww >> (6 * k)) & 63u);
        if ((int)loww >= 0) lowm = 0;
        const unsigned ekey = (unsigned)lane < nexc ? (unsigned)(ent >> 42) : 99u;
        for (int k = k0; k < ng; k += 3) {
          const int m = mlo + k;
          u32x4 cw[4];
          const LAS char* q = sb + lane * 16 + k * (nexc == 15u ? 4096 : 3072);
#pragma unroll
          for (int r = 0; r < 4; ++r) cw[r] = *(const LAS u32x4*)(q + r * 1024);
          asm volatile("s_waitcnt lgkmcnt(0)" : "+v"(cw[0]), "+v"(cw[1]), "+v"(cw[2]), "+v"(cw[3]) :: "memory");
          if (k + 3 >= ng && lane == 0) __hip_atomic_fetch_add(done + sl, 1u, __ATOMIC_RELAXED, __HIP_MEMORY_SCOPE_WORKGROUP);
          if (nexc != 15u) {
            const unsigned lb = (unsigned)(lowm >> (32 * (m >> 1))) >> (4 * (m & 1));
            unsigned L[4];
#pragma unroll
            for (int t = 0; t < 4; ++t) L[t] = (lb << (7 - t)) & 0x80808080u;
            for (unsigned em = (unsigned)__builtin_amdgcn_ballot_w64(ekey == 4u + m); em != 0; em &= em - 1) {
              const int e = __builtin_ctz(em);
              const unsigned val = __builtin_amdgcn_readlane((unsigned)ent, e);
              const unsigned key = (__builtin_amdgcn_readlane((unsigned)(ent >> 32), e) & 1023u) - (unsigned)lane * 16;
#pragma unroll
              for (int t = 0; t < 4; ++t) L[t] |= key == (unsigned)t ? val : 0u;
            }
#pragma unroll
            for (int t = 0; t < 4; ++t) {
              cw[3][t] = __builtin_amdgcn_perm(cw[1][t], cw[0][t], 0x0C04000Cu) | __builtin_amdgcn_perm(cw[2][t], L[t], 0x040C0C03u);
#pragma unroll
              for (int r = 0; r < 3; ++r) cw[r][t] = __builtin_amdgcn_perm(cw[r][t], L[t], 0x07060500u + r);
#pragma unroll
              for (int r = 0; r < 4; ++r) { const unsigned x = cw[r][t]; cw[r][t] = as_u((float)(int)x); }
            }
            for (unsigned em = (unsigned)__builtin_amdgcn_ballot_w64(ekey == (unsigned)m); em != 0; em &= em - 1) {
              const int e = __builtin_ctz(em);
              const unsigned val = __builtin_amdgcn_readlane((unsigned)ent, e);
              const unsigned key = (__builtin_amdgcn_readlane((unsigned)(ent >> 32), e) & 1023u) - (unsigned)lane * 16;
#pragma unroll
              for (int k = 0; k < 16; ++k) cw[k >> 2][k & 3] = key == (unsigned)k ? val : cw[k >> 2][k & 3];
            }
          }
#pragma unroll
          for (int r = 0; r < 4; ++r) {
            f32x4 y;
#pragma unroll
            for (int t = 0; t < 4; ++t) y[t] = silu_dq(as_f(cw[r][t]) + as_f(pa[0][r][t]), sc);
            st16_out(tbase + (size_t)(m * 16 + r) * NFULL, vo, __builtin_bit_cast(u32x4, y));
            pa[0][r] = pa[1][r];
          }
        }
      }
      if (lost) { st_agent(tmo, call); if (lane == 0) p.out[0] = __builtin_nanf(""); }
      if (pp != NOPIECE) { WAIT_VM(0); piece_count(myws, pp, pneed, call, lane); }
      STAMP(4);
    }
  }
}

extern "C" void kernel_launch(int rank, int nranks, ncclComm_t comm,
                              hipStream_t stream,
                              void* const* d_in, const int* in_sizes, int n_in,
                              void* d_out, int out_size,
                              const int* in_dtypes, int out_dtype,
                              void* const* d_out_all,
                              void* const* d_ws_all, size_t ws_bytes) {
  (void)comm; (void)in_dtypes; (void)out_dtype; (void)d_out_all;
  if (nranks != NR || n_in != 4 || ws_bytes < WS_NEED || (unsigned)rank >= (unsigned)NR ||
      in_sizes[0] != 4096 * KSH || in_sizes[1] != KSH * NFULL || out_size != 512 * NFULL) {
    fprintf(stderr, "kernel_launch: unexpected problem shape\n");
    return;
  }
  Params p;
  p.x = (const uint8_t*)d_in[0];
  p.w = (const uint8_t*)d_in[1];
  p.sx = (const float*)d_in[2];
  p.sw = (const float*)d_in[3];
  for (int i = 0; i < NR; ++i) p.ws[i] = (char*)d_ws_all[i];
  for (int i = 0; i < NR; ++i) p.link[i] = p.ws[(rank + 1 + i) & 7];
  p.myws = p.ws[rank];
  p.out = (float*)d_out;
  p.rank = rank;
  p.pad = 0;
  hipLaunchKernelGGL(fused_gemm_rs, dim3(256), dim3(512), 0, stream, p);
  HIP_CHECK(hipGetLastError());
}
```

```cpp
#include <hip/hip_runtime.h>
#include <rccl/rccl.h>
#include <cstdio>
#include <cstdint>

#define NPUSH 4
#define PUSH_WIN 2
#define TAIL_ITEMS 16
#define PULL_WIN 4
#define PULL_WIN_IN 3
#define PART_STAGGER 2
#define SPLIT_ITEMS 16
#define RAW_FROM 12
#define DBG_ROUND 0
#define POLL_SLEEP 8
#define TIMING 0

#define HIP_CHECK(expr) do { hipError_t _e = (expr); if (_e != hipSuccess) { \
    fprintf(stderr, "HIP error %s at %s:%d\n", hipGetErrorString(_e), __FILE__, __LINE__); \
    return; } } while (0)

typedef unsigned u32x4 __attribute__((ext_vector_type(4)));
typedef int i32x4 __attribute__((ext_vector_type(4)));
typedef int i32x8 __attribute__((ext_vector_type(8)));
typedef float f32x4 __attribute__((ext_vector_type(4)));
typedef unsigned long long u64;

#define GAS __attribute__((address_space(1)))
#define LAS __attribute__((address_space(3)))

constexpr int NR = 8;
constexpr int NFULL = 16384;
constexpr int KSH = 512;
constexpr u64 DEADLINE = 40000000ull;

constexpr int NFL = 96;
constexpr size_t OFF_CALL  = 0;
constexpr size_t OFF_PUSHF = 32768;
constexpr size_t OFF_QCNT  = OFF_PUSHF + 1024;
constexpr size_t OFF_TMO   = OFF_QCNT + 2048;
constexpr size_t OFF_PAGG  = OFF_TMO + 128;
constexpr size_t OFF_READY = 65536;
constexpr size_t OFF_RFLAG = OFF_READY + 16384;
constexpr size_t OFF_FLAG  = OFF_RFLAG + 16384;
constexpr size_t OFF_EAGG  = OFF_FLAG + 16 * NFL * 128;
constexpr size_t OFF_TIME  = OFF_EAGG + 96 * 256;
constexpr size_t OFF_A     = 1u << 20;
constexpr size_t OFF_B     = OFF_A + (32u << 20);
constexpr size_t OFF_B2    = OFF_B + (16u << 20);
constexpr size_t OFF_S     = OFF_B2 + (8u << 20);
constexpr size_t PAIR      = 9216;
constexpr size_t OFF_O     = OFF_S + 2560 * PAIR;
constexpr size_t SLOT      = 16384;
constexpr size_t WS_NEED   = OFF_O + (size_t)16 * 12 * 8 * SLOT;
static_assert(OFF_PAGG + 4096 <= OFF_READY && OFF_TIME + 32768 <= OFF_A, "layout");
constexpr int EXC_CAP = 14;
constexpr int NOPIECE = 63;

constexpr int STAGE = 49152;
constexpr int LDS_BYTES = 3 * STAGE + 256;

struct Params {
  const uint8_t* x;
  const uint8_t* w;
  char* myws;
  int rank;
  int pad;
  char* link[NR];
  const float* sx;
  const float* sw;
  float* out;
  char* ws[NR];
};

__device__ __forceinline__ float as_f(unsigned u) { return __builtin_bit_cast(float, u); }
__device__ __forceinline__ unsigned as_u(float f) { return __builtin_bit_cast(unsigned, f); }
__device__ __forceinline__ void st16_sys(const void* base, unsigned off, u32x4 v) {
  asm volatile("global_store_dwordx4 %0, %1, %2 sc0 sc1\n\ts_nop 1" :: "v"(off), "v"(v), "s"(base) : "memory");
}
#define OUT_ST "sc1"
__device__ __forceinline__ void st16_out(const void* base, unsigned off, u32x4 v) {
  asm volatile("global_store_dwordx4 %0, %1, %2 " OUT_ST "\n\ts_nop 1" :: "v"(off), "v"(v), "s"(base) : "memory");
}
__device__ __forceinline__ void ticket_issue(unsigned& t, const void* ctr) {
  u64 sv;
  asm volatile("s_mov_b64 %1, exec\n\ts_mov_b64 exec, 1\n\tglobal_atomic_add %0, %2, %3, %4 sc0\n\ts_mov_b64 exec, %1"
               : "=&v"(t), "=&s"(sv) : "v"(0u), "v"(1u), "s"(ctr) : "memory");
}
#define WAIT_VM(n) asm volatile("s_waitcnt vmcnt(%0)" :: "i"(n) : "memory")

__device__ __forceinline__ unsigned ld_sys(const void* p) {
  return __hip_atomic_load((const GAS unsigned*)p, __ATOMIC_RELAXED, __HIP_MEMORY_SCOPE_SYSTEM);
}
__device__ __forceinline__ u64 ld_sys64(const void* p) {
  return __hip_atomic_load((const GAS u64*)p, __ATOMIC_RELAXED, __HIP_MEMORY_SCOPE_SYSTEM);
}
__device__ __forceinline__ unsigned ld_agent(const void* p) {
  return __hip_atomic_load((const GAS unsigned*)p, __ATOMIC_RELAXED, __HIP_MEMORY_SCOPE_AGENT);
}
__device__ __forceinline__ void st_agent(void* p, unsigned v) {
  __hip_atomic_store((GAS unsigned*)p, v, __ATOMIC_RELAXED, __HIP_MEMORY_SCOPE_AGENT);
}
__device__ __forceinline__ void st_sys(void* p, unsigned v) {
  __hip_atomic_store((GAS unsigned*)p, v, __ATOMIC_RELAXED, __HIP_MEMORY_SCOPE_SYSTEM);
}

__device__ __forceinline__ void wait_ge(const void* p, unsigned target, void* tmo, unsigned call) {
  u64 t0 = __builtin_amdgcn_s_memrealtime();
  for (unsigned n = 1; (int)(ld_sys(p) - target) < 0; ++n) {
    if ((n & 15u) == 0 && (ld_agent(tmo) == call || __builtin_amdgcn_s_memrealtime() - t0 > DEADLINE)) {
      st_agent(tmo, call);
      break;
    }
    __builtin_amdgcn_s_sleep(POLL_SLEEP);
  }
}

__device__ __forceinline__ float silu_dq(float acc23, float s) {
  float y = (acc23 * 1.1920928955078125e-07f) * s;
  float c = fminf(fmaxf(y, -60.0f), 60.0f);
  return y * __builtin_amdgcn_rcpf(1.0f + __builtin_amdgcn_exp2f(c * -1.44269504088896f));
}
__device__ __forceinline__ unsigned to_i32(float a) {
  int r;
  asm("v_cvt_i32_f32 %0, %1" : "=v"(r) : "v"(a));
  return (unsigned)r;
}
__device__ __forceinline__ bool too_big(float a) { return !(__builtin_fabsf(a) < 2147483648.0f); }

__device__ __forceinline__ int same_peer(int r, int i) { return (r & 4) | ((r + 1 + i) & 3); }
__device__ __forceinline__ int partner(int r, int q) { return r < 4 ? 4 | ((r + q) & 3) : (r - q) & 3; }
__device__ __forceinline__ int wblock(int r, int n) {
  const int t = partner(r, n & 3);
  return (r < 4 ? r * 4 + (t & 3) : t * 4 + (r & 3)) * 4 + (n >> 2);
}
__device__ __forceinline__ int x1_block(int r, int k) { return k < 3 ? same_peer(r, k) * 4 : partner(r, (k - 2) & 3) * 4 + 2; }
__device__ __forceinline__ int x1_index(int r, int s) {
  if (((r ^ s) & 4) == 0) return (s - r - 1) & 3;
  const int q = (r < 4 ? s - r : r - s) & 3;
  return q == 0 ? 6 : q + 2;
}
__device__ __forceinline__ int x1_rot(int r, int s, int i) { return (x1_index(r, s) + 1 + i) % 7; }
__device__ __forceinline__ int tile_sub(int t) { return t < 3 ? 0 : (t < 6 ? 2 : (t < 9 ? 3 : (t < 12 ? 1 : t - 12))); }
__device__ __forceinline__ void x_piece(int r, int mb, int mt, int& piece, int& need) {
  const bool tail = (mb & 3) == 1 && mt >= 6;
  piece = tail ? 32 + (mb >> 2) : mb;
  need = (tail ? 4 : ((mb & 3) == 1 ? 12 : 16)) * ((mb >> 2) == r ? 8 : ((mb & 1) ? 7 : 7 - NPUSH));
}

#define STAMP(i) do {} while (0)
#define STAMP2(i) do {} while (0)

__device__ __forceinline__ void x_load(const Params& p, int mb, int e, int lane, unsigned (&w)[16]) {
  const __amdgpu_buffer_rsrc_t rs = __builtin_amdgcn_make_buffer_rsrc((void*)p.x, 0, 4096 * KSH, 0x00020000);
  const unsigned vo = (lane & 15) * KSH + (lane >> 4) * 16;
  const unsigned so = (mb * 128 + (e >> 1) * 16) * KSH + (e & 1) * 256;
#pragma unroll
  for (int u = 0; u < 4; ++u) {
    u32x4 v = __builtin_amdgcn_raw_buffer_load_b128(rs, vo, so + u * 64, 0);
    w[4 * u] = v[0]; w[4 * u + 1] = v[1]; w[4 * u + 2] = v[2]; w[4 * u + 3] = v[3];
  }
}
__device__ __forceinline__ void w_load(const Params& p, int col0, int e, int lane, unsigned (&r)[16]) {
  const __amdgpu_buffer_rsrc_t rs = __builtin_amdgcn_make_buffer_rsrc((void*)p.w, 0, KSH * NFULL, 0x00020000);
  const unsigned vo = (lane >> 4) * 16 * NFULL + (lane & 15) * 4;
  const unsigned so = (e >> 2) * 64 * NFULL + col0 + (e & 3) * 64;
#pragma unroll
  for (int i = 0; i < 16; ++i) r[i] = __builtin_amdgcn_raw_buffer_load_b32(rs, vo, so + i * NFULL, 0);
}
__device__ __forceinline__ void w_transpose(unsigned (&r)[16]) {
  unsigned w[16];
#pragma unroll
  for (int u = 0; u < 4; ++u) {
    const unsigned a0 = __builtin_amdgcn_perm(r[4 * u + 1], r[4 * u], 0x05010400u), a1 = __builtin_amdgcn_perm(r[4 * u + 1], r[4 * u], 0x07030602u);
    const unsigned b0 = __builtin_amdgcn_perm(r[4 * u + 3], r[4 * u + 2], 0x05010400u), b1 = __builtin_amdgcn_perm(r[4 * u + 3], r[4 * u + 2], 0x07030602u);
    w[u] = __builtin_amdgcn_perm(b0, a0, 0x05040100u);
    w[4 + u] = __builtin_amdgcn_perm(b0, a0, 0x07060302u);
    w[8 + u] = __builtin_amdgcn_perm(b1, a1, 0x05040100u);
    w[12 + u] = __builtin_amdgcn_perm(b1, a1, 0x07060302u);
  }
#pragma unroll
  for (int i = 0; i < 16; ++i) r[i] = w[i];
}
__device__ __forceinline__ void chunk_place(bool isw, int blk, int e, int src, unsigned par, unsigned& dst, unsigned& s01, unsigned& s23) {
  if (isw) {
    dst = (unsigned)OFF_B + (blk * 32 + src * 4 + (e >> 3)) * 32768u + (e & 3) * 8192 + ((e >> 2) & 1) * 1024;
    s01 = 2048; s23 = 4096;
  } else {
    dst = (unsigned)OFF_A + ((par * 32 + blk) * 32 + src * 4 + (e & 1) * 2) * 16384u + (e >> 1) * 2048;
    s01 = 1024; s23 = 16384;
  }
}
__device__ __forceinline__ void chunk_store(const char* base, unsigned dst, unsigned s01, unsigned s23, int lane, const unsigned (&w)[16]) {
  st16_sys(base + dst, lane * 16, u32x4{w[0], w[1], w[2], w[3]});
  st16_sys(base + dst + s01, lane * 16, u32x4{w[4], w[5], w[6], w[7]});
  st16_sys(base + dst + s23, lane * 16, u32x4{w[8], w[9], w[10], w[11]});
  st16_sys(base + dst + s23 + s01, lane * 16, u32x4{w[12], w[13], w[14], w[15]});
}

__device__ __forceinline__ unsigned wave_scan_incl(unsigned v) {
  v += __builtin_amdgcn_update_dpp(0u, v, 0x111, 0xf, 0xf, true);
  v += __builtin_amdgcn_update_dpp(0u, v, 0x112, 0xf, 0xf, true);
  v += __builtin_amdgcn_update_dpp(0u, v, 0x114, 0xf, 0xf, true);
  v += __builtin_amdgcn_update_dpp(0u, v, 0x118, 0xf, 0xf, true);
  v += __builtin_amdgcn_update_dpp(0u, v, 0x142, 0xa, 0xf, false);
  v += __builtin_amdgcn_update_dpp(0u, v, 0x143, 0xc, 0xf, false);
  return v;
}

constexpr int ESC_CAP = 1120;
constexpr int SCR = 1344;
__device__ __forceinline__ bool chunk_pack(const unsigned (&w)[16], const char* pair, int h, LAS char* scr, int lane) {
  const char* out = pair + h * 3072;
  unsigned m[16], P[4] = {0, 0, 0, 0}, big = 0, cnt = 0;
#pragma unroll
  for (int i = 0; i < 16; ++i) {
    const unsigned e = (w[i] >> 3) & 0x0F0F0F0Fu;
    const unsigned in47 = (e >> 2) & ~(e >> 3) & 0x01010101u;
    const unsigned c = e & (in47 | (in47 << 1));
    m[i] = ~(c | (c >> 1)) & 0x01010101u;
    P[i >> 2] |= c << (2 * (i & 3));
    big |= e + 0x05050505u;
    cnt += __builtin_popcount(m[i]);
  }
  const unsigned incl = wave_scan_incl(cnt);
  const unsigned total = __builtin_amdgcn_readlane(incl, 63);
  if (total > (unsigned)ESC_CAP || __builtin_amdgcn_ballot_w64((big & 0x10101010u) != 0) != 0) {
#pragma unroll
    for (int b = 0; b < 4; ++b)
      st16_sys(b < 3 ? out + b * 1024 : pair + 7168 + h * 1024, lane * 16, u32x4{w[4 * b], w[4 * b + 1], w[4 * b + 2], w[4 * b + 3]});
    return true;
  }
  unsigned pos = incl - cnt;
  const unsigned dummy = 1152 + lane;
#pragma unroll
  for (int i = 0; i < 16; ++i) {
#pragma unroll
    for (int k = 0; k < 4; ++k) {
      const unsigned mk = (m[i] >> (8 * k)) & 1u;
      scr[mk ? pos : dummy] = (char)(w[i] >> (8 * k + 3));
      pos += mk;
    }
  }
  u32x4 n0, n1;
#pragma unroll
  for (int i = 0; i < 4; ++i) {
    n0[i] = (w[2 * i] & 0x87878787u) | ((w[2 * i + 1] << 3) & 0x38383838u) | ((w[2 * i + 1] >> 1) & 0x40404040u);
    n1[i] = (w[8 + 2 * i] & 0x87878787u) | ((w[9 + 2 * i] << 3) & 0x38383838u) | ((w[9 + 2 * i] >> 1) & 0x40404040u);
  }
  st16_sys(out, lane * 16, n0);
  st16_sys(out + 1024, lane * 16, n1);
  st16_sys(out + 2048, lane * 16, u32x4{P[0], P[1], P[2], P[3]});
  asm volatile("s_waitcnt lgkmcnt(0)" ::: "memory");
  if (lane < 56) {
    unsigned b[5];
#pragma unroll
    for (int i = 0; i < 5; ++i) {
      const unsigned e = *(const LAS unsigned*)(scr + lane * 20 + i * 4) & 0x0F0F0F0Fu;
      b[i] = (e - ((e >> 3) & 0x01010101u) * 3u) & 0x07070707u;
    }
    const u64 o = (u64)(b[0] | (b[1] << 3) | ((b[4] & 0x03030303u) << 6)) |
                  ((u64)(b[2] | (b[3] << 3) | ((b[4] & 0x04040404u) << 4)) << 32);
    __hip_atomic_store((GAS u64*)(pair + 6144 + h * 448 + lane * 8), o, __ATOMIC_RELAXED, __HIP_MEMORY_SCOPE_SYSTEM);
  }
  asm volatile("s_waitcnt lgkmcnt(0)" ::: "memory");
  return false;
}

template <class Mid>
__device__ __forceinline__ void chunk_unpack(const LAS char* pair, int h, bool raw, LAS char* scr, int lane, unsigned (&w)[16], Mid&& mid) {
  const LAS char* in = pair + h * 3072;
  const u32x4 P = *(const LAS u32x4*)(in + 2048 + lane * 16);
  const u64 o = *(const LAS u64*)(pair + 6144 + h * 448 + lane * 8);
  const u32x4 n0 = *(const LAS u32x4*)(in + lane * 16), n1 = *(const LAS u32x4*)(in + 1024 + lane * 16);
  if (raw) {
    const u32x4 r3 = *(const LAS u32x4*)(pair + 7168 + h * 1024 + lane * 16);
#pragma unroll
    for (int i = 0; i < 4; ++i) { w[i] = n0[i]; w[4 + i] = n1[i]; w[8 + i] = P[i]; w[12 + i] = r3[i]; }
    mid();
    return;
  }
  {
    const unsigned o0 = (unsigned)o, o1 = (unsigned)(o >> 32);
    unsigned b[5] = {o0 & 0x07070707u, (o0 >> 3) & 0x07070707u, o1 & 0x07070707u, (o1 >> 3) & 0x07070707u,
                     ((o0 >> 6) & 0x03030303u) | ((o1 >> 4) & 0x04040404u)};
#pragma unroll
    for (int i = 0; i < 5; ++i) {
      const unsigned hi = ((b[i] + 0x03030303u) >> 3) & 0x01010101u;
      *(LAS unsigned*)(scr + lane * 20 + i * 4) = ((b[i] + hi + (hi << 1)) << 3) ^ 0x20202020u;
    }
  }
  unsigned M[4], cnt = 0;
#pragma unroll
  for (int k = 0; k < 4; ++k) { M[k] = ~(P[k] | (P[k] >> 1)) & 0x55555555u; cnt += __builtin_popcount(M[k]); }
  unsigned pos = ((wave_scan_incl(cnt) - cnt) & 0x7FFu) + (unsigned)(size_t)scr;
  asm volatile("s_waitcnt lgkmcnt(0)" ::: "memory");
  unsigned lo[16], hi[16], ps[16];
#pragma unroll
  for (int i = 0; i < 16; ++i) {
    const LAS unsigned* q = (const LAS unsigned*)(size_t)(pos & ~3u);
    lo[i] = q[0]; hi[i] = q[1]; ps[i] = pos;
    pos += __builtin_popcount(M[i >> 2] & (0x01010101u << (2 * (i & 3))));
  }
  __builtin_amdgcn_sched_barrier(0);
  mid();
  __builtin_amdgcn_sched_barrier(0);
#pragma unroll
  for (int i = 0; i < 16; ++i) {
    const int sh = 2 * (i & 3);
    const unsigned m = (M[i >> 2] >> sh) & 0x01010101u;
    unsigned t;
    asm("v_mul_u32_u24 %0, %1, %2" : "=v"(t) : "s"(0x010101u), "v"(m));
    const unsigned sel = (t << 8) | ((m << 2) ^ 0x04040404u);
    const unsigned esc = __builtin_amdgcn_perm(0u, __builtin_amdgcn_alignbyte(hi[i], lo[i], ps[i]), sel);
    const unsigned ex = ((sh <= 3 ? P[i >> 2] << (3 - sh) : P[i >> 2] >> (sh - 3)) & 0x18181818u) | 0x20202020u;
    const unsigned n = i < 8 ? n0[(i >> 1) & 3] : n1[(i >> 1) & 3];
    const unsigned sm = (i & 1) ? (((n >> 3) & 0x07070707u) | ((n << 1) & 0x80808080u)) : (n & 0x87878787u);
    w[i] = sm | (ex ^ esc);
  }
}

__device__ __forceinline__ void piece_flag(char* myws, int piece, int need, unsigned t, unsigned call, int lane) {
  if (t + 1 == call * (unsigned)need && lane == 0) st_agent(myws + OFF_RFLAG + (size_t)piece * 128, call);
}
__device__ __forceinline__ void piece_count(char* myws, int piece, int need, unsigned call, int lane) {
  unsigned t;
  ticket_issue(t, myws + OFF_READY + (size_t)piece * 128);
  asm volatile("s_waitcnt vmcnt(0)" : "+v"(t) :: "memory");
  piece_flag(myws, piece, need, __builtin_amdgcn_readfirstlane(t), call, lane);
}

constexpr int NIN = (6 - NPUSH) * 8 + 340;
constexpr int NITEM = NIN + 256;
__device__ __forceinline__ void item_describe(int r, int s, int t, unsigned& d0, unsigned& d1, unsigned& d2) {
  const bool same = ((r ^ s) & 4) == 0;
  const int nin = same ? NIN : NIN - 4;
  if (t < nin) {
    constexpr int N1 = (6 - NPUSH) * 8;
    const int ks = x1_index(r, s);
    const int nc = same ? 4 : 3, ns = same ? 2 : 3;
    auto cross = [&](int i) { return partner(r, (i + (!same && i + 3 >= ks ? 2 : 1)) & 3); };
    int blk, h;
    bool isw = false;
    int u = t - N1;
    if (u < 0) { blk = x1_block(r, x1_rot(r, s, NPUSH + (t >> 3))); h = t & 7; }
    else if (u < 64) { isw = true; blk = u >> 4; h = u & 15; }
    else if ((u -= 64) < nc * 8) { blk = cross(u >> 3) * 4 + 3; h = u & 7; }
    else if ((u -= nc * 8) < ns * 6) { const int i = u / 6; blk = same_peer(r, i + (same && i >= ks ? 1 : 0)) * 4 + 1; h = u % 6; }
    else if ((u -= ns * 6) < nc * 2) { blk = cross(u >> 1) * 4 + 1; h = 6 + (u & 1); }
    else if ((u -= nc * 2) < 192) { isw = true; blk = 4 + (u >> 4); h = u & 15; }
    else { u -= 192; blk = r * 4 + (u >> 3); h = u & 7; }
    int piece = 40 + blk, need = 256;
    if (!isw) x_piece(r, blk, h, piece, need);
    unsigned dst, s01, s23;
    chunk_place(isw, blk, 2 * h, s, 0, dst, s01, s23);
    d0 = (unsigned)OFF_S + (isw ? 256 + wblock(r, blk) * 16 + h : blk * 8 + h) * (unsigned)PAIR;
    d1 = (isw ? 32 + blk : blk) * 128 | (2 * h) << 16;
    d2 = dst >> 10 | (unsigned)isw << 17 | piece << 18 | (need >> 2) << 24;
  } else {
    const int u = t - nin;
    int n, tl, w;
    if (same) {
      const int i = (r - s - 1) & 3;
      if (u < 64) { n = (u >> 3) & 3; tl = u < 32 ? i : 9 + i; w = u & 7; }
      else if (u < 208) { const int x = (u - 64) % 12; n = 4 + (u - 64) / 12; tl = x < 8 ? i : 9 + i; w = x & 7; }
      else { n = 4 + ((u - 208) >> 2); tl = 9 + i; w = 4 + (u & 3); }
    } else {
      const int qr = (r < 4 ? s - r : r - s) & 3;
      int qq, c = 0, kind;
      if (u < 24) { qq = u >> 3; kind = 0; w = u & 7; }
      else if (u < 60) { const int x = (u - 24) % 12; qq = (u - 24) / 12; kind = x < 8 ? 1 : 2; w = x < 8 ? x : x - 4; }
      else {
        const int b = u < 204 ? (u - 60) >> 4 : (u - 204) >> 2;
        c = (1 + b / 3) & 3; qq = b % 3;
        kind = u < 204 ? u & 1 : 2;
        w = u < 204 ? ((u - 60) >> 1) & 7 : 4 + (u & 3);
      }
      const int q = qq + (qq >= qr ? 1 : 0);
      n = c * 4 + q;
      tl = 3 + 3 * kind + ((qr - q - 1) & 3);
    }
    const bool half = tl >= 9 && w >= 4;
    const int hi = half && !same;
    d0 = (unsigned)OFF_O + ((n * 12 + tl) * 8 + w) * (unsigned)SLOT + hi * 8192;
    d1 = ((48 + n * 3 + (tl < 6 ? 0 : (tl < 9 ? 1 : 2))) * 128 + w * 16) | (half ? 2u : 1u) << 24;
    d2 = ((tile_sub(tl) * 128 + (w >> 2) * 64) * NFULL + wblock(s, n) * 256 + (w & 3) * 64) | (hi * 2) << 24;
  }
}

constexpr int RING = 4;
constexpr int LSLOT = 16384;
static_assert(2 * RING * LSLOT + 6 * SCR + NITEM * 12 <= 3 * STAGE, "LDS");

__device__ __forceinline__ unsigned lds_peek(LAS unsigned* p) {
  unsigned v;
  asm volatile("ds_read_b32 %0, %1\n\ts_waitcnt lgkmcnt(0)" : "=&v"(v) : "v"(p) : "memory");
  return __builtin_amdgcn_readfirstlane(v);
}
__device__ __forceinline__ u64 uni64(u64 v) {
  return ((u64)(unsigned)__builtin_amdgcn_readfirstlane((unsigned)(v >> 32)) << 32) | (u64)(unsigned)__builtin_amdgcn_readfirstlane((unsigned)v);
}

__global__ void __launch_bounds__(512, 2) fused_gemm_rs(Params p) {
  __shared__ __attribute__((aligned(16))) char smem[LDS_BYTES];
  const int tid = threadIdx.x;
  const int lane = tid & 63;
  const int wave = __builtin_amdgcn_readfirstlane(tid >> 6);
  const int rank = p.rank;
  const int j = blockIdx.x & 7;
  const int grp = blockIdx.x >> 3;
  char* const myws = p.myws;
  char* const pws = p.link[grp & 7];
  asm volatile("" :: "s"(p.x), "s"(p.w), "s"(myws), "s"(pws), "s"(rank));
  volatile LAS unsigned* sctl = (volatile LAS unsigned*)(LAS char*)(smem + 3 * STAGE);

  const int peer = (rank + 1 + grp) & 7;
  unsigned w0[16], wp[NPUSH / 2][16];
  const int pw = j * 4 + wave;
  const int ew = (wave >> 1) * 400 + (blockIdx.x - 56) * 2 + (wave & 1);
  auto enc_id = [&](int n, bool& isw, int& a, int& b, int& e) {
    isw = (n >= 256 && n < 768) || n >= 1024;
    if (isw) { const int i = (n < 768 ? n - 256 : n - 512) >> 5; a = i & 3; b = i >> 2; e = n & 31; }
    else { a = ((n >> 4) & 7) * 4 + ((n & 128) ? 0 : 2) + (n >= 768 ? 1 : 0); b = 0; e = n & 15; }
  };
  auto enc_load = [&](int n, unsigned (&w)[16]) {
    bool isw; int a, b, e;
    enc_id(n, isw, a, b, e);
    if (isw) w_load(p, wblock(a, b) * 256, e, lane, w); else x_load(p, a, e, lane, w);
  };
  LAS unsigned* const table = (LAS unsigned*)((LAS char*)smem + 2 * RING * LSLOT + 6 * SCR);
  unsigned cticket;
  ticket_issue(cticket, myws + OFF_CALL + (size_t)blockIdx.x * 128 + wave * 16);
  STAMP(0);
  if (grp >= 7) enc_load(ew, w0);
  else if (wave < 4) {
#pragma unroll
    for (int it = 0; it < NPUSH / 2; ++it) x_load(p, x1_block(peer, x1_rot(peer, rank, (pw >> 4) + 2 * it)), pw & 15, lane, wp[it]);
  }
  if (grp < 7) {
    if (tid >= 15 && tid < 48) sctl[tid] = 0;
    asm volatile("s_waitcnt lgkmcnt(0)" ::: "memory");
    __builtin_amdgcn_s_barrier();
    if (wave >= 4) {
      for (int t = tid - 256; t < NITEM; t += 256) {
        unsigned d0, d1, d2;
        item_describe(rank, peer, t, d0, d1, d2);
        table[t * 3] = d0; table[t * 3 + 1] = d1; table[t * 3 + 2] = d2;
      }
      asm volatile("s_waitcnt lgkmcnt(0)" ::: "memory");
      if (lane == 0) __hip_atomic_fetch_add((LAS unsigned*)(sctl + 15), 1u, __ATOMIC_RELAXED, __HIP_MEMORY_SCOPE_WORKGROUP);
    }
  }
  if (grp < 7 && wave < 4) WAIT_VM(NPUSH * 2); else WAIT_VM(0);
  asm volatile("" : "+v"(cticket) :: "memory");
  const unsigned call = (unsigned)__builtin_amdgcn_readfirstlane(cticket) + 1;
  const unsigned par = call & 1;
  if (grp >= 7) STAMP(6);
  void* const tmo = myws + OFF_TMO;

  const bool dbg_skip = false;
  if (dbg_skip) {
  } else if (grp >= 7) {
    LAS char* const scr = (LAS char*)smem + wave * SCR;
    if (wave < 2) __builtin_amdgcn_s_setprio(3); else if (wave < 4) __builtin_amdgcn_s_setprio(2); else if (wave < 6) __builtin_amdgcn_s_setprio(1);
    for (int n = ew; n < 2560; n += 1600) {
      bool isw; int a, b, e;
      enc_id(n, isw, a, b, e);
      if (n != ew) enc_load(n, w0);
      if (isw) w_transpose(w0);
      const int other = isw ? partner(a, b & 3) : 0;
      const bool low = (a & 3) == 0 || ((a & 3) == 1 && e < 12);
      const bool mine = isw ? (a == rank || other == rank) : ((a >> 2) == rank || low == ((((a >> 2) ^ rank) & 4) == 0));
      const int sp = isw ? 32 + wblock(a, b) : a;
      if (isw || (a >> 2) != rank) {
        const int slot = isw ? 512 + (sp - 32) * 32 + e : a * 16 + e;
        const bool raw = chunk_pack(w0, myws + OFF_S + ((size_t)par * 1280 + (slot >> 1)) * PAIR, slot & 1, scr, lane);
        WAIT_VM(0);
        char* const agg = myws + OFF_EAGG + (size_t)sp * 256;
        const u64 mine64 = (1ull << 32) | (raw ? 1u << e : 0u);
        u64 tot = 0;
        if (lane == 0) tot = __hip_atomic_fetch_add((GAS u64*)(agg + par * 128), mine64, __ATOMIC_RELAXED, __HIP_MEMORY_SCOPE_AGENT);
        tot = uni64(tot) + mine64;
        if ((unsigned)(tot >> 32) == (isw ? 32u : 16u)) {
          if (lane < NR && lane != rank && (!isw || lane == a || lane == other))
            __hip_atomic_store((GAS u64*)(p.ws[lane] + OFF_FLAG + (size_t)((par * 8 + rank) * NFL + (isw ? 32 + b : a)) * 128),
                               ((u64)call << 32) | (unsigned)tot, __ATOMIC_RELAXED, __HIP_MEMORY_SCOPE_SYSTEM);
          if (lane == 0) __hip_atomic_store((GAS u64*)(agg + (par ^ 1) * 128), 0ull, __ATOMIC_RELAXED, __HIP_MEMORY_SCOPE_AGENT);
        }
      }
      if (mine || isw) {
        unsigned dst, s01, s23;
        chunk_place(isw, isw ? b : a, e, rank, par, dst, s01, s23);
        if (!mine) dst += (unsigned)(OFF_B2 - OFF_B) + ((sp - 32) * 4 - (b * 32 + rank * 4)) * 32768u;
        chunk_store(myws, dst, s01, s23, lane, w0);
        int piece = mine ? 40 + b : 32 + sp, need = mine ? 256 : 32;
        if (!isw) x_piece(rank, a, e >> 1, piece, need);
        WAIT_VM(0);
        piece_count(myws, piece, need, call, lane);
        if (!isw && (a >> 2) == rank) piece_count(myws, 56 + (a & 3), 16, call, lane);
      }
    }
    __builtin_amdgcn_s_setprio(0);
  } else if (wave < 4) {
    char* const dws = pws;
    const __amdgpu_buffer_rsrc_t drs = __builtin_amdgcn_make_buffer_rsrc(dws, 0, (int)WS_NEED, 0x00020000);
    STAMP(6);
#pragma unroll
    for (int it = 0; it < NPUSH / 2; ++it) {
      unsigned dst, s01, s23;
      chunk_place(false, x1_block(peer, x1_rot(peer, rank, (pw >> 4) + 2 * it)), pw & 15, rank, par, dst, s01, s23);
      const unsigned (&w)[16] = wp[it];
#pragma unroll
      for (int b = 0; b < 4; ++b) {
        __builtin_amdgcn_raw_buffer_store_b128(u32x4{w[4 * b], w[4 * b + 1], w[4 * b + 2], w[4 * b + 3]}, drs, lane * 16,
                                               dst + (b & 1) * s01 + (b >> 1) * s23, 17);
        if (it > 0 || b >= PUSH_WIN - 1) WAIT_VM(PUSH_WIN - 1);
      }
    }
    WAIT_VM(0);
    STAMP(7);
    unsigned ag[NPUSH / 2];
#pragma unroll
    for (int it = 0; it < NPUSH / 2; ++it) ticket_issue(ag[it], myws + OFF_PAGG + (size_t)(peer * 8 + (pw >> 4) + 2 * it) * 64);
#pragma unroll
    for (int it = 0; it < NPUSH / 2; ++it) {
      asm volatile("s_waitcnt vmcnt(0)" : "+v"(ag[it]) :: "memory");
      if ((unsigned)__builtin_amdgcn_readfirstlane(ag[it]) + 1 == call * 16u && lane == 0)
        st_sys(dws + OFF_PUSHF + (size_t)x1_rot(peer, rank, (pw >> 4) + 2 * it) * 128 + rank * 4, call);
    }
  }
  if (blockIdx.x == 64 && tid < NR)
    st_agent(myws + OFF_QCNT + (size_t)((par ^ 1) * 8 + tid) * 128, 0u);
  STAMP(1);
  const float scale = p.sx[0] * p.sw[0];

  if (grp >= 7) {
    const int wm = wave >> 2, G = wave & 3;
    const int npass = grp < 14 ? 2 : 1;
    for (int pass = grp >= 19 && grp < 31 ? -2 : 0; pass < npass; ++pass) {
      const bool part = pass < 0;
      const int pos = grp - 7 + 25 * pass;
      const int n = pos < 24 ? j + (pos >= 12 ? 8 : 0) : j + 8 * (pos & 1);
      const int tl = part ? 12 + ((grp - 19) & 1) * 2 + (pass & 1) : (pos < 12 ? pos : (pos < 24 ? pos - 12 : 12 + ((pos - 24) >> 1)));
      const int sub = tile_sub(tl);
      const int fb = j * 6 + ((grp - 19) >> 1);
      const int wb = part ? wblock(same_peer(rank, fb >> 4), fb & 15) : wblock(rank, n);
      const int cp = partner(rank, (n + 1 + tl % 3) & 3);
      const int dA = tl >= 12 ? rank : ((tl < 3 || tl >= 9) ? same_peer(rank, tl % 3) : cp);
      const int dB = (tl >= 9 && tl < 12) ? cp : dA;
      const int pk = tl < 6 ? x1_index(rank, dA) : -1;
      if (pass == -2 && wave == 0)
        for (int i = (grp - 19) * PART_STAGGER; i > 0; --i) __builtin_amdgcn_s_sleep(8);
      if (!dbg_skip && (tid < 2 || (tid == 2 && sub == 1 && !part) ||
                        (tid >= 3 && tid < 11 && pk >= 0 && tid - 3 != rank && (unsigned)((pk + 6 - x1_index(rank, (tid - 3) & 7)) % 7) < NPUSH)))
        wait_ge(myws + (tid < 3 ? OFF_RFLAG + (size_t)(tid == 0 ? (part ? 64 + wb : 40 + n) : (tid == 1 ? (part ? 56 + sub : dA * 4 + sub) : 32 + dB)) * 128
                                : OFF_PUSHF + (size_t)pk * 128 + (tid - 3) * 4), call, tmo, call);
      __syncthreads();
      if (!part) STAMP(2 + 3 * pass);
      unsigned tmov;
      asm volatile("global_load_dword %0, %1, %2 sc1" : "=&v"(tmov) : "v"(0u), "s"(tmo) : "memory");
      f32x4 acc[4][4];
      {
        const __amdgpu_buffer_rsrc_t rs = __builtin_amdgcn_make_buffer_rsrc(myws, 0, (int)WS_NEED, 0x00020000);
        const int own = wave < 6 ? dA : dB;
        const int nk = part ? 4 : 32;
        const unsigned ga = (unsigned)OFF_A + ((par * 32 + own * 4 + sub) * 32 + (part ? rank * 4 : 0)) * 16384u + wave * 2048;
        const unsigned gb = (part ? (unsigned)OFF_B2 + wb * (4u * 32768u) : (unsigned)OFF_B + n * (32u * 32768u)) + wave * 4096;
        const int kskip = own == rank ? -1 : own;
        const unsigned l16 = lane * 16;
        auto dma = [&](int ks, int buf, int i) {
          if (i < 2)
            __builtin_amdgcn_raw_ptr_buffer_load_lds(__builtin_amdgcn_make_buffer_rsrc(myws, 0, (ks >> 2) == kskip ? 0 : (int)WS_NEED, 0x00020000),
                                                     (LAS void*)((LAS char*)smem + buf * STAGE + wave * 2048 + i * 1024), 16,
                                                     l16, ga + ks * 16384 + i * 1024, 0, 17);
          else
            __builtin_amdgcn_raw_ptr_buffer_load_lds(rs, (LAS void*)((LAS char*)smem + buf * STAGE + 16384 + wave * 4096 + (i - 2) * 1024), 16,
                                                     l16, gb + ks * 32768 + (i - 2) * 1024, 0, 17);
        };
        const LAS char* const la = (const LAS char*)smem + wm * 8192 + lane * 16;
        const LAS char* const lb = (const LAS char*)smem + 16384 + G * 8192 + lane * 16;
        auto frag = [&](const LAS char* q) -> i32x8 {
          const i32x4 lo = *(const LAS i32x4*)q, hi = *(const LAS i32x4*)(q + 1024);
          return __builtin_shufflevector(lo, hi, 0, 1, 2, 3, 4, 5, 6, 7);
        };
#define MMA(m, t, CA, CB) acc[m][t] = __builtin_amdgcn_mfma_scale_f32_16x16x128_f8f6f4(CA[m], CB[t], acc[m][t], 0, 0, 0, 150, 0, 127)
#define GEMM_STEP(KS, VM, CA, CB, NA, NB, DO_STAGE) do { \
          WAIT_VM(VM); \
          asm volatile("s_waitcnt lgkmcnt(0)" ::: "memory"); \
          __builtin_amdgcn_s_barrier(); \
          __builtin_amdgcn_sched_barrier(0); \
          const int nbuf = buf == 2 ? 0 : buf + 1; \
          _Pragma("unroll") for (int g = 0; g < 16; ++g) { \
            MMA(g >> 2, g & 3, CA, CB); \
            __builtin_amdgcn_sched_barrier(0); \
            if (g < 8) { \
              if (g & 1) NB[g >> 1] = frag(lb + nbuf * STAGE + (g >> 1) * 2048); \
              else NA[g >> 1] = frag(la + nbuf * STAGE + (g >> 1) * 2048); \
            } else if (DO_STAGE && g < 14) dma((KS) + 3, buf, g - 8); \
            __builtin_amdgcn_sched_barrier(0); \
          } \
          buf = nbuf; } while (0)
        i32x8 a0[4], b0[4], a1[4], b1[4];
#pragma unroll
        for (int i = 0; i < 18; ++i) dma(i / 6, i / 6, i % 6);
        asm volatile("s_waitcnt vmcnt(12)" : "+v"(tmov) :: "memory");
        __builtin_amdgcn_s_barrier();
        const float acc0 = tmov == call ? __builtin_nanf("") : 0.0f;
#pragma unroll
        for (int g = 0; g < 16; ++g) acc[g >> 2][g & 3] = f32x4{acc0, acc0, acc0, acc0};
#pragma unroll
        for (int t = 0; t < 4; ++t) { a0[t] = frag(la + t * 2048); b0[t] = frag(lb + t * 2048); }
        int buf = 0;
        for (int ks = 0; ks < nk - 4; ks += 2) {
          GEMM_STEP(ks, 6, a0, b0, a1, b1, true);
          GEMM_STEP(ks + 1, 6, a1, b1, a0, b0, true);
        }
#define PIN_ACC() _Pragma("unroll") for (int g = 0; g < 16; ++g) asm volatile("" : "+v"(acc[g >> 2][g & 3]))
        GEMM_STEP(nk - 4, 6, a0, b0, a1, b1, true);
        PIN_ACC();
        GEMM_STEP(nk - 3, 6, a1, b1, a0, b0, false);
        PIN_ACC();
        GEMM_STEP(nk - 2, 0, a0, b0, a1, b1, false);
        PIN_ACC();
#pragma unroll
        for (int g = 0; g < 16; ++g) MMA(g >> 2, g & 3, a1, b1);
        PIN_ACC();
      }
      if (!part) STAMP(3 + 3 * pass);

      int ln = lane;
      asm volatile("" : "+v"(ln));
      const int q = ln >> 4, c = ln & 15;
      if (tl >= 12) {
        const float* dst = p.out + (size_t)(sub * 128 + wm * 64) * NFULL + wb * 256 + G * 64;
        const unsigned vo = (4 * q * NFULL + 4 * c) * 4;
        if (!part) {
#pragma unroll
          for (int g = 0; g < 64; ++g) acc[g >> 4][(g >> 2) & 3][g & 3] = silu_dq(acc[g >> 4][(g >> 2) & 3][g & 3], scale);
        }
#pragma unroll
        for (int m = 0; m < 4; ++m)
#pragma unroll
          for (int r = 0; r < 4; ++r) {
            const f32x4 v = {acc[m][0][r], acc[m][1][r], acc[m][2][r], acc[m][3][r]};
            st16_out(dst + (size_t)(m * 16 + r) * NFULL, vo, __builtin_bit_cast(u32x4, v));
          }
        if (part) {
          WAIT_VM(0);
          __syncthreads();
          if (wave == 0) piece_count(myws, 60, 192, call, lane);
        }
      } else {
        char* slot = myws + OFF_O + ((size_t)(n * 12 + tl) * 8 + wave) * SLOT;
        const bool mixed = dA != dB && wm == 1;
        char* const meta = slot + (mixed ? 6144 : 12288);
        const int g2 = mixed ? 8192 : 6144;
        LAS char* const scr = (LAS char*)smem + wave * 2048;
        if (wave < 2) __builtin_amdgcn_s_setprio(3); else if (wave < 4) __builtin_amdgcn_s_setprio(2); else if (wave < 6) __builtin_amdgcn_s_setprio(1);
        unsigned gmask = 0;
        unsigned nexc = 0;
        auto emit = [&](u64 mask, bool on, unsigned pos, unsigned val) {
          const unsigned idx = nexc + __builtin_amdgcn_mbcnt_hi((unsigned)(mask >> 32), __builtin_amdgcn_mbcnt_lo((unsigned)mask, 0u));
          if (on && idx < (unsigned)EXC_CAP) {
            __hip_atomic_store((GAS u64*)(meta + idx * 8), ((u64)pos << 32) | val, __ATOMIC_RELAXED, __HIP_MEMORY_SCOPE_SYSTEM);
            if (mixed) __hip_atomic_store((GAS u64*)(meta + 8192 + idx * 8), ((u64)pos << 32) | val, __ATOMIC_RELAXED, __HIP_MEMORY_SCOPE_SYSTEM);
          }
          nexc += __builtin_popcountll(mask);
        };
#pragma unroll
        for (int h = 0; h < 2; ++h) {
          u32x4 v[2][3];
          unsigned m30 = 0, m7 = 0, m6 = 0;
#pragma unroll
          for (int g = 0; g < 8; ++g) {
            const int m = 2 * h + (g >> 2), t = g & 3;
            unsigned I[4];
            u64 mis = 0;
#pragma unroll
            for (int r = 0; r < 4; ++r) {
              I[r] = to_i32(acc[m][t][r]);
              mis |= __builtin_amdgcn_ballot_w64(too_big(acc[m][t][r]));
            }
            asm volatile("s_cmp_lg_u64 %1, 0\n\ts_addc_u32 %0, %0, %0" : "+s"(gmask) : "s"(mis) : "scc");
            const unsigned a01 = __builtin_amdgcn_perm(I[1], I[0], 0x07030400u), a23 = __builtin_amdgcn_perm(I[3], I[2], 0x07030400u);
            const unsigned L = __builtin_amdgcn_perm(a23, a01, 0x05040100u), H = __builtin_amdgcn_perm(a23, a01, 0x07060302u);
            m30 = (m30 >> 1) | ((H ^ (H << 1)) & 0x80808080u);
            m7 = (m7 >> 1) | (L & 0x80808080u);
            m6 = (m6 >> 1) | ((L << 1) & 0x80808080u);
#pragma unroll
            for (int r = 0; r < 3; ++r) v[g >> 2][r][t] = __builtin_amdgcn_perm(I[3], I[r], 0x03020105u + r);
            const bool fine = (L & 0x3F3F3F3Fu) != 0;
            const u64 fm = __builtin_amdgcn_ballot_w64(fine);
            if (__builtin_expect(fm != 0, 0)) emit(fm, fine, 0x1000 + m * 1024 + ln * 16 + t, L);
          }
          const unsigned cnt = __builtin_popcount(m30) + __builtin_popcount(m7) + __builtin_popcount(m6);
          const unsigned incl = wave_scan_incl(cnt);
          if ((unsigned)__builtin_amdgcn_readlane(incl, 63) > 256u || __builtin_amdgcn_ballot_w64(cnt > 15u) != 0) nexc = 99;
          else {
            LAS char* q = scr + (incl - cnt);
            for (u64 ev = ((u64)m7 << 32) | m30; ev != 0; ev &= ev - 1) *q++ = (char)__builtin_ctzll(ev);
            for (; m6 != 0; m6 &= m6 - 1) *q++ = (char)(64 + __builtin_ctz(m6));
          }
          asm volatile("s_waitcnt lgkmcnt(0)" ::: "memory");
          const unsigned P = (*(const LAS unsigned*)(scr + ln * 4) & 0x7F7F7F7Fu) | ((cnt * 0x10204080u) & 0x80808080u);
#pragma unroll
          for (int g = 0; g < 8; ++g)
#pragma unroll
            for (int r = 0; r < 3; ++r) {
              const unsigned k = r == 2 ? 0x40000040u : 0x40000000u;
              const unsigned x = v[g >> 2][r][g & 3];
              v[g >> 2][r][g & 3] = (x & ~k) | (__builtin_amdgcn_alignbit(P, P, (r * 8 + g + 2) & 31) & k);
            }
#pragma unroll
          for (int i = 0; i < 6; ++i) st16_sys(slot + h * g2 + i * 1024, ln * 16, v[i / 3][i % 3]);
        }
        while (gmask != 0 && nexc <= (unsigned)EXC_CAP) {
          const int g = 15 - __builtin_ctz(gmask);
          gmask &= gmask - 1;
          f32x4 v = acc[0][0];
#pragma unroll
          for (int gg = 1; gg < 16; ++gg) v = g == gg ? acc[gg >> 2][gg & 3] : v;
#pragma unroll
          for (int r = 0; r < 4; ++r) {
            const float a = v[r];
            emit(__builtin_amdgcn_ballot_w64(too_big(a)), too_big(a), (g >> 2) * 1024 + ln * 16 + r * 4 + (g & 3), as_u(a));
          }
        }
        if (mixed && n >= RAW_FROM) nexc = 99;
        if (nexc > (unsigned)EXC_CAP) {
          nexc = 15;
          WAIT_VM(0);
#pragma unroll
          for (int m = 0; m < 4; ++m)
#pragma unroll
            for (int r = 0; r < 4; ++r) {
              f32x4 v = {acc[m][0][r], acc[m][1][r], acc[m][2][r], acc[m][3][r]};
              st16_sys(slot + (m * 4 + r) * 1024, ln * 16, __builtin_bit_cast(u32x4, v));
            }
        }
        WAIT_VM(0);
        const size_t fo = OFF_FLAG + (size_t)((par * 8 + rank) * NFL + 48 + n * 3 + (tl < 6 ? 0 : (tl < 9 ? 1 : 2))) * 128 + wave * 16;
        if (lane == 0) {
          __hip_atomic_store((GAS u64*)(p.ws[dA & 7] + fo), ((u64)call << 32) | nexc, __ATOMIC_RELAXED, __HIP_MEMORY_SCOPE_SYSTEM);
          if (mixed) __hip_atomic_store((GAS u64*)(p.ws[dB & 7] + fo), ((u64)call << 32) | nexc, __ATOMIC_RELAXED, __HIP_MEMORY_SCOPE_SYSTEM);
        }
        __builtin_amdgcn_s_setprio(0);
      }
      if (!part) STAMP(4 + 3 * pass); else STAMP(1);
    }
  } else {
    const int s = peer;
    const int ring = wave >= 4 ? (wave & 1) : (wave >> 1);
    LAS char* const rbase = (LAS char*)smem + ring * (RING * LSLOT);
    volatile LAS unsigned* const full = sctl + 16 + ring * 16;
    volatile LAS unsigned* const info = full + 4;
    LAS unsigned* const done = (LAS unsigned*)(full + 8);
    volatile LAS unsigned* const claim = full + 12;
    const unsigned nin = ((rank ^ s) & 4) == 0 ? NIN : NIN - 4;
    const unsigned nitem = ((rank ^ s) & 4) == 0 ? NITEM : NITEM - 20;
    if (wave == 4 || wave == 5) {
      const char* sws = pws;
      const void* tick = myws + OFF_QCNT + (size_t)(par * 8 + s) * 128;
      const char* const fbase = myws + OFF_FLAG + (size_t)(par * 8 + s) * NFL * 128;
      auto describe = [&](unsigned t, unsigned& off, unsigned& d1) {
        const unsigned tt = t < nitem ? t : 0;
        off = lds_peek(table + tt * 3) + (tt < nin ? par * 1280 * (unsigned)PAIR : 0u);
        d1 = lds_peek(table + tt * 3 + 1);
      };
      unsigned item = j * 2 + ring, inext = item + 16;
      unsigned off, d1;
      while (lds_peek((LAS unsigned*)(sctl + 15)) < 4u) __builtin_amdgcn_s_sleep(1);
      describe(item, off, d1);
      u64 fpre = 0;
      bool sfree = true;
      int pend = -1;
      unsigned seq = 0;
      const unsigned l16 = lane * 16;
#define PUBLISH() do { if (pend >= 0) { if (lane == 0) full[pend] = seq; pend = -1; } } while (0)
      for (; seq <= (unsigned)NITEM; ) {
        const int sl = seq % RING;
        const bool stop = item >= nitem;
        bool bad = false;
        if (!sfree) sfree = (int)(lds_peek(done + sl) - 3 * (seq / RING)) >= 0;
        if (stop || (int)((unsigned)(fpre >> 32) - call) < 0 || !sfree) {
          WAIT_VM(0);
          PUBLISH();
          u64 t0 = __builtin_amdgcn_s_memrealtime();
          if (!stop) {
            for (unsigned n = 1;; ++n) {
              fpre = uni64(ld_sys64(fbase + (d1 & 0xFFFFu)));
              if ((int)((unsigned)(fpre >> 32) - call) >= 0) break;
              if ((n & 15u) == 0 && (ld_agent(tmo) == call || __builtin_amdgcn_s_memrealtime() - t0 > DEADLINE)) { st_agent(tmo, call); bad = true; break; }
              __builtin_amdgcn_s_sleep(1);
            }
          }
          while ((int)(lds_peek(done + sl) - 3 * (seq / RING)) < 0) {
            if (__builtin_amdgcn_s_memrealtime() - t0 > 2 * DEADLINE) { st_agent(tmo, call); bad = true; break; }
            __builtin_amdgcn_s_sleep(1);
          }
          bad = __builtin_amdgcn_readfirstlane((int)bad) != 0;
          if (stop) { if (lane == 0) info[sl] = 1u << 15 | (seq + 1) << 17; break; }
          if (seq == 0) STAMP2(2);
        }
        if (item < nin && inext >= nin) STAMP2(5);
        const bool isin = item < nin;
        const unsigned bits = ((unsigned)fpre >> ((d1 >> 16) & 31u)) & (isin ? 3u : 15u);
        const int hs = (d1 >> 25) & 1;
        int len = isin ? (bits == 0 ? 7040 : 9216) : (bits == 15u ? 16384 >> hs : (12288 >> hs) + (bits != 0 ? 128 : 0));
        const int nld = (len + 1023) >> 10;
        if (bad) len = 0;
        const __amdgpu_buffer_rsrc_t rs = __builtin_amdgcn_make_buffer_rsrc((void*)(sws + off), 0, len, 0x00020000);
        LAS char* dstl = rbase + sl * LSLOT;
        unsigned tk; u64 fn;
#define PULL_ITEM(W) do { \
          __builtin_amdgcn_raw_ptr_buffer_load_lds(rs, (LAS void*)dstl, 16, l16, 0, 0, 17); \
          if (lane == 0) claim[sl] = 0; \
          if (lane == 0) info[sl] = item | bits << 10 | (unsigned)bad << 14 | (unsigned)(item + SPLIT_ITEMS >= nitem) << 16 | (seq + 1) << 17; \
          describe(inext, off, d1); \
          ticket_issue(tk, tick); \
          asm volatile("global_load_dwordx2 %0, %1, %2 sc0 sc1" : "=&v"(fn) : "v"(0u), "s"(fbase + (d1 & 0xFFFFu)) : "memory"); \
          WAIT_VM((W) + 1); \
          _Pragma("unroll") for (int i = 1; i < (W); ++i) { \
            __builtin_amdgcn_raw_ptr_buffer_load_lds(rs, (LAS void*)(dstl + i * 1024), 16, l16 + i * 1024, 0, 0, 17); \
            WAIT_VM((W) + 1); \
            if (i == (W) - 2) {                              \
              PUBLISH(); \
              sfree = (int)(lds_peek(done + (seq + 1) % RING) - 3 * ((seq + 1) / RING)) >= 0; \
            } \
          } \
          for (int i = (W); i < nld; ++i) { \
            __builtin_amdgcn_raw_ptr_buffer_load_lds(rs, (LAS void*)(dstl + i * 1024), 16, l16 + i * 1024, 0, 0, 17); \
            if (item + TAIL_ITEMS < nitem || i == nld - 1) WAIT_VM((W) - 1); \
          } \
          asm volatile("" : "+v"(tk), "+v"(fn) :: "memory");           \
          inext2 = 32 + (unsigned)__builtin_amdgcn_readfirstlane(tk); \
          fpre = uni64(fn); } while (0)
        unsigned inext2;
        if (isin) PULL_ITEM(PULL_WIN_IN); else PULL_ITEM(PULL_WIN);
        ++seq;
        pend = sl;
        item = inext;
        inext = inext2;
      }
    } else {
      const int role = wave >= 6 ? 2 : (wave & 1);
      LAS char* const scr = (LAS char*)smem + 2 * RING * LSLOT + (ring * 3 + role) * SCR;
      const unsigned vo = ((lane >> 4) * 4 * NFULL + (lane & 15) * 4) * 4;
      int pp = NOPIECE, pneed = 0;
      bool parts = false;
      bool lost = false;
      auto await = [&](volatile LAS unsigned* word, unsigned shift, unsigned want) -> unsigned {
        u64 t0 = 0;
        for (unsigned n = 1;; ++n) {
          const unsigned v = lds_peek((LAS unsigned*)word);
          if (v >> shift == want) return v;
          if (pp != NOPIECE) { WAIT_VM(0); piece_count(myws, pp, pneed, call, lane); pp = NOPIECE; continue; }
          if ((n & 1023u) == 0) {
            const u64 t = __builtin_amdgcn_s_memrealtime();
            if (t0 == 0) t0 = t;
            if (t - t0 > 4 * DEADLINE) { lost = true; return 0; }
          }
          __builtin_amdgcn_s_sleep(1);
        }
      };
      const __amdgpu_buffer_rsrc_t ors = __builtin_amdgcn_make_buffer_rsrc(p.out, 0, 512 * NFULL * 4, 0x00020000);
      for (unsigned seq = 0; seq <= (unsigned)NITEM && !lost; ) {
        const int sl = seq % RING;
        const unsigned inf = await(info + sl, 17, seq + 1);
        if (lost || ((inf >> 15) & 1u)) break;
        const unsigned item = inf & 1023u, bits = (inf >> 10) & 15u;
        bool bad = (inf >> 14) & 1u;
        unsigned cl = 0;
        if (lane == 0) cl = __hip_atomic_fetch_add((LAS unsigned*)(claim + sl), 1u, __ATOMIC_RELAXED, __HIP_MEMORY_SCOPE_WORKGROUP);
        const unsigned d1 = table[item * 3 + 1], d2v = table[item * 3 + 2];
        const int c = __builtin_amdgcn_readfirstlane(cl);
        const unsigned d2 = __builtin_amdgcn_readfirstlane(d2v);
        const int ng = 4 >> ((__builtin_amdgcn_readfirstlane(d1) >> 25) & 1u), mlo = d2 >> 24;
        const bool split = ng == 2 && ((inf >> 16) & 1u);
        const int nparts = ng == 4 || split ? 2 : 1;
        const bool through = c + 1 >= nparts;
        if (through) ++seq;
        if (c >= nparts) {
          if (lane == 0) __hip_atomic_fetch_add(done + sl, 1u, __ATOMIC_RELAXED, __HIP_MEMORY_SCOPE_WORKGROUP);
          continue;
        }
        const unsigned want = (inf >> 17);
        if (item < nin) {
          await(full + sl, 0, want);
          if (lost) break;
          unsigned w[16], t;
          chunk_unpack(rbase + sl * LSLOT, c, ((bits >> c) & 1u) || bad, scr, lane, w, [&]() {
            WAIT_VM(0);
            ticket_issue(t, myws + (pp != NOPIECE ? OFF_READY + (size_t)pp * 128 : OFF_CALL + (size_t)blockIdx.x * 128 + wave * 16 + 8));
          });
          asm volatile("s_waitcnt lgkmcnt(0)" ::: "memory");
          if (through && lane == 0) __hip_atomic_fetch_add(done + sl, 1u, __ATOMIC_RELAXED, __HIP_MEMORY_SCOPE_WORKGROUP);
          const bool isw = (d2 >> 17) & 1u;
          chunk_store(myws, ((d2 & 0x1FFFFu) << 10) + (isw ? c * 8192u : c * 32768u + par * (16u << 20)),
                      isw ? 2048 : 1024, isw ? 4096 : 16384, lane, w);
          asm volatile("s_waitcnt vmcnt(4)" : "+v"(t) :: "memory");
          if (pp != NOPIECE) piece_flag(myws, pp, pneed, __builtin_amdgcn_readfirstlane(t), call, lane);
          pp = (d2 >> 18) & 63u;
          pneed = (d2 >> 24) * 4;
          continue;
        }
        if (pp != NOPIECE) { WAIT_VM(0); piece_count(myws, pp, pneed, call, lane); pp = NOPIECE; }
        const int kb = split ? 0 : 2 * c;
        const int ka = split ? c : 0, kz = split ? c + 1 : 2;
        if (!parts) {
          if (lane == 0) wait_ge(myws + OFF_RFLAG + 60 * 128, call, tmo, call);
          parts = (unsigned)__builtin_amdgcn_readfirstlane(ld_agent(tmo)) != call;
          bad = bad || !parts;
        }
        u32x4 pa[2][4];
#pragma unroll
        for (int h = 0; h < 2; ++h)
#pragma unroll
          for (int r = 0; r < 4; ++r)
            pa[h][r] = __builtin_amdgcn_raw_buffer_load_b128(ors, ka + h < kz ? vo + ((d2 & 0x7FFFFFu) + ((mlo + kb + ka + h) * 16 + r) * NFULL) * 4 : 0x80000000u, 0, 17);
        await(full + sl, 0, want);
        bad = bad || lost;
        const unsigned nexc = bad ? 15u : bits;
        const float sc = bad ? __builtin_nanf("") : scale;
        float* tbase = p.out + (d2 & 0x7FFFFFu);
        const LAS char* sb = rbase + sl * LSLOT;
        u64 ent = *(const LAS u64*)(sb + ng * 3072 + (lane & 15) * 8);
        u32x4 cw[4], ot[4];
        const int gs = nexc == 15u ? 4096 : 3072;
#pragma unroll
        for (int r = 0; r < 4; ++r) {
          cw[r] = *(const LAS u32x4*)(sb + lane * 16 + (kb + ka) * gs + r * 1024);
          ot[r] = *(const LAS u32x4*)(sb + lane * 16 + (kb + (ka ^ 1)) * gs + r * 1024);
        }
        asm volatile("s_waitcnt lgkmcnt(0)" : "+v"(cw[0]), "+v"(cw[1]), "+v"(cw[2]), "+v"(cw[3]), "+v"(ot[0]), "+v"(ot[1]), "+v"(ot[2]), "+v"(ot[3]), "+v"(ent) :: "memory");
        if (through && lane == 0) __hip_atomic_fetch_add(done + sl, 1u, __ATOMIC_RELAXED, __HIP_MEMORY_SCOPE_WORKGROUP);
        const unsigned ekey = (unsigned)lane < nexc ? (unsigned)(ent >> 42) : 99u;
        unsigned m30 = 0, m7 = 0, m6 = 0;
        if (nexc != 15u) {
          unsigned Pa = 0, Pb = 0;
#pragma unroll
          for (int t = 0; t < 4; ++t)
#pragma unroll
            for (int r = 0; r < 3; ++r) {
              const unsigned kk = (r == 2 ? 0x01010000u : 1u << (8 * r)) << t, xa = cw[r][t], xb = ot[r][t];
              Pa |= __builtin_amdgcn_alignbit(xa, xa, 30 - r * 8 - t) & kk;
              Pb |= __builtin_amdgcn_alignbit(xb, xb, 30 - r * 8 - t) & kk;
            }
          const unsigned P = (Pa << (4 * ka)) | (Pb << (4 - 4 * ka));
          *(LAS unsigned*)(scr + lane * 4) = P & 0x7F7F7F7Fu;
          const unsigned cnt = __builtin_amdgcn_udot4((P >> 7) & 0x01010101u, 0x08040201u, 0u, false);
          const unsigned pos = ((wave_scan_incl(cnt) - cnt) & 0x3FFu) + (unsigned)(size_t)scr;
          asm volatile("s_waitcnt lgkmcnt(0)" ::: "memory");
          const LAS unsigned* eq = (const LAS unsigned*)(size_t)(pos & ~3u);
          const unsigned e0 = eq[0], e1 = eq[1], e2 = eq[2];
          const u64 ev = ((u64)__builtin_amdgcn_alignbyte(e2, e1, pos) << 32) | __builtin_amdgcn_alignbyte(e1, e0, pos);
          const u64 keep = cnt >= 8u ? 0x0101010101010101ull : 0x0101010101010101ull & ((1ull << (8 * cnt)) - 1);
          const u64 one = ~(ev >> 6) & keep;
          u64 mm = 0;
#pragma unroll
          for (int i = 0; i < 8; ++i) mm |= ((one >> (8 * i)) & 1u) << ((ev >> (8 * i)) & 63u);
          for (u64 six = one ^ keep; six != 0; six &= six - 1) m6 |= 1u << ((unsigned)(ev >> __builtin_ctzll(six)) & 31u);
          if (__builtin_expect(__builtin_amdgcn_ballot_w64(cnt > 8u) != 0, 0))
            for (unsigned i = 8; i < cnt; ++i) {
              const unsigned e = *(const LAS unsigned char*)(size_t)(pos + i);
              if (e & 64u) m6 |= 1u << (e & 31u); else mm |= 1ull << e;
            }
          asm volatile("s_waitcnt lgkmcnt(0)" ::: "memory");
          m30 = (unsigned)mm >> (4 * ka); m7 = (unsigned)(mm >> 32) >> (4 * ka); m6 >>= 4 * ka;
        }
        __builtin_amdgcn_s_waitcnt(0x0F70);
        for (int k = ka; k < kz; ++k) {
          const int m = mlo + kb + k;
          if (nexc != 15u) {
            unsigned L[4];
#pragma unroll
            for (int t = 0; t < 4; ++t) L[t] = ((m7 << (7 - t)) & 0x80808080u) | ((m6 << (6 - t)) & 0x40404040u);
            for (unsigned em = (unsigned)__builtin_amdgcn_ballot_w64(ekey == 4u + m); em != 0; em &= em - 1) {
              const int e = __builtin_ctz(em);
              const unsigned val = __builtin_amdgcn_readlane((unsigned)ent, e);
              const unsigned key = (__builtin_amdgcn_readlane((unsigned)(ent >> 32), e) & 1023u) - (unsigned)lane * 16;
#pragma unroll
              for (int t = 0; t < 4; ++t) L[t] |= key == (unsigned)t ? val : 0u;
            }
#pragma unroll
            for (int t = 0; t < 4; ++t) {
              cw[3][t] = __builtin_amdgcn_perm(cw[1][t], cw[0][t], 0x0C04000Cu) | __builtin_amdgcn_perm(cw[2][t], L[t], 0x040C0C03u);
#pragma unroll
              for (int r = 0; r < 3; ++r) cw[r][t] = __builtin_amdgcn_perm(cw[r][t], L[t], 0x07060500u + r);
#pragma unroll
              for (int r = 0; r < 4; ++r) {
                const unsigned x = cw[r][t];
                cw[r][t] = as_u((float)(int)((x & ~0x40000000u) | (((x >> 1) ^ (m30 << (30 - r * 8 - t))) & 0x40000000u)));
              }
            }
            for (unsigned em = (unsigned)__builtin_amdgcn_ballot_w64(ekey == (unsigned)m); em != 0; em &= em - 1) {
              const int e = __builtin_ctz(em);
              const unsigned val = __builtin_amdgcn_readlane((unsigned)ent, e);
              const unsigned key = (__builtin_amdgcn_readlane((unsigned)(ent >> 32), e) & 1023u) - (unsigned)lane * 16;
#pragma unroll
              for (int i = 0; i < 16; ++i) cw[i >> 2][i & 3] = key == (unsigned)i ? val : cw[i >> 2][i & 3];
            }
          }
#pragma unroll
          for (int r = 0; r < 4; ++r) {
            f32x4 y;
#pragma unroll
            for (int t = 0; t < 4; ++t) y[t] = silu_dq(as_f(cw[r][t]) + as_f(pa[0][r][t]), sc);
            st16_out(tbase + (size_t)(m * 16 + r) * NFULL, vo, __builtin_bit_cast(u32x4, y));
            pa[0][r] = pa[1][r];
            cw[r] = ot[r];
          }
          m30 >>= 4; m7 >>= 4; m6 >>= 4;
        }
      }
      if (lost) { st_agent(tmo, call); if (lane == 0) p.out[0] = __builtin_nanf(""); }
      if (pp != NOPIECE) { WAIT_VM(0); piece_count(myws, pp, pneed, call, lane); }
    }
  }
}

extern "C" void kernel_launch(int rank, int nranks, ncclComm_t comm,
                              hipStream_t stream,
                              void* const* d_in, const int* in_sizes, int n_in,
                              void* d_out, int out_size,
                              const int* in_dtypes, int out_dtype,
                              void* const* d_out_all,
                              void* const* d_ws_all, size_t ws_bytes) {
  (void)comm; (void)in_dtypes; (void)out_dtype; (void)d_out_all;
  if (nranks != NR || n_in != 4 || ws_bytes < WS_NEED || (unsigned)rank >= (unsigned)NR ||
      in_sizes[0] != 4096 * KSH || in_sizes[1] != KSH * NFULL || out_size != 512 * NFULL) {
    fprintf(stderr, "kernel_launch: unexpected problem shape\n");
    return;
  }
  Params p;
  p.x = (const uint8_t*)d_in[0];
  p.w = (const uint8_t*)d_in[1];
  p.sx = (const float*)d_in[2];
  p.sw = (const float*)d_in[3];
  for (int i = 0; i < NR; ++i) p.ws[i] = (char*)d_ws_all[i];
  for (int i = 0; i < NR; ++i) p.link[i] = p.ws[(rank + 1 + i) & 7];
  p.myws = p.ws[rank];
  p.out = (float*)d_out;
  p.rank = rank;
  p.pad = 0;
  hipLaunchKernelGGL(fused_gemm_rs, dim3(256), dim3(512), 0, stream, p);
  HIP_CHECK(hipGetLastError());
}
```
